```python
import numpy as np

kernel_name = "gpu_gemm_ar_m8192_k8192_n8192_f32_relu"


ABSMAX_RTOL = 0.001

COMPUTE_DTYPE = "float32"
EPILOGUE = "relu"
IN_QUANT = None
OUT_QUANT = None
IN_DTYPES = {"x": "float32", "w_mat": "float32"}
WORLD_SIZE = 8

def setup_inputs():

    x = np.random.randn(8192, 8192).astype(np.float32)
    w_mat = (np.random.randn(8192, 8192) / np.sqrt(8192)).astype(np.float32)
    return {"x": x, "w_mat": w_mat}

def _epilogue(y):
    return np.maximum(y, 0.0).astype(np.float32)

def reference(x, w_mat):
    y = (x @ w_mat).astype(np.float32)
    return _epilogue(y)
```

```cpp
#include <hip/hip_runtime.h>
#include <rccl/rccl.h>
#include <cstdio>

#define HIP_CHECK(expr) do { hipError_t _e = (expr); if (_e != hipSuccess) { \
    fprintf(stderr, "HIP error %s at %s:%d\n", hipGetErrorString(_e), __FILE__, __LINE__); \
    return; } } while (0)

namespace {

typedef _Float16 h8 __attribute__((ext_vector_type(8)));
typedef _Float16 h2 __attribute__((ext_vector_type(2)));
typedef float f4 __attribute__((ext_vector_type(4)));
typedef float f2 __attribute__((ext_vector_type(2)));
typedef unsigned u4 __attribute__((ext_vector_type(4)));
typedef unsigned long long u64;
#define LAS __attribute__((address_space(3)))
extern "C" __device__ int llvm_writelane(int, int, int) __asm("llvm.amdgcn.writelane.i32");

constexpr int NR = 8;
constexpr int GN = 8192, KSH = 1024;
constexpr int NWG = 256, NTH = 512;
constexpr int EPL = 8;
constexpr int NCOMM = 7 * EPL;
constexpr int NX = NCOMM / 8;
constexpr int NGEMM = NWG - NCOMM;
#define WIN 8
constexpr int NBLK = 2048;
constexpr int NITEM = 768;
constexpr int NPKT = 2048;
constexpr int NI_IN = NITEM / EPL, NI = NI_IN + NPKT / EPL, NI_PAD = 384;
constexpr int PKT_BYTES = 512 + 16384;
constexpr int STAGE = 6 * 8192;
constexpr int LDS_BYTES = 3 * STAGE;
constexpr int NSLOT = 8, NWORK = 6;
constexpr int CTL = NSLOT * PKT_BYTES;
static_assert(CTL + NI_PAD * 4 + 64 <= LDS_BYTES && NI <= NI_PAD && NCOMM % 8 == 0, "layout");

constexpr size_t CNT_OFF = 0;
constexpr size_t TMO_OFF = 64 << 10;
constexpr size_t XF_OFF = 128 << 10;
constexpr size_t WF_OFF = 160 << 10;
constexpr size_t RF_OFF = 256 << 10;
constexpr size_t ST_OFF = 512 << 10;
constexpr size_t X16_OFF = 1 << 20;
constexpr size_t W16_OFF = X16_OFF + (32ull << 20);
constexpr size_t S16_OFF = W16_OFF + (64ull << 20);
constexpr size_t PK_OFF = S16_OFF + (32ull << 20);
constexpr size_t WS_NEED = PK_OFF + (size_t)NPKT * PKT_BYTES;
static_assert(RF_OFF + (size_t)NR * EPL * NI_PAD * 8 <= ST_OFF, "flags");
constexpr u64 DEADLINE_TICKS = 50000000ull;

struct Params {
    const float* x; const float* w; float* out;
    unsigned char* ws[NR];
    int rank;
};
#define XMODE 0

#define SYS_LOAD(p) __hip_atomic_load((p), __ATOMIC_RELAXED, __HIP_MEMORY_SCOPE_SYSTEM)
#define SYS_STORE(p, v) __hip_atomic_store((p), (v), __ATOMIC_RELAXED, __HIP_MEMORY_SCOPE_SYSTEM)
#define AG_LOAD(p) __hip_atomic_load((p), __ATOMIC_RELAXED, __HIP_MEMORY_SCOPE_AGENT)
#define AG_STORE(p, v) __hip_atomic_store((p), (v), __ATOMIC_RELAXED, __HIP_MEMORY_SCOPE_AGENT)
#define WAIT_V(n) asm volatile("s_waitcnt vmcnt(" #n ")" ::: "memory")
#define WAIT_L0() asm volatile("s_waitcnt lgkmcnt(0)" ::: "memory")
#define BAR() do { asm volatile("" ::: "memory"); __builtin_amdgcn_s_barrier(); asm volatile("" ::: "memory"); } while (0)
constexpr int AUX_SYS = 17;
constexpr int AUX_AGENT = 16;

#define STAMP(i) do { } while (0)

__device__ __forceinline__ bool give_up(unsigned* tmo, unsigned call, u64 t0) {
    if ((unsigned)__builtin_amdgcn_readfirstlane(AG_LOAD(tmo)) == call) return true;
    if (__builtin_amdgcn_s_memrealtime() - t0 > DEADLINE_TICKS) { AG_STORE(tmo, call); return true; }
    return false;
}

struct Item { int isw, grp, b, kb; };
__device__ __forceinline__ Item decode_item(int it, unsigned dst) {
    Item r; int s;
    if (it < 256) { s = it >> 1; r.isw = it & 1; }
    else if (it < 384) { s = it - 128; r.isw = 0; }
    else { s = it - 256; r.isw = 1; }
    r.b = s >> 3; r.kb = s & 7; r.grp = r.isw ? (int)(dst & 1) : (int)(dst >> 1);
    return r;
}
__device__ __forceinline__ int encode_item(const Item& I) { const int s = I.b * 8 + I.kb; return I.b < 16 ? 2 * s + I.isw : s + (I.isw ? 256 : 128); }
__device__ __forceinline__ Item decode_block(int u) {
    Item r; int s;
    if (u < 768) { s = u / 6; const int e = u - s * 6; r.isw = e >= 4; r.grp = e >= 4 ? e - 4 : e; }
    else if (u < 1280) { const int t = u - 768; s = 128 + (t >> 2); r.isw = 0; r.grp = t & 3; }
    else { const int t = u - 1280; s = 128 + (t >> 1); r.isw = 1; r.grp = t & 1; }
    r.b = s >> 3; r.kb = s & 7;
    return r;
}
__device__ __forceinline__ size_t staged_off(const Item& I) { return S16_OFF + (size_t)(I.isw ? 1024 + (I.grp * 64 + I.b) * 8 + I.kb : (I.grp * 32 + I.b) * 8 + I.kb) * 16384; }
__device__ __forceinline__ size_t ready_off(int src, int idx, int first) { return RF_OFF + ((size_t)(src * EPL + (idx & (EPL - 1))) * NI_PAD + first + idx / EPL) * 8; }
__device__ __forceinline__ void load_item(const Params& P, const Item& I, int lane, f4 (&v)[32]) {
    const int m = lane & 15, q = lane >> 4;
    const unsigned base = I.isw ? (unsigned)(I.kb * 128 * GN + I.grp * 4096 + I.b * 64) * 4u : (unsigned)((I.grp * 2048 + I.b * 64) * KSH + I.kb * 128) * 4u;
    const unsigned voff = I.isw ? (unsigned)(4 * q * GN + 4 * m) * 4u : (unsigned)(m * KSH + 4 * q) * 4u;
    const __amdgpu_buffer_rsrc_t rs = __builtin_amdgcn_make_buffer_rsrc((void*)(I.isw ? P.w : P.x), 0, KSH * GN * 4, 0x00020000);
#pragma unroll
    for (int n = 0; n < 32; ++n) {
        const unsigned ox = ((n >> 1) & 3) * (16 * KSH * 4) + (n >> 3) * 128 + (n & 1) * 64;
        const unsigned ow = ((n >> 3) * 32 + ((n & 7) < 4 ? (n & 7) : 12 + (n & 7))) * (GN * 4);
        v[n] = __builtin_bit_cast(f4, __builtin_amdgcn_raw_buffer_load_b128(rs, voff, base + (I.isw ? ow : ox), 0));
    }
}
__device__ __forceinline__ unsigned pk16(float a, float b) {
    const f2 p = {a, b};
    return __builtin_bit_cast(unsigned, __builtin_convertvector(p, h2));
}
__device__ __forceinline__ void convert_phase(const Params& P, int gw, unsigned call, int lane) {
    unsigned char* ws = P.ws[P.rank];
    if (gw >= NGEMM) __builtin_amdgcn_s_sleep(80);
    for (int u = gw; u < NBLK; u += 8 * NGEMM) {
        f4 v[32];
        const Item I = decode_block(u);
        load_item(P, I, lane, v);
        float fm = 0.f;
#pragma unroll
        for (int i = 0; i < 32; ++i) fm = fmaxf(fmaxf(fm, fmaxf(fabsf(v[i][0]), fabsf(v[i][1]))), fmaxf(fabsf(v[i][2]), fabsf(v[i][3])));
        unsigned mx = __builtin_bit_cast(unsigned, fm);
        if (__any(mx >= 0x7f800000u)) {
            mx = 0;
#pragma unroll
            for (int i = 0; i < 32; ++i)
#pragma unroll
                for (int c = 0; c < 4; ++c) { const unsigned u = __builtin_bit_cast(unsigned, v[i][c]) & 0x7fffffffu; mx = max(mx, u < 0x7f800000u ? u : 0u); }
        }
#pragma unroll
        for (int o = 32; o >= 1; o >>= 1) mx = max(mx, (unsigned)__shfl_xor((int)mx, o));
        const int E = (int)((unsigned)__builtin_amdgcn_readfirstlane(mx) >> 23) - 127;
        const int code = ((14 - E) >> 3) + 16;
        const int s = (code - 16) * 8;
#pragma unroll
        for (int i = 0; i < 32; ++i) {
#pragma unroll
            for (int c = 0; c < 4; ++c) v[i][c] = __builtin_amdgcn_ldexpf(v[i][c], s);
            asm volatile("" : "+v"(v[i]));
        }
        u4 h[16];
        if (!I.isw) {
#pragma unroll
            for (int f = 0; f < 16; ++f) { const f4 a = v[2 * f], b = v[2 * f + 1];
                h[f] = (u4){pk16(a[0], a[1]), pk16(a[2], a[3]), pk16(b[0], b[1]), pk16(b[2], b[3])};
                asm volatile("; x" : "+v"(h[f])); }
        } else {
#pragma unroll
            for (int kq = 0; kq < 4; ++kq)
#pragma unroll
                for (int i = 0; i < 4; ++i) {
                    h[kq * 4 + i] = (u4){pk16(v[kq * 8 + 0][i], v[kq * 8 + 1][i]), pk16(v[kq * 8 + 2][i], v[kq * 8 + 3][i]),
                                         pk16(v[kq * 8 + 4][i], v[kq * 8 + 5][i]), pk16(v[kq * 8 + 6][i], v[kq * 8 + 7][i])};
                    asm volatile("; w" : "+v"(h[kq * 4 + i])); }
        }
        const __amdgpu_buffer_rsrc_t rs = __builtin_amdgcn_make_buffer_rsrc(ws + staged_off(I), 0, 16384, 0x00020000);
#pragma unroll
        for (int f = 0; f < 16; ++f) __builtin_amdgcn_raw_buffer_store_b128(h[f], rs, lane * 16 + f * 1024, 0, AUX_SYS);
        const bool mine = I.grp == (I.isw ? (P.rank & 1) : (P.rank >> 1));
        const int blk = I.b * 64 + P.rank * 8 + I.kb;
        if (mine) {
            const __amdgpu_buffer_rsrc_t r2 = __builtin_amdgcn_make_buffer_rsrc(ws + (I.isw ? W16_OFF : X16_OFF) + (size_t)blk * 16384, 0, 16384, 0x00020000);
#pragma unroll
            for (int f = 0; f < 16; ++f) __builtin_amdgcn_raw_buffer_store_b128(h[f], r2, lane * 16 + f * 1024, 0, AUX_AGENT);
        }
        WAIT_V(0);
        if (lane == 0) {
            const int nd = I.isw ? 4 : 2, d0 = I.isw ? I.grp : 2 * I.grp, dstep = I.isw ? 2 : 1;
            const size_t fo = ready_off(P.rank, encode_item(I), 0);
            for (int d = 0; d < nd; ++d) {
                const unsigned dst = (unsigned)(d0 + d * dstep) & (NR - 1);
                if (dst != (unsigned)P.rank) SYS_STORE((u64*)(P.ws[dst] + fo), ((u64)call << 32) | (unsigned)code);
            }
            if (mine) AG_STORE((unsigned*)(ws + (I.isw ? WF_OFF : XF_OFF) + (size_t)blk * 4), (call << 8) | (unsigned)code);
        }
    }
}

__device__ __forceinline__ size_t packet_origin(int g, int pk) {
    const int t = pk >> 3, wv = pk & 7, patch = t >> 5;
    const int row = (g >> 1) * 2048 + ((patch & 1) * 4 + (t & 3)) * 256 + (wv >> 1) * 64;
    const int col = (g & 1) * 4096 + ((patch >> 1) * 8 + ((t & 31) >> 2)) * 128 + (wv & 1) * 64;
    return (size_t)row * GN + col;
}

__device__ __forceinline__ unsigned lds_get(LAS unsigned char* p) {
    unsigned v;
    asm volatile("ds_read_b32 %0, %1\n\ts_waitcnt lgkmcnt(0)" : "=v"(v) : "v"(p) : "memory");
    return (unsigned)__builtin_amdgcn_readfirstlane(v);
}
__device__ __forceinline__ void lds_put(LAS unsigned char* p, unsigned v) { asm volatile("ds_write_b32 %0, %1" :: "v"(p), "v"(v) : "memory"); }
constexpr unsigned DEAD = 0xffffffffu;

__device__ __forceinline__ void comm_wg(const Params& P, int b, int wid, unsigned call, int lane, LAS unsigned char* lds) {
    const unsigned src = (unsigned)(P.rank + 1 + b % 7) & (NR - 1);
    const int k = b / 7;
    unsigned char* ws = P.ws[P.rank];
    LAS unsigned char* info = lds + CTL;
    LAS unsigned char* full = info + NI_PAD * 4;
    LAS unsigned char* freed = full + 32;
    if (threadIdx.x < NI_PAD + 16) lds_put(info + threadIdx.x * 4, 0u);
    WAIT_L0();
    BAR();
    const int n0 = XMODE == 2 ? NI_IN : 0, n1 = XMODE == 1 ? NI_IN : NI;
    if (wid == 1) {
        unsigned* tmo = (unsigned*)(ws + TMO_OFF);
        const u64* flags = (const u64*)(ws + RF_OFF) + (size_t)(src * EPL + k) * NI_PAD;
        const u64 t0 = __builtin_amdgcn_s_memrealtime();
        bool dead = false;
        for (int base = n0; base < n1; base += 64) {
            const int n = base + lane;
            bool todo = n < n1;
            for (;;) {
                if (todo) {
                    const u64 f = SYS_LOAD(flags + n);
                    if ((unsigned)(f >> 32) == call || XMODE == 2) { lds_put(info + n * 4, (unsigned)f); todo = false; }
                    else if (dead) { lds_put(info + n * 4, DEAD); todo = false; }
                }
                if (!__any(todo)) break;
                if (give_up(tmo, call, t0)) dead = true; else __builtin_amdgcn_s_sleep(8);
            }
        }
    } else if (wid == 0) {
        const unsigned char* rws = P.ws[src] + lane * 16;
        int issued = 0, pub = n0, endv = 0;
        auto publish = [&](int landed, int nissued) {
            while (pub < nissued && __builtin_amdgcn_readlane(endv, pub & 63) <= landed) { lds_put(full + (pub & (NSLOT - 1)) * 4, (unsigned)pub + 1u); ++pub; }
        };
        for (int n = n0; n < n1; ++n) {
            const unsigned want = (unsigned)(n - n0 < NSLOT ? 0 : n - NSLOT + 1);
            unsigned inf = lds_get(info + n * 4);
            if (inf == 0u || lds_get(freed + (n & (NSLOT - 1)) * 4) != want) {
                WAIT_V(0);
                publish(issued, n);
                while ((inf = lds_get(info + n * 4)) == 0u || lds_get(freed + (n & (NSLOT - 1)) * 4) != want) __builtin_amdgcn_s_sleep(1);
            }
            unsigned bytes; const unsigned char* g;
            if (n < NI_IN) { bytes = 16384u; g = rws + staged_off(decode_item(n * EPL + k, (unsigned)P.rank)); }
            else { bytes = min(inf, (unsigned)PKT_BYTES); g = rws + PK_OFF + (size_t)((n - NI_IN) * EPL + k) * PKT_BYTES; }
            if (inf == DEAD) bytes = 0u;
            LAS unsigned char* d = lds + (unsigned)(n & (NSLOT - 1)) * PKT_BYTES;
            for (unsigned o = 0; o < bytes; o += 1024) {
                asm volatile("s_waitcnt vmcnt(%0)" :: "i"(WIN - 1) : "memory");
                if (o + lane * 16 < bytes) __builtin_amdgcn_global_load_lds((const unsigned*)(g + o), (LAS unsigned*)(d + o), 16, 0, AUX_SYS);
                publish(issued - (WIN - 1), n);
                ++issued;
            }
            endv = llvm_writelane(issued, n & 63, endv);
        }
        WAIT_V(0);
        publish(issued, n1);
    } else {
        const int m = lane & 15, q = lane >> 4;
        for (int n = n0 + wid - 2; n < n1; n += NWORK) {
            const unsigned slot = (unsigned)(n & (NSLOT - 1));
            while (lds_get(full + slot * 4) != (unsigned)n + 1u) __builtin_amdgcn_s_sleep(2);
            const unsigned inf = lds_get(info + n * 4);
            LAS unsigned char* d = lds + slot * PKT_BYTES;
            if (n < NI_IN) {
                u4 h[16];
#pragma unroll
                for (int f = 0; f < 16; ++f) h[f] = *(LAS u4*)(d + f * 1024 + lane * 16);
                WAIT_L0();
                lds_put(freed + slot * 4, (unsigned)n + 1u);
                if (inf != DEAD) {
                    const Item I = decode_item(n * EPL + k, (unsigned)P.rank);
                    const int blk = I.b * 64 + (int)src * 8 + I.kb;
                    const __amdgpu_buffer_rsrc_t rs = __builtin_amdgcn_make_buffer_rsrc(ws + (I.isw ? W16_OFF : X16_OFF) + (size_t)blk * 16384, 0, 16384, 0x00020000);
#pragma unroll
                    for (int f = 0; f < 16; ++f) __builtin_amdgcn_raw_buffer_store_b128(h[f], rs, lane * 16 + f * 1024, 0, AUX_AGENT);
                    WAIT_V(0);
                    if (lane == 0) AG_STORE((unsigned*)(ws + (I.isw ? WF_OFF : XF_OFF) + (size_t)blk * 4), (call << 8) | (inf & 63u));
                }
            } else {
                float* o = P.out + packet_origin((int)src, (n - NI_IN) * EPL + k) + (size_t)(4 * q) * GN + 4 * m;
                if (inf != DEAD) {
                    const u64 mk = *(LAS u64*)(d + lane * 8);
                    const int mlo = (int)mk, mhi = (int)(mk >> 32);
                    unsigned base = 512;
#pragma unroll
                    for (int g4 = 0; g4 < 16; ++g4) {
                        f4 t;
#pragma unroll
                        for (int ni = 0; ni < 4; ++ni) {
                            const unsigned lo = (unsigned)__builtin_amdgcn_readlane(mlo, g4 * 4 + ni), hi = (unsigned)__builtin_amdgcn_readlane(mhi, g4 * 4 + ni);
                            const u64 mm = ((u64)hi << 32) | lo;
                            const unsigned pos = __builtin_amdgcn_mbcnt_hi(hi, __builtin_amdgcn_mbcnt_lo(lo, 0u));
                            const float val = *(LAS float*)(d + base + pos * 4);
                            t[ni] = __builtin_amdgcn_inverse_ballot_w64(mm) ? val : 0.f;
                            base += 4u * (unsigned)__builtin_popcountll(mm);
                        }
                        *(f4*)(o + (size_t)((g4 >> 2) * 16 + (g4 & 3)) * GN) = t;
                    }
                } else {
                    const float qn = __builtin_nanf("");
#pragma unroll
                    for (int g4 = 0; g4 < 16; ++g4) *(f4*)(o + (size_t)((g4 >> 2) * 16 + (g4 & 3)) * GN) = (f4){qn, qn, qn, qn};
                }
                WAIT_L0();
                lds_put(freed + slot * 4, (unsigned)n + 1u);
            }
        }
    }
}

__device__ __forceinline__ void gemm_tile(const Params& P, int tile, int wid, unsigned call, int lane, LAS unsigned char* lds, int stamp0) {
    unsigned char* ws = P.ws[P.rank];
    unsigned* tmo = (unsigned*)(ws + TMO_OFF);
    const int patch = tile >> 5;
    const int rb0 = ((patch & 1) * 4 + (tile & 3)) * 4;
    const int cb0 = ((patch >> 1) * 8 + ((tile & 31) >> 2)) * 2;
    const unsigned* xf = (const unsigned*)(ws + XF_OFF) + rb0 * 64;
    const unsigned* wf = (const unsigned*)(ws + WF_OFF) + cb0 * 64;
    if (wid == 0) {
        const u64 t0 = __builtin_amdgcn_s_memrealtime();
        bool dead = false;
        for (int g = 0; g < 6 && !dead; ++g) {
            const unsigned* f = (g < 4 ? xf + g * 64 : wf + (g - 4) * 64) + lane;
            while (!__all((SYS_LOAD(f) >> 8) == (call & 0xffffffu))) {
                if (give_up(tmo, call, t0)) { dead = true; break; }
                __builtin_amdgcn_s_sleep(32);
            }
        }
        __builtin_amdgcn_fence(__ATOMIC_ACQUIRE, "");
        WAIT_V(0);
    }
    BAR();
    STAMP(stamp0);
    const int wr = wid >> 1, wc = wid & 1;
    const bool bad = (unsigned)__builtin_amdgcn_readfirstlane(AG_LOAD(tmo)) == call;
    const int ev = 8 * ((int)(SYS_LOAD(xf + wr * 64 + lane) & 63u) + (int)(SYS_LOAD(wf + wc * 64 + lane) & 63u) - 32);
    WAIT_V(0);

    const unsigned char* ga = ws + X16_OFF + (size_t)rb0 * (1 << 20) + wid * 1024 + lane * 16;
    const unsigned char* gb = ws + W16_OFF + (size_t)cb0 * (1 << 20) + wid * 1024 + lane * 16;
    auto stage = [&](int kt, int sb) {
        LAS unsigned char* d = lds + (unsigned)sb * STAGE + (unsigned)wid * 1024u;
        const size_t ko = (size_t)(kt & 127) * 8192;
#pragma unroll
        for (int p = 0; p < 4; ++p) __builtin_amdgcn_global_load_lds((const unsigned*)(ga + (size_t)p * (1 << 20) + ko), (LAS unsigned*)(d + p * 8192), 16, 0, 0);
#pragma unroll
        for (int p = 0; p < 2; ++p) __builtin_amdgcn_global_load_lds((const unsigned*)(gb + (size_t)p * (1 << 20) + ko), (LAS unsigned*)(d + (4 + p) * 8192), 16, 0, 0);
    };
    f4 acc[4][4];
#pragma unroll
    for (int i = 0; i < 4; ++i)
#pragma unroll
        for (int j = 0; j < 4; ++j) acc[i][j] = (f4){0.f, 0.f, 0.f, 0.f};
    const int pk = tile * 8 + wid;
    float* o = P.out + packet_origin(P.rank, pk) + (size_t)(4 * (lane >> 4)) * GN + 4 * (lane & 15);
    bool spilled = false;
    stage(0, 0); stage(1, 1);
    int sb = 0, kb = 0, ecur = 0;
    const unsigned aoff = (unsigned)wr * 8192u + lane * 16u, boff = (unsigned)(4 + wc) * 8192u + lane * 16u;
    while (kb < 64) {
        ecur = __builtin_amdgcn_readlane(ev, kb);
        const u64 diff = __builtin_amdgcn_ballot_w64(ev != ecur) >> kb;
        const int kend = diff ? kb + (int)__builtin_ctzll(diff) : 64;
        for (int t = 2 * kb; t < 2 * kend; ++t) {
            WAIT_V(6);
            BAR();
            const int s2 = sb == 0 ? 2 : sb - 1;
            stage(t + 2, s2);
            LAS unsigned char* s = lds + (unsigned)sb * STAGE;
            h8 a[2][4], b[2][4];
#pragma unroll
            for (int kq = 0; kq < 2; ++kq)
#pragma unroll
                for (int i = 0; i < 4; ++i) { b[kq][i] = *(LAS h8*)(s + boff + (kq * 4 + i) * 1024); a[kq][i] = *(LAS h8*)(s + aoff + (kq * 4 + i) * 1024); }
            __builtin_amdgcn_sched_barrier(0);
#pragma unroll
            for (int kq = 0; kq < 2; ++kq)
#pragma unroll
                for (int i = 0; i < 4; ++i)
#pragma unroll
                    for (int j = 0; j < 4; ++j) acc[i][j] = __builtin_amdgcn_mfma_f32_16x16x32_f16(a[kq][i], b[kq][j], acc[i][j], 0, 0, 0);
            sb = sb == 2 ? 0 : sb + 1;
        }
        kb = kend;
        if (kb < 64) {
#pragma unroll
            for (int g4 = 0; g4 < 16; ++g4) {
                f4* p = (f4*)(o + (size_t)((g4 >> 2) * 16 + (g4 & 3)) * GN);
                f4 t;
#pragma unroll
                for (int ni = 0; ni < 4; ++ni) t[ni] = __builtin_amdgcn_ldexpf(acc[g4 >> 2][ni][g4 & 3], -ecur);
                if (spilled) t += *p;
                *p = t;
            }
#pragma unroll
            for (int i = 0; i < 4; ++i)
#pragma unroll
                for (int j = 0; j < 4; ++j) acc[i][j] = (f4){0.f, 0.f, 0.f, 0.f};
            spilled = true;
        }
    }
    WAIT_V(0);
    BAR();
    STAMP(stamp0 + 1);
    LAS unsigned char* d = lds + (unsigned)wid * PKT_BYTES;
    int mlo = 0, mhi = 0;
    unsigned base = 512;
#pragma unroll
    for (int g4 = 0; g4 < 16; ++g4) {
        f4* p = (f4*)(o + (size_t)((g4 >> 2) * 16 + (g4 & 3)) * GN);
        f4 t;
#pragma unroll
        for (int ni = 0; ni < 4; ++ni) t[ni] = __builtin_amdgcn_ldexpf(acc[g4 >> 2][ni][g4 & 3], -ecur);
        if (spilled) t += *p;
#pragma unroll
        for (int ni = 0; ni < 4; ++ni) {
            float val = bad ? __builtin_nanf("") : t[ni];
            const bool nz = !(val <= 0.f);
            val = nz ? val : 0.f;
            t[ni] = val;
            const u64 mm = __builtin_amdgcn_ballot_w64(nz);
            const unsigned pos = __builtin_amdgcn_mbcnt_hi((unsigned)(mm >> 32), __builtin_amdgcn_mbcnt_lo((unsigned)mm, 0u));
            if (nz) *(LAS float*)(d + base + pos * 4) = val;
            mlo = llvm_writelane((int)mm, g4 * 4 + ni, mlo);
            mhi = llvm_writelane((int)(mm >> 32), g4 * 4 + ni, mhi);
            base += 4u * (unsigned)__builtin_popcountll(mm);
        }
        *p = t;
    }
    *(LAS u64*)(d + lane * 8) = ((u64)(unsigned)mhi << 32) | (unsigned)mlo;
    WAIT_L0();
    const unsigned bytes = (base + 15u) & ~15u;
    const __amdgpu_buffer_rsrc_t rs = __builtin_amdgcn_make_buffer_rsrc(ws + PK_OFF + (size_t)pk * PKT_BYTES, 0, PKT_BYTES, 0x00020000);
    for (unsigned ofs = lane * 16u; ofs < bytes; ofs += 1024) __builtin_amdgcn_raw_buffer_store_b128(*(LAS u4*)(d + ofs), rs, ofs, 0, AUX_SYS);
    WAIT_V(0);
    if (lane == 0) {
        const size_t fo = ready_off(P.rank, pk, NI_IN);
        for (int i = 1; i < NR; ++i) SYS_STORE((u64*)(P.ws[(unsigned)(P.rank + i) & (NR - 1)] + fo), ((u64)call << 32) | bytes);
    }
    WAIT_L0();
    BAR();
    STAMP(stamp0 + 2);
}

__global__ __launch_bounds__(NTH, 2) void fused_gemm_gather(Params P) {
    extern __shared__ __attribute__((aligned(16))) unsigned char smem[];
    LAS unsigned char* lds = (LAS unsigned char*)smem;
    const int b = blockIdx.x, lane = threadIdx.x & 63;
    const int wid = __builtin_amdgcn_readfirstlane(threadIdx.x >> 6);
    unsigned call = 0;
    if (lane == 0) call = __hip_atomic_fetch_add((unsigned*)(P.ws[P.rank] + CNT_OFF + (size_t)(b * 8 + wid) * 16), 1u, __ATOMIC_RELAXED, __HIP_MEMORY_SCOPE_AGENT) + 1u;
    STAMP(0);
    call = (unsigned)__builtin_amdgcn_readfirstlane(call);
    if (b < NCOMM) {
        comm_wg(P, b, wid, call, lane, lds);
        STAMP(2);
    } else {
        if (XMODE != 2) convert_phase(P, wid * NGEMM + (b - NCOMM), call, lane);
        STAMP(7);
        const int patch = b & 7, slot = (b >> 3) - NCOMM / 8;
        const int own = patch * 32 + slot, extra = ((patch + 4) & 7) * 32 + (32 - NX) + slot;
        const int n = XMODE ? 0 : slot < NX ? 2 : 1;
        for (int i = 0; i < n; ++i) {
            const int tile = (n == 2 && ((patch >= 4) == (i == 0))) ? extra : own;
            gemm_tile(P, tile, wid, call, lane, lds, 1 + 3 * i);
        }
    }
}

}

extern "C" void kernel_launch(int rank, int nranks, ncclComm_t comm, hipStream_t stream,
                              void* const* d_in, const int* in_sizes, int n_in,
                              void* d_out, int out_size, const int* in_dtypes, int out_dtype,
                              void* const* d_out_all, void* const* d_ws_all, size_t ws_bytes) {
    (void)comm; (void)in_sizes; (void)n_in; (void)out_size; (void)in_dtypes; (void)out_dtype; (void)d_out_all;
    if (nranks != NR || (unsigned)rank >= (unsigned)NR || ws_bytes < WS_NEED) { fprintf(stderr, "unsupported configuration\n"); return; }
    HIP_CHECK(hipFuncSetAttribute((const void*)fused_gemm_gather, hipFuncAttributeMaxDynamicSharedMemorySize, LDS_BYTES));
    Params P;
    P.x = (const float*)d_in[0]; P.w = (const float*)d_in[1]; P.out = (float*)d_out; P.rank = rank;
    for (int p = 0; p < NR; ++p) P.ws[p] = (unsigned char*)d_ws_all[p];
    hipLaunchKernelGGL(fused_gemm_gather, dim3(NWG), dim3(NTH), LDS_BYTES, stream, P);
    HIP_CHECK(hipGetLastError());
}
```

```cpp
#include <hip/hip_runtime.h>
#include <rccl/rccl.h>
#include <cstdio>

#define HIP_CHECK(expr) do { hipError_t _e = (expr); if (_e != hipSuccess) { \
    fprintf(stderr, "HIP error %s at %s:%d\n", hipGetErrorString(_e), __FILE__, __LINE__); \
    return; } } while (0)

namespace {

typedef _Float16 h8 __attribute__((ext_vector_type(8)));
typedef _Float16 h2 __attribute__((ext_vector_type(2)));
typedef float f4 __attribute__((ext_vector_type(4)));
typedef float f2 __attribute__((ext_vector_type(2)));
typedef unsigned u4 __attribute__((ext_vector_type(4)));
typedef unsigned long long u64;
#define LAS __attribute__((address_space(3)))
extern "C" __device__ int llvm_writelane(int, int, int) __asm("llvm.amdgcn.writelane.i32");

constexpr int NR = 8;
constexpr int GN = 8192, KSH = 1024;
constexpr int NWG = 256, NTH = 512;
constexpr int EPL = 8;
constexpr int NCOMM = 7 * EPL;
constexpr int NX = NCOMM / 8;
constexpr int NGEMM = NWG - NCOMM;
#define WIN 9
constexpr int NBLK = 2048;
constexpr int NITEM = 768;
constexpr int NPKT = 2048;
constexpr int NTOT = NITEM + NPKT;
#define DRAW 2
#define NPUSH 16
#define OUT_AUX 18
#define LOWQ 3
constexpr int PKT_BYTES = 512 + 16384;
#define PKT_STRIDE PKT_BYTES
constexpr int STAGE = 6 * 8192;
constexpr int LDS_BYTES = 152 * 1024;
constexpr int NSLOT = 8, NWORK = 6;
constexpr int SLOT_BYTES = 17 * 1024;
constexpr int CTL = NSLOT * SLOT_BYTES;
constexpr int CTL_WORDS = 32 + 8 + 8 + 4 + NTOT;
static_assert(CTL + CTL_WORDS * 4 <= LDS_BYTES && 3 * STAGE <= LDS_BYTES && 8 * PKT_BYTES <= LDS_BYTES && NCOMM % 8 == 0 && NTOT % 64 == 0 && NTOT % DRAW == 0 && NPUSH % DRAW == 0 && NPUSH < 64 && LOWQ + DRAW <= 16, "layout");

constexpr size_t CNT_OFF = 0;
constexpr size_t TMO_OFF = 64 << 10;
constexpr size_t LQ_OFF = 96 << 10;
constexpr size_t XF_OFF = 128 << 10;
constexpr size_t WF_OFF = 160 << 10;
constexpr size_t RF_OFF = 256 << 10;
constexpr size_t ST_OFF = 512 << 10;
constexpr size_t X16_OFF = 1 << 20;
constexpr size_t W16_OFF = X16_OFF + (32ull << 20);
constexpr size_t S16_OFF = W16_OFF + (64ull << 20);
constexpr size_t PK_OFF = S16_OFF + (32ull << 20);
constexpr size_t WS_NEED = PK_OFF + (size_t)NPKT * PKT_STRIDE;
static_assert(RF_OFF + (size_t)NR * NTOT * 8 <= ST_OFF, "flags");
constexpr u64 DEADLINE_TICKS = 50000000ull;

struct Params {
    const float* x; const float* w; float* out;
    unsigned char* ws[NR];
    int rank;
};
#define XMODE 0

#define SYS_LOAD(p) __hip_atomic_load((p), __ATOMIC_RELAXED, __HIP_MEMORY_SCOPE_SYSTEM)
#define SYS_STORE(p, v) __hip_atomic_store((p), (v), __ATOMIC_RELAXED, __HIP_MEMORY_SCOPE_SYSTEM)
#define AG_LOAD(p) __hip_atomic_load((p), __ATOMIC_RELAXED, __HIP_MEMORY_SCOPE_AGENT)
#define AG_STORE(p, v) __hip_atomic_store((p), (v), __ATOMIC_RELAXED, __HIP_MEMORY_SCOPE_AGENT)
#define WAIT_V(n) asm volatile("s_waitcnt vmcnt(" #n ")" ::: "memory")
#define WAIT_L0() asm volatile("s_waitcnt lgkmcnt(0)" ::: "memory")
#define BAR() do { asm volatile("" ::: "memory"); __builtin_amdgcn_s_barrier(); asm volatile("" ::: "memory"); } while (0)
constexpr int AUX_SYS = 17;
constexpr int AUX_AGENT = 16;

#define TNOW() 0ull
#define STAMPV(i, v) do { } while (0)
#define STAMP(i) do { } while (0)

__device__ __forceinline__ bool give_up(unsigned* tmo, unsigned call, u64 t0) {
    if ((unsigned)__builtin_amdgcn_readfirstlane(AG_LOAD(tmo)) == call) return true;
    if (__builtin_amdgcn_s_memrealtime() - t0 > DEADLINE_TICKS) { AG_STORE(tmo, call); return true; }
    return false;
}

struct Item { int isw, grp, b, kb; };
__device__ __forceinline__ Item decode_item(int it, unsigned dst) {
    Item r; int s;
    if (it < 256) { s = it >> 1; r.isw = it & 1; }
    else if (it < 384) { s = it - 128; r.isw = 0; }
    else { s = it - 256; r.isw = 1; }
    r.b = s >> 3; r.kb = s & 7; r.grp = r.isw ? (int)(dst & 1) : (int)(dst >> 1);
    return r;
}
__device__ __forceinline__ int encode_item(const Item& I) { const int s = I.b * 8 + I.kb; return I.b < 16 ? 2 * s + I.isw : s + (I.isw ? 256 : 128); }
__device__ __forceinline__ Item decode_block(int u) {
    Item r; int s;
    if (u < 768) { s = u / 6; const int e = u - s * 6; r.isw = e >= 4; r.grp = e >= 4 ? e - 4 : e; }
    else if (u < 1280) { const int t = u - 768; s = 128 + (t >> 2); r.isw = 0; r.grp = t & 3; }
    else { const int t = u - 1280; s = 128 + (t >> 1); r.isw = 1; r.grp = t & 1; }
    r.b = s >> 3; r.kb = s & 7;
    return r;
}
__device__ __forceinline__ size_t staged_off(const Item& I) { return S16_OFF + (size_t)(I.isw ? 1024 + (I.grp * 64 + I.b) * 8 + I.kb : (I.grp * 32 + I.b) * 8 + I.kb) * 16384; }
__device__ __forceinline__ size_t ready_off(int src, int id) { return RF_OFF + ((size_t)src * NTOT + id) * 8; }
__device__ __forceinline__ void load_item(const Params& P, const Item& I, int lane, f4 (&v)[32]) {
    const int m = lane & 15, q = lane >> 4;
    const unsigned base = I.isw ? (unsigned)(I.kb * 128 * GN + I.grp * 4096 + I.b * 64) * 4u : (unsigned)((I.grp * 2048 + I.b * 64) * KSH + I.kb * 128) * 4u;
    const unsigned voff = I.isw ? (unsigned)(4 * q * GN + 4 * m) * 4u : (unsigned)(m * KSH + 4 * q) * 4u;
    const __amdgpu_buffer_rsrc_t rs = __builtin_amdgcn_make_buffer_rsrc((void*)(I.isw ? P.w : P.x), 0, KSH * GN * 4, 0x00020000);
#pragma unroll
    for (int n = 0; n < 32; ++n) {
        const unsigned ox = ((n >> 1) & 3) * (16 * KSH * 4) + (n >> 3) * 128 + (n & 1) * 64;
        const unsigned ow = ((n >> 3) * 32 + ((n & 7) < 4 ? (n & 7) : 12 + (n & 7))) * (GN * 4);
        v[n] = __builtin_bit_cast(f4, __builtin_amdgcn_raw_buffer_load_b128(rs, voff, base + (I.isw ? ow : ox), 0));
    }
}
__device__ __forceinline__ unsigned pk16(float a, float b) {
    const f2 p = {a, b};
    return __builtin_bit_cast(unsigned, __builtin_convertvector(p, h2));
}
__device__ __forceinline__ void convert_phase(const Params& P, int gw, unsigned call  , int lane) {
    unsigned char* ws = P.ws[P.rank];
    if (gw >= NGEMM) __builtin_amdgcn_s_sleep(80);
    for (int u = gw; u < NBLK; u += 8 * NGEMM) {
        f4 v[32];
        const Item I = decode_block(u);
        load_item(P, I, lane, v);
        float fm = 0.f;
#pragma unroll
        for (int i = 0; i < 32; ++i) fm = fmaxf(fmaxf(fm, fmaxf(fabsf(v[i][0]), fabsf(v[i][1]))), fmaxf(fabsf(v[i][2]), fabsf(v[i][3])));
        unsigned mx = __builtin_bit_cast(unsigned, fm);
        if (__any(mx >= 0x7f800000u)) {
            mx = 0;
#pragma unroll
            for (int i = 0; i < 32; ++i)
#pragma unroll
                for (int c = 0; c < 4; ++c) { const unsigned u = __builtin_bit_cast(unsigned, v[i][c]) & 0x7fffffffu; mx = max(mx, u < 0x7f800000u ? u : 0u); }
        }
#pragma unroll
        for (int o = 32; o >= 1; o >>= 1) mx = max(mx, (unsigned)__shfl_xor((int)mx, o));
        const int E = (int)((unsigned)__builtin_amdgcn_readfirstlane(mx) >> 23) - 127;
        const int code = ((14 - E) >> 3) + 16;
        const int s = (code - 16) * 8;
#pragma unroll
        for (int i = 0; i < 32; ++i) {
#pragma unroll
            for (int c = 0; c < 4; ++c) v[i][c] = __builtin_amdgcn_ldexpf(v[i][c], s);
            asm volatile("" : "+v"(v[i]));
        }
        u4 h[16];
        if (!I.isw) {
#pragma unroll
            for (int f = 0; f < 16; ++f) { const f4 a = v[2 * f], b = v[2 * f + 1];
                h[f] = (u4){pk16(a[0], a[1]), pk16(a[2], a[3]), pk16(b[0], b[1]), pk16(b[2], b[3])};
                asm volatile("; x" : "+v"(h[f])); }
        } else {
#pragma unroll
            for (int kq = 0; kq < 4; ++kq)
#pragma unroll
                for (int i = 0; i < 4; ++i) {
                    h[kq * 4 + i] = (u4){pk16(v[kq * 8 + 0][i], v[kq * 8 + 1][i]), pk16(v[kq * 8 + 2][i], v[kq * 8 + 3][i]),
                                         pk16(v[kq * 8 + 4][i], v[kq * 8 + 5][i]), pk16(v[kq * 8 + 6][i], v[kq * 8 + 7][i])};
                    asm volatile("; w" : "+v"(h[kq * 4 + i])); }
        }
        const int blk = I.b * 64 + P.rank * 8 + I.kb, item = encode_item(I);
        const size_t doff = (I.isw ? W16_OFF : X16_OFF) + (size_t)blk * 16384, foff = (I.isw ? WF_OFF : XF_OFF) + (size_t)blk * 4;
        const int nd = I.isw ? 4 : 2, d0 = I.isw ? I.grp : 2 * I.grp, dstep = I.isw ? 2 : 1;
        if (item < NPUSH) {
            for (int d = 0; d < nd; ++d) {
                const __amdgpu_buffer_rsrc_t rs = __builtin_amdgcn_make_buffer_rsrc(P.ws[(unsigned)(d0 + ((d + I.kb) & (nd - 1)) * dstep) & (NR - 1)] + doff, 0, 16384, 0x00020000);
#pragma unroll
                for (int f = 0; f < 16; ++f) __builtin_amdgcn_raw_buffer_store_b128(h[f], rs, lane * 16 + f * 1024, 0, AUX_SYS);
            }
            WAIT_V(0);
            if (lane == 0)
                for (int d = 0; d < nd; ++d) SYS_STORE((unsigned*)(P.ws[(unsigned)(d0 + d * dstep) & (NR - 1)] + foff), (call << 8) | (unsigned)code);
            continue;
        }
        const __amdgpu_buffer_rsrc_t rs = __builtin_amdgcn_make_buffer_rsrc(ws + staged_off(I), 0, 16384, 0x00020000);
#pragma unroll
        for (int f = 0; f < 16; ++f) __builtin_amdgcn_raw_buffer_store_b128(h[f], rs, lane * 16 + f * 1024, 0, AUX_SYS);
        const bool mine = I.grp == (I.isw ? (P.rank & 1) : (P.rank >> 1));
        if (mine) {
            const __amdgpu_buffer_rsrc_t r2 = __builtin_amdgcn_make_buffer_rsrc(ws + doff, 0, 16384, 0x00020000);
#pragma unroll
            for (int f = 0; f < 16; ++f) __builtin_amdgcn_raw_buffer_store_b128(h[f], r2, lane * 16 + f * 1024, 0, AUX_AGENT);
        }
        WAIT_V(0);
        if (lane == 0) {
            const size_t fo = ready_off(P.rank, item);
            for (int d = 0; d < nd; ++d) {
                const unsigned dst = (unsigned)(d0 + d * dstep) & (NR - 1);
                if (dst != (unsigned)P.rank) SYS_STORE((u64*)(P.ws[dst] + fo), ((u64)call << 32) | (unsigned)code);
            }
            if (mine) AG_STORE((unsigned*)(ws + foff), (call << 8) | (unsigned)code);
        }
    }
}

__device__ __forceinline__ size_t packet_origin(int g, int pk) {
    const int t = pk >> 3, wv = pk & 7, patch = t >> 5;
    const int row = (g >> 1) * 2048 + ((patch & 1) * 4 + (t & 3)) * 256 + (wv >> 1) * 64;
    const int col = (g & 1) * 4096 + ((patch >> 1) * 8 + ((t & 31) >> 2)) * 128 + (wv & 1) * 64;
    return (size_t)row * GN + col;
}

__device__ __forceinline__ unsigned lds_get(LAS unsigned char* p) {
    unsigned v;
    asm volatile("ds_read_b32 %0, %1\n\ts_waitcnt lgkmcnt(0)" : "=v"(v) : "v"(p) : "memory");
    return (unsigned)__builtin_amdgcn_readfirstlane(v);
}
__device__ __forceinline__ void lds_put(LAS unsigned char* p, unsigned v) { asm volatile("ds_write_b32 %0, %1" :: "v"(p), "v"(v) : "memory"); }
__device__ __forceinline__ void lds_put64(LAS unsigned char* p, u64 v) { asm volatile("ds_write_b64 %0, %1" :: "v"(p), "v"(v) : "memory"); }
constexpr unsigned DEAD = 0xffffffffu;

__device__ __forceinline__ void comm_wg(const Params& P, int b, int wid, unsigned call, int lane, LAS unsigned char* lds) {
    const unsigned src = (unsigned)(P.rank + 1 + b % 7) & (NR - 1);
    unsigned char* ws = P.ws[P.rank];
    LAS unsigned char* list = lds + CTL;
    LAS unsigned char* full = list + 128;
    LAS unsigned char* freed = full + 32;
    LAS unsigned char* started = freed + 32;
    LAS unsigned char* fin = started + 4;
    LAS unsigned char* info = started + 16;
    for (int i = threadIdx.x; i < CTL_WORDS; i += NTH) lds_put(list + i * 4, 0u);
    WAIT_L0();
    BAR();
    const int n0 = XMODE == 2 ? NITEM : 0, n1 = XMODE == 1 ? NITEM : NTOT;
    if (wid == 1) {
        __builtin_amdgcn_s_setprio(2);
        unsigned* tmo = (unsigned*)(ws + TMO_OFF);
        unsigned* lq = (unsigned*)(ws + LQ_OFF + (b % 7) * 256);
        const unsigned tbase = (call - 1u) * (unsigned)(NTOT - NPUSH + EPL * DRAW) - NPUSH;
        const u64* flags = (const u64*)(ws + RF_OFF) + (size_t)src * NTOT;
        const u64 t0 = __builtin_amdgcn_s_memrealtime();
        bool dead = false, todo = lane >= NPUSH, more = true;
        unsigned drawn = 0, posted = 0, iter = 0;
        int base = n0, idq = 0;
        for (;;) {
            const bool want = more && drawn - lds_get(started) <= (unsigned)LOWQ;
            const bool poll = base < n1;
            unsigned t = 0; u64 f = 0;
            if (want && lane == 0) t = __hip_atomic_fetch_add(lq, (unsigned)DRAW, __ATOMIC_RELAXED, __HIP_MEMORY_SCOPE_AGENT);
            if (poll && todo) f = SYS_LOAD(flags + base + lane);
            if (poll) {
                if (todo) {
                    if ((unsigned)(f >> 32) == call || XMODE == 2) { lds_put(info + (base + lane) * 4, (unsigned)f); todo = false; }
                    else if (dead) { lds_put(info + (base + lane) * 4, DEAD); todo = false; }
                }
                if (!__any(todo)) { base += 64; todo = true; }
                else if ((++iter & 15u) == 0u && give_up(tmo, call, t0)) dead = true;
            }
            if (want) {
                t = (unsigned)__builtin_amdgcn_readfirstlane(t) - tbase;
                if (t >= (unsigned)NTOT) more = false;
                else if ((int)t >= n0 && (int)t < n1) {
#pragma unroll
                    for (int i = 0; i < DRAW; ++i) idq = llvm_writelane((int)t + i, (int)((drawn + i) & 15u), idq);
                    drawn += DRAW;
                }
            }
            while (posted < drawn) {
                const unsigned id = (unsigned)__builtin_amdgcn_readlane(idq, (int)(posted & 15u));
                const unsigned inf = lds_get(info + id * 4);
                if (inf == 0u) break;
                lds_put64(list + (posted & 15u) * 8, ((u64)inf << 32) | ((posted + 1u) << 12) | id);
                ++posted;
            }
            if (!more && posted == drawn) break;
            if (!want) __builtin_amdgcn_s_sleep(2);
        }
        lds_put64(list + (posted & 15u) * 8, (u64)((posted + 1u) << 12));
    } else if (wid == 0) {
        __builtin_amdgcn_s_setprio(3);
        const unsigned char* rws = P.ws[src];
        const unsigned voff = (unsigned)lane * 16u;
        int issued = 0, pub = 0, endv = 0, idv = 0;
        int pend = 0x7fffffff;
        int n = 0;
        auto publish = [&](int landed) {
            pend = 0x7fffffff;
            while (pub < n) {
                const int e = __builtin_amdgcn_readlane(endv, pub & 63);
                if (e > landed) { pend = e + (WIN - 1); break; }
                lds_put(full + (pub & (NSLOT - 1)) * 4, (unsigned)__builtin_amdgcn_readlane(idv, pub & 63));
                ++pub;
            }
        };
        u64 ts_first = 0, ts_in = 0, stall = 0, nstall = 0;
        u64 ent; unsigned fr;
        auto fetch = [&](int i) { asm volatile("ds_read_b64 %0, %2\n\tds_read_b32 %1, %3" : "=&v"(ent), "=&v"(fr) : "v"(list + (i & 15) * 8), "v"(freed + (i & (NSLOT - 1)) * 4) : "memory"); };
        fetch(0);
        for (;; ++n) {
            const unsigned want = (unsigned)(n < NSLOT ? 0 : n - NSLOT + 1);
            unsigned tag, inf;
            auto ready = [&]() -> bool {
                asm volatile("s_waitcnt lgkmcnt(0)" : "+v"(ent), "+v"(fr) :: "memory");
                tag = (unsigned)__builtin_amdgcn_readfirstlane((int)ent); inf = (unsigned)__builtin_amdgcn_readfirstlane((int)(ent >> 32));
                return (tag >> 12) == (unsigned)n + 1u && (inf == 0u || (unsigned)__builtin_amdgcn_readfirstlane((int)fr) == want);
            };
            if (!ready()) {
                fetch(n);
                if (!ready()) {
                    const u64 ts = TNOW();
                    WAIT_V(0);
                    publish(issued);
                    do { __builtin_amdgcn_s_sleep(1); fetch(n); } while (!ready());
                    if (n != 0) { stall += TNOW() - ts; ++nstall; }
                }
            }
            if (inf == 0u) break;
            lds_put(started, (unsigned)n + 1u);
            const unsigned id = tag & 4095u;
            unsigned bytes; size_t off;
            if (id < (unsigned)NITEM) { bytes = 16384u; off = staged_off(decode_item((int)id, (unsigned)P.rank)); }
            else { bytes = min(inf, (unsigned)PKT_BYTES); off = PK_OFF + (size_t)(id - NITEM) * PKT_STRIDE; }
            if (inf == DEAD) bytes = 0u;
            const __amdgpu_buffer_rsrc_t rs = __builtin_amdgcn_make_buffer_rsrc((void*)(rws + off), 0, (int)bytes, 0x00020000);
            LAS unsigned char* d = lds + (unsigned)(n & (NSLOT - 1)) * SLOT_BYTES;
            for (unsigned o = 0; o < bytes; o += 1024) {
                asm volatile("s_waitcnt vmcnt(%0)" :: "i"(WIN - 1) : "memory");
                __builtin_amdgcn_raw_ptr_buffer_load_lds(rs, (LAS void*)(d + o), 16, voff, o, 0, AUX_SYS);
                if (issued >= pend) publish(issued - (WIN - 1));
                ++issued;
                if (o + 3072 >= bytes && o + 2048 < bytes) fetch(n + 1);
            }
            if (bytes <= 2048u) fetch(n + 1);
            endv = llvm_writelane(issued, n & 63, endv);
            idv = llvm_writelane((int)tag, n & 63, idv);
            if (pub == n) pend = issued + (WIN - 1);
        }
        lds_put(fin, (unsigned)n + 1u);
        WAIT_V(0);
        publish(issued);
        STAMPV(3, ts_first); STAMPV(4, ts_in); STAMPV(5, stall); STAMPV(6, nstall); STAMPV(7, (u64)n);
    } else {
        const int m = lane & 15, q = lane >> 4;
        for (int n = wid - 2;; n += NWORK) {
            const unsigned slot = (unsigned)(n & (NSLOT - 1));
            unsigned fw;
            bool stop = false;
            while (((fw = lds_get(full + slot * 4)) >> 12) != (unsigned)n + 1u) {
                const unsigned fn = lds_get(fin);
                if (fn != 0u && (unsigned)n + 1u >= fn) { stop = true; break; }
                __builtin_amdgcn_s_sleep(2);
            }
            if (stop) break;
            const int id = (int)(fw & 4095u);
            const unsigned inf = lds_get(info + id * 4);
            LAS unsigned char* d = lds + slot * SLOT_BYTES;
            if (id < NITEM) {
                u4 h[16];
#pragma unroll
                for (int f = 0; f < 16; ++f) h[f] = *(LAS u4*)(d + f * 1024 + lane * 16);
                WAIT_L0();
                lds_put(freed + slot * 4, (unsigned)n + 1u);
                if (inf != DEAD) {
                    const Item I = decode_item(id, (unsigned)P.rank);
                    const int blk = I.b * 64 + (int)src * 8 + I.kb;
                    const __amdgpu_buffer_rsrc_t rs = __builtin_amdgcn_make_buffer_rsrc(ws + (I.isw ? W16_OFF : X16_OFF) + (size_t)blk * 16384, 0, 16384, 0x00020000);
#pragma unroll
                    for (int f = 0; f < 16; ++f) __builtin_amdgcn_raw_buffer_store_b128(h[f], rs, lane * 16 + f * 1024, 0, AUX_AGENT);
                    WAIT_V(0);
                    if (lane == 0) AG_STORE((unsigned*)(ws + (I.isw ? WF_OFF : XF_OFF) + (size_t)blk * 4), (call << 8) | (inf & 63u));
                }
            } else {
                const __amdgpu_buffer_rsrc_t ro = __builtin_amdgcn_make_buffer_rsrc(P.out + packet_origin((int)src, id - NITEM), 0, 64 * GN * 4, 0x00020000);
                const unsigned ov = (unsigned)(4 * q * GN + 4 * m) * 4u;
                if (inf != DEAD) {
                    const u64 mk = *(LAS u64*)(d + lane * 8);
                    const int mlo = (int)mk, mhi = (int)(mk >> 32);
                    unsigned base = 512;
#pragma unroll
                    for (int g4 = 0; g4 < 16; ++g4) {
                        f4 t;
#pragma unroll
                        for (int ni = 0; ni < 4; ++ni) {
                            const unsigned lo = (unsigned)__builtin_amdgcn_readlane(mlo, g4 * 4 + ni), hi = (unsigned)__builtin_amdgcn_readlane(mhi, g4 * 4 + ni);
                            const u64 mm = ((u64)hi << 32) | lo;
                            const unsigned pos = __builtin_amdgcn_mbcnt_hi(hi, __builtin_amdgcn_mbcnt_lo(lo, 0u));
                            const float val = *(LAS float*)(d + base + pos * 4);
                            t[ni] = __builtin_amdgcn_inverse_ballot_w64(mm) ? val : 0.f;
                            base += 4u * (unsigned)__builtin_popcountll(mm);
                        }
                        __builtin_amdgcn_raw_buffer_store_b128(__builtin_bit_cast(u4, t), ro, ov + (unsigned)(((g4 >> 2) * 16 + (g4 & 3)) * GN * 4), 0, OUT_AUX);
                    }
                } else {
#pragma unroll
                    for (int g4 = 0; g4 < 16; ++g4) __builtin_amdgcn_raw_buffer_store_b128((u4){0x7fc00000u, 0x7fc00000u, 0x7fc00000u, 0x7fc00000u}, ro, ov + (unsigned)(((g4 >> 2) * 16 + (g4 & 3)) * GN * 4), 0, 0);
                }
                WAIT_L0();
                lds_put(freed + slot * 4, (unsigned)n + 1u);
            }
        }
    }
}

__device__ __forceinline__ void gemm_tile(const Params& P, int tile, int wid, unsigned call, int lane, LAS unsigned char* lds, int stamp0) {
    unsigned char* ws = P.ws[P.rank];
    unsigned* tmo = (unsigned*)(ws + TMO_OFF);
    const int patch = tile >> 5;
    const int rb0 = ((patch & 1) * 4 + (tile & 3)) * 4;
    const int cb0 = ((patch >> 1) * 8 + ((tile & 31) >> 2)) * 2;
    const unsigned* xf = (const unsigned*)(ws + XF_OFF) + rb0 * 64;
    const unsigned* wf = (const unsigned*)(ws + WF_OFF) + cb0 * 64;
    if (wid == 0) {
        const u64 t0 = __builtin_amdgcn_s_memrealtime();
        bool dead = false;
        for (int g = 0; g < 6 && !dead; ++g) {
            const unsigned* f = (g < 4 ? xf + g * 64 : wf + (g - 4) * 64) + lane;
            while (!__all((SYS_LOAD(f) >> 8) == (call & 0xffffffu))) {
                if (give_up(tmo, call, t0)) { dead = true; break; }
                __builtin_amdgcn_s_sleep(32);
            }
        }
        __builtin_amdgcn_fence(__ATOMIC_ACQUIRE, "");
        WAIT_V(0);
    }
    BAR();
    STAMP(stamp0);
    const int wr = wid >> 1, wc = wid & 1;
    const bool bad = (unsigned)__builtin_amdgcn_readfirstlane(AG_LOAD(tmo)) == call;
    const int ev = 8 * ((int)(SYS_LOAD(xf + wr * 64 + lane) & 63u) + (int)(SYS_LOAD(wf + wc * 64 + lane) & 63u) - 32);
    WAIT_V(0);

    const unsigned char* ga = ws + X16_OFF + (size_t)rb0 * (1 << 20) + wid * 1024 + lane * 16;
    const unsigned char* gb = ws + W16_OFF + (size_t)cb0 * (1 << 20) + wid * 1024 + lane * 16;
    auto stage = [&](int kt, int sb) {
        LAS unsigned char* d = lds + (unsigned)sb * STAGE + (unsigned)wid * 1024u;
        const size_t ko = (size_t)(kt & 127) * 8192;
#pragma unroll
        for (int p = 0; p < 4; ++p) __builtin_amdgcn_global_load_lds((const unsigned*)(ga + (size_t)p * (1 << 20) + ko), (LAS unsigned*)(d + p * 8192), 16, 0, 0);
#pragma unroll
        for (int p = 0; p < 2; ++p) __builtin_amdgcn_global_load_lds((const unsigned*)(gb + (size_t)p * (1 << 20) + ko), (LAS unsigned*)(d + (4 + p) * 8192), 16, 0, 0);
    };
    f4 acc[4][4];
#pragma unroll
    for (int i = 0; i < 4; ++i)
#pragma unroll
        for (int j = 0; j < 4; ++j) acc[i][j] = (f4){0.f, 0.f, 0.f, 0.f};
    const int pk = tile * 8 + wid;
    float* o = P.out + packet_origin(P.rank, pk) + (size_t)(4 * (lane >> 4)) * GN + 4 * (lane & 15);
    bool spilled = false;
    stage(0, 0); stage(1, 1);
    int sb = 0, kb = 0, ecur = 0;
    const unsigned aoff = (unsigned)wr * 8192u + lane * 16u, boff = (unsigned)(4 + wc) * 8192u + lane * 16u;
    while (kb < 64) {
        ecur = __builtin_amdgcn_readlane(ev, kb);
        const u64 diff = __builtin_amdgcn_ballot_w64(ev != ecur) >> kb;
        const int kend = diff ? kb + (int)__builtin_ctzll(diff) : 64;
        for (int t = 2 * kb; t < 2 * kend; ++t) {
            WAIT_V(6);
            BAR();
            const int s2 = sb == 0 ? 2 : sb - 1;
            stage(t + 2, s2);
            LAS unsigned char* s = lds + (unsigned)sb * STAGE;
            h8 a[2][4], b[2][4];
#pragma unroll
            for (int kq = 0; kq < 2; ++kq)
#pragma unroll
                for (int i = 0; i < 4; ++i) { b[kq][i] = *(LAS h8*)(s + boff + (kq * 4 + i) * 1024); a[kq][i] = *(LAS h8*)(s + aoff + (kq * 4 + i) * 1024); }
            __builtin_amdgcn_sched_barrier(0);
#pragma unroll
            for (int kq = 0; kq < 2; ++kq)
#pragma unroll
                for (int i = 0; i < 4; ++i)
#pragma unroll
                    for (int j = 0; j < 4; ++j) acc[i][j] = __builtin_amdgcn_mfma_f32_16x16x32_f16(a[kq][i], b[kq][j], acc[i][j], 0, 0, 0);
            sb = sb == 2 ? 0 : sb + 1;
        }
        kb = kend;
        if (kb < 64) {
#pragma unroll
            for (int g4 = 0; g4 < 16; ++g4) {
                f4* p = (f4*)(o + (size_t)((g4 >> 2) * 16 + (g4 & 3)) * GN);
                f4 t;
#pragma unroll
                for (int ni = 0; ni < 4; ++ni) t[ni] = __builtin_amdgcn_ldexpf(acc[g4 >> 2][ni][g4 & 3], -ecur);
                if (spilled) t += *p;
                *p = t;
            }
#pragma unroll
            for (int i = 0; i < 4; ++i)
#pragma unroll
                for (int j = 0; j < 4; ++j) acc[i][j] = (f4){0.f, 0.f, 0.f, 0.f};
            spilled = true;
        }
    }
    WAIT_V(0);
    BAR();
    STAMP(stamp0 + 1);
    LAS unsigned char* d = lds + (unsigned)wid * PKT_BYTES;
    int mlo = 0, mhi = 0;
    unsigned base = 512;
#pragma unroll
    for (int g4 = 0; g4 < 16; ++g4) {
        f4* p = (f4*)(o + (size_t)((g4 >> 2) * 16 + (g4 & 3)) * GN);
        f4 t;
#pragma unroll
        for (int ni = 0; ni < 4; ++ni) t[ni] = __builtin_amdgcn_ldexpf(acc[g4 >> 2][ni][g4 & 3], -ecur);
        if (spilled) t += *p;
#pragma unroll
        for (int ni = 0; ni < 4; ++ni) {
            float val = bad ? __builtin_nanf("") : t[ni];
            const bool nz = !(val <= 0.f);
            val = nz ? val : 0.f;
            t[ni] = val;
            const u64 mm = __builtin_amdgcn_ballot_w64(nz);
            const unsigned pos = __builtin_amdgcn_mbcnt_hi((unsigned)(mm >> 32), __builtin_amdgcn_mbcnt_lo((unsigned)mm, 0u));
            if (nz) *(LAS float*)(d + base + pos * 4) = val;
            mlo = llvm_writelane((int)mm, g4 * 4 + ni, mlo);
            mhi = llvm_writelane((int)(mm >> 32), g4 * 4 + ni, mhi);
            base += 4u * (unsigned)__builtin_popcountll(mm);
        }
        *p = t;
    }
    *(LAS u64*)(d + lane * 8) = ((u64)(unsigned)mhi << 32) | (unsigned)mlo;
    WAIT_L0();
    const unsigned bytes = (base + 15u) & ~15u;
    const __amdgpu_buffer_rsrc_t rs = __builtin_amdgcn_make_buffer_rsrc(ws + PK_OFF + (size_t)pk * PKT_STRIDE, 0, PKT_BYTES, 0x00020000);
    for (unsigned ofs = lane * 16u; ofs < bytes; ofs += 1024) __builtin_amdgcn_raw_buffer_store_b128(*(LAS u4*)(d + ofs), rs, ofs, 0, AUX_SYS);
    WAIT_V(0);
    if (lane == 0) {
        const size_t fo = ready_off(P.rank, NITEM + pk);
        for (int i = 1; i < NR; ++i) SYS_STORE((u64*)(P.ws[(unsigned)(P.rank + i) & (NR - 1)] + fo), ((u64)call << 32) | bytes);
    }
    WAIT_L0();
    BAR();
    STAMP(stamp0 + 2);
}

__global__ __launch_bounds__(NTH, 2) void fused_gemm_gather(Params P) {
    extern __shared__ __attribute__((aligned(16))) unsigned char smem[];
    LAS unsigned char* lds = (LAS unsigned char*)smem;
    const int b = blockIdx.x, lane = threadIdx.x & 63;
    const int wid = __builtin_amdgcn_readfirstlane(threadIdx.x >> 6);
    unsigned call = 0;
    if (lane == 0) call = __hip_atomic_fetch_add((unsigned*)(P.ws[P.rank] + CNT_OFF + (size_t)(b * 8 + wid) * 16), 1u, __ATOMIC_RELAXED, __HIP_MEMORY_SCOPE_AGENT) + 1u;
    STAMP(0);
    if (b < NCOMM) {
        call = (unsigned)__builtin_amdgcn_readfirstlane(call);
        comm_wg(P, b, wid, call, lane, lds);
        STAMP(2);
    } else {
        if (XMODE != 2) convert_phase(P, wid * NGEMM + (b - NCOMM), call, lane);
        call = (unsigned)__builtin_amdgcn_readfirstlane(call);
        STAMP(7);
        const int patch = b & 7, slot = (b >> 3) - NCOMM / 8;
        const int own = patch * 32 + slot, extra = ((patch + 4) & 7) * 32 + (32 - NX) + slot;
        const int n = XMODE ? 0 : slot < NX ? 2 : 1;
        for (int i = 0; i < n; ++i) {
            const int tile = (n == 2 && ((patch >= 4) == (i == 0))) ? extra : own;
            gemm_tile(P, tile, wid, call, lane, lds, 1 + 3 * i);
        }
    }
}

}

extern "C" void kernel_launch(int rank, int nranks, ncclComm_t comm, hipStream_t stream,
                              void* const* d_in, const int* in_sizes, int n_in,
                              void* d_out, int out_size, const int* in_dtypes, int out_dtype,
                              void* const* d_out_all, void* const* d_ws_all, size_t ws_bytes) {
    (void)comm; (void)in_sizes; (void)n_in; (void)out_size; (void)in_dtypes; (void)out_dtype; (void)d_out_all;
    if (nranks != NR || (unsigned)rank >= (unsigned)NR || ws_bytes < WS_NEED) { fprintf(stderr, "unsupported configuration\n"); return; }
    HIP_CHECK(hipFuncSetAttribute((const void*)fused_gemm_gather, hipFuncAttributeMaxDynamicSharedMemorySize, LDS_BYTES));
    Params P;
    P.x = (const float*)d_in[0]; P.w = (const float*)d_in[1]; P.out = (float*)d_out; P.rank = rank;
    for (int p = 0; p < NR; ++p) P.ws[p] = (unsigned char*)d_ws_all[p];
    hipLaunchKernelGGL(fused_gemm_gather, dim3(NWG), dim3(NTH), LDS_BYTES, stream, P);
    HIP_CHECK(hipGetLastError());
}
```

```cpp
#include <hip/hip_runtime.h>
#include <rccl/rccl.h>
#include <cstdio>

#define HIP_CHECK(expr) do { hipError_t _e = (expr); if (_e != hipSuccess) { \
    fprintf(stderr, "HIP error %s at %s:%d\n", hipGetErrorString(_e), __FILE__, __LINE__); \
    return; } } while (0)

namespace {

typedef _Float16 h8 __attribute__((ext_vector_type(8)));
typedef _Float16 h2 __attribute__((ext_vector_type(2)));
typedef float f4 __attribute__((ext_vector_type(4)));
typedef float f2 __attribute__((ext_vector_type(2)));
typedef unsigned u4 __attribute__((ext_vector_type(4)));
typedef unsigned long long u64;
#define LAS __attribute__((address_space(3)))
extern "C" __device__ int llvm_writelane(int, int, int) __asm("llvm.amdgcn.writelane.i32");

constexpr int NR = 8;
constexpr int GN = 8192, KSH = 1024;
constexpr int NWG = 256, NTH = 512;
constexpr int EPL = 8;
constexpr int NCOMM = 7 * EPL;
constexpr int NGEMM = NWG - NCOMM;
#define WIN 8
#define DRAW 2
#define LOWQ 3
#define NPUSH 0
#define SLEEP0 4
#define SLEEP1 10
constexpr int NUNIT = 256;
constexpr int NSOLO = 24;
constexpr int NITEM = 768;
constexpr int NPKT = 9 * 256;
constexpr int NTOT = NITEM + NPKT;
constexpr int PKT_BYTES = 512 + 16384;
constexpr int STAGE = 6 * 8192;
constexpr int LDS_BYTES = 152 * 1024;
constexpr int NSLOT = 8, NWORK = 6;
constexpr int SLOT_BYTES = 17 * 1024;
constexpr int CTL = NSLOT * SLOT_BYTES;
constexpr int CTL_WORDS = 32 + 8 + 8 + 4 + NTOT;
static_assert(CTL + CTL_WORDS * 4 <= LDS_BYTES && 3 * STAGE <= LDS_BYTES && 8 * PKT_BYTES + 32 <= LDS_BYTES, "LDS");
static_assert(NTOT % 64 == 0 && NTOT <= 4096 && NPUSH % DRAW == 0 && NPUSH < 64 && LOWQ + DRAW < 16, "lists");
static_assert(NGEMM == 200 && NSOLO * 8 <= NGEMM && 2 * NSOLO + 2 * NGEMM >= NUNIT, "schedule");

constexpr size_t CNT_OFF = 0;
constexpr size_t TMO_OFF = 64 << 10;
constexpr size_t LQ_OFF = 96 << 10;
constexpr size_t XF_OFF = 128 << 10;
constexpr size_t WF_OFF = 160 << 10;
constexpr size_t RF_OFF = 256 << 10;
constexpr size_t ST_OFF = 512 << 10;
constexpr size_t X16_OFF = 1 << 20;
constexpr size_t W16_OFF = X16_OFF + (48ull << 20);
constexpr size_t S16_OFF = W16_OFF + (48ull << 20);
constexpr size_t PK_OFF = S16_OFF + (32ull << 20);
constexpr size_t WS_NEED = PK_OFF + (size_t)NPKT * PKT_BYTES + 1024;
static_assert(RF_OFF + (size_t)NR * NTOT * 8 <= ST_OFF, "flags");
constexpr u64 DEADLINE_TICKS = 50000000ull;

struct Params {
    const float* x; const float* w; float* out;
    unsigned char* ws[NR];
    int rank;
    unsigned geo[NR];
    unsigned pos;
    unsigned need[16];
    unsigned order[NUNIT / 4];
};

#define SYS_LOAD(p) __hip_atomic_load((p), __ATOMIC_RELAXED, __HIP_MEMORY_SCOPE_SYSTEM)
#define SYS_STORE(p, v) __hip_atomic_store((p), (v), __ATOMIC_RELAXED, __HIP_MEMORY_SCOPE_SYSTEM)
#define AG_LOAD(p) __hip_atomic_load((p), __ATOMIC_RELAXED, __HIP_MEMORY_SCOPE_AGENT)
#define AG_STORE(p, v) __hip_atomic_store((p), (v), __ATOMIC_RELAXED, __HIP_MEMORY_SCOPE_AGENT)
#define WAIT_V(n) asm volatile("s_waitcnt vmcnt(" #n ")" ::: "memory")
#define WAIT_L0() asm volatile("s_waitcnt lgkmcnt(0)" ::: "memory")
#define BAR() do { asm volatile("" ::: "memory"); __builtin_amdgcn_s_barrier(); asm volatile("" ::: "memory"); } while (0)
constexpr int AUX_SYS = 17;
constexpr int AUX_AGENT = 16;
constexpr int AUX_OUT = 18;

#define TNOW() 0ull
#define STAMPV(i, v) do { } while (0)
#define STAMP(i) do { } while (0)

__device__ __forceinline__ bool give_up(unsigned* tmo, unsigned call, u64 t0) {
    if ((unsigned)__builtin_amdgcn_readfirstlane(AG_LOAD(tmo)) == call) return true;
    if (__builtin_amdgcn_s_memrealtime() - t0 > DEADLINE_TICKS) { AG_STORE(tmo, call); return true; }
    return false;
}
__device__ __forceinline__ unsigned char* ws_of(const Params& P, unsigned r) {
    unsigned char* p = P.ws[0];
#pragma unroll
    for (int i = 1; i < NR; ++i) p = r == (unsigned)i ? P.ws[i] : p;
    return p;
}
__host__ __device__ __forceinline__ unsigned nib(unsigned v, unsigned i) { return (v >> (4 * i)) & 15u; }

__host__ __device__ __forceinline__ int item_id(int isw, int lc, int b, int kb) {
    return (lc == 0 ? b * 16 + isw * 8 : 256 + ((lc - 1) * 2 + 1 - isw) * 128 + b * 8) + kb;
}
struct Item { int isw, lc, b, kb; };
__device__ __forceinline__ Item decode_item(int it) {
    Item r;
    if (it < 256) { r.lc = 0; r.b = it >> 4; r.isw = (it >> 3) & 1; }
    else { const int s = (it - 256) >> 7; r.lc = 1 + (s >> 1); r.isw = 1 - (s & 1); r.b = (it >> 3) & 15; }
    r.kb = it & 7;
    return r;
}
__host__ __device__ __forceinline__ unsigned patch_lr(unsigned p) { return (0x2A450u >> (2 * p)) & 3u; }
__host__ __device__ __forceinline__ unsigned patch_lc(unsigned p) { return (0x24A44u >> (2 * p)) & 3u; }
__device__ __forceinline__ size_t staged_off(int isw, int cell, int b, int kb) { return S16_OFF + (size_t)(((isw * 8 + cell) * 16 + b) * 8 + kb) * 16384; }
__device__ __forceinline__ size_t ready_off(int src, int id) { return RF_OFF + ((size_t)src * NTOT + id) * 8; }
__device__ __forceinline__ size_t packet_origin(unsigned geo, int pk) {
    const unsigned p = (unsigned)pk >> 8, tt = ((unsigned)pk >> 3) & 31u, wv = pk & 7;
    const unsigned row = (nib(geo, patch_lr(p)) & 7u) * 1024 + (tt & 3) * 256 + (wv >> 1) * 64;
    const unsigned col = (nib(geo, 3 + patch_lc(p)) & 7u) * 1024 + (tt >> 2) * 128 + (wv & 1) * 64;
    return (size_t)row * GN + col;
}

__device__ __forceinline__ unsigned pk16(float a, float b) {
    const f2 p = {a, b};
    return __builtin_bit_cast(unsigned, __builtin_convertvector(p, h2));
}
__device__ __forceinline__ void store_block(unsigned char* dst, const u4 (&h)[16], int lane, bool sys) {
    const __amdgpu_buffer_rsrc_t rs = __builtin_amdgcn_make_buffer_rsrc(dst, 0, 16384, 0x00020000);
    if (sys) {
#pragma unroll
        for (int f = 0; f < 16; ++f) __builtin_amdgcn_raw_buffer_store_b128(h[f], rs, lane * 16 + f * 1024, 0, AUX_SYS);
    } else {
#pragma unroll
        for (int f = 0; f < 16; ++f) __builtin_amdgcn_raw_buffer_store_b128(h[f], rs, lane * 16 + f * 1024, 0, AUX_AGENT);
    }
}
__device__ __forceinline__ void convert_phase(const Params& P, int j, int wid, unsigned call  , int lane, LAS unsigned char* lds) {
    if (wid != 0) for (int i = wid == 1 ? SLEEP0 : SLEEP1; i > 0; --i) __builtin_amdgcn_s_sleep(32);
    const int m = lane & 15, q = lane >> 4;
    for (int rnd = (wid < 2 && j < NSOLO * 8) ? 0 : 1; rnd < 3; ++rnd) {
        const int u = rnd == 0 ? wid * NSOLO + (j >> 3) : 2 * NSOLO + j + (rnd - 1) * NGEMM;
        if (u >= NUNIT) break;
        const unsigned uo = (P.order[u >> 2] >> (8 * (u & 3))) & 255u;
        const int isw = (int)(uo >> 7), cell = (int)((uo >> 4) & 7u), b = (int)(uo & 15u), kb = rnd == 0 ? j & 7 : wid;
        f4 v[32];
        {
            const int first = cell * 1024 + b * 64;
            const unsigned base = isw ? (unsigned)(kb * 128 * GN + first) * 4u : (unsigned)(first * KSH + kb * 128) * 4u;
            const unsigned voff = isw ? (unsigned)(4 * q * GN + 4 * m) * 4u : (unsigned)(m * KSH + 4 * q) * 4u;
            const __amdgpu_buffer_rsrc_t rs = __builtin_amdgcn_make_buffer_rsrc((void*)(isw ? P.w : P.x), 0, KSH * GN * 4, 0x00020000);
#pragma unroll
            for (int n = 0; n < 32; ++n) {
                const unsigned ox = ((n >> 1) & 3) * (16 * KSH * 4) + (n >> 3) * 128 + (n & 1) * 64;
                const unsigned ow = ((n >> 3) * 32 + ((n & 7) < 4 ? (n & 7) : 12 + (n & 7))) * (GN * 4);
                v[n] = __builtin_bit_cast(f4, __builtin_amdgcn_raw_buffer_load_b128(rs, voff, base + (isw ? ow : ox), 0));
            }
        }
        float fm = 0.f;
#pragma unroll
        for (int i = 0; i < 32; ++i) fm = fmaxf(fmaxf(fm, fmaxf(fabsf(v[i][0]), fabsf(v[i][1]))), fmaxf(fabsf(v[i][2]), fabsf(v[i][3])));
        unsigned mx = __builtin_bit_cast(unsigned, fm);
        if (__any(mx >= 0x7f800000u)) {
            mx = 0;
#pragma unroll
            for (int i = 0; i < 32; ++i)
#pragma unroll
                for (int c = 0; c < 4; ++c) { const unsigned t = __builtin_bit_cast(unsigned, v[i][c]) & 0x7fffffffu; mx = max(mx, t < 0x7f800000u ? t : 0u); }
        }
#pragma unroll
        for (int o = 32; o >= 1; o >>= 1) mx = max(mx, (unsigned)__shfl_xor((int)mx, o));
        const int E = (int)((unsigned)__builtin_amdgcn_readfirstlane(mx) >> 23) - 127;
        const int code = ((14 - E) >> 3) + 16;
        const int s = (code - 16) * 8;
#pragma unroll
        for (int i = 0; i < 32; ++i) {
#pragma unroll
            for (int c = 0; c < 4; ++c) v[i][c] = __builtin_amdgcn_ldexpf(v[i][c], s);
            asm volatile("" : "+v"(v[i]));
        }
        u4 h[16];
        if (!isw) {
#pragma unroll
            for (int f = 0; f < 16; ++f) { const f4 a = v[2 * f], c = v[2 * f + 1];
                h[f] = (u4){pk16(a[0], a[1]), pk16(a[2], a[3]), pk16(c[0], c[1]), pk16(c[2], c[3])};
                asm volatile("; x" : "+v"(h[f])); }
        } else {
#pragma unroll
            for (int kq = 0; kq < 4; ++kq)
#pragma unroll
                for (int i = 0; i < 4; ++i) {
                    h[kq * 4 + i] = (u4){pk16(v[kq * 8 + 0][i], v[kq * 8 + 1][i]), pk16(v[kq * 8 + 2][i], v[kq * 8 + 3][i]),
                                         pk16(v[kq * 8 + 4][i], v[kq * 8 + 5][i]), pk16(v[kq * 8 + 6][i], v[kq * 8 + 7][i])};
                    asm volatile("; w" : "+v"(h[kq * 4 + i])); }
        }
        const unsigned need = P.need[isw * 8 + cell];
        for (int d = 0; d < 3; ++d) {
            const unsigned e = need >> (8 * d), dst = e & 7u; const int lc = (int)((e >> 3) & 3u);
            if (dst == (unsigned)P.rank || item_id(isw, lc, b, kb) < NPUSH)
                store_block(P.ws[dst] + (isw ? W16_OFF : X16_OFF) + (size_t)((lc * 16 + b) * 64 + P.rank * 8 + kb) * 16384, h, lane, true);
        }
        store_block(P.ws[P.rank] + staged_off(isw, cell, b, kb), h, lane, true);
        LAS unsigned* codes = (LAS unsigned*)(lds + rnd * 32);
        if (lane == 0) codes[wid] = (unsigned)code;
        asm volatile("s_waitcnt vmcnt(0) lgkmcnt(0)" ::: "memory");
        if (rnd != 0) BAR();
        if (wid == 0 || rnd == 0) {
            const unsigned cl = (unsigned)__builtin_amdgcn_readfirstlane(call);
            if (lane < 24 && (rnd != 0 || (lane & 7) == 0)) {
                const unsigned e = need >> (8 * (lane >> 3)), dst = e & 7u; const int lc = (int)((e >> 3) & 3u);
                const int k2 = rnd == 0 ? kb : lane & 7, id = item_id(isw, lc, b, k2);
                const unsigned c = rnd == 0 ? (unsigned)code : codes[lane & 7];
                unsigned char* dws = ws_of(P, dst);
                if (dst == (unsigned)P.rank || id < NPUSH) SYS_STORE((unsigned*)(dws + (isw ? WF_OFF : XF_OFF)) + (lc * 16 + b) * 64 + P.rank * 8 + k2, (cl << 8) | c);
                else SYS_STORE((u64*)(dws + ready_off(P.rank, id)), ((u64)cl << 32) | c);
            }
        }
        if (rnd == 0) STAMP(6);
    }
}

__device__ __forceinline__ unsigned lds_get(LAS unsigned char* p) {
    unsigned v;
    asm volatile("ds_read_b32 %0, %1\n\ts_waitcnt lgkmcnt(0)" : "=v"(v) : "v"(p) : "memory");
    return (unsigned)__builtin_amdgcn_readfirstlane(v);
}
__device__ __forceinline__ void lds_put(LAS unsigned char* p, unsigned v) { asm volatile("ds_write_b32 %0, %1" :: "v"(p), "v"(v) : "memory"); }
__device__ __forceinline__ void lds_put64(LAS unsigned char* p, u64 v) { asm volatile("ds_write_b64 %0, %1" :: "v"(p), "v"(v) : "memory"); }
constexpr unsigned DEAD = 0xffffffffu;
__device__ __forceinline__ unsigned cut7(unsigned j) { return (256u * j + 3u) / 7u; }

__device__ __forceinline__ void comm_wg(const Params& P, int b, int wid, unsigned call, int lane, LAS unsigned char* lds) {
    const unsigned src = (unsigned)(P.rank + 1 + b % 7) & (NR - 1);
    unsigned char* ws = P.ws[P.rank];
    LAS unsigned char* list = lds + CTL;
    LAS unsigned char* full = list + 128;
    LAS unsigned char* freed = full + 32;
    LAS unsigned char* started = freed + 32;
    LAS unsigned char* fin = started + 4;
    LAS unsigned char* info = started + 16;
    for (int i = threadIdx.x; i < CTL_WORDS; i += NTH) lds_put(list + i * 4, 0u);
    WAIT_L0();
    BAR();
    if (wid == 1) {
        __builtin_amdgcn_s_setprio(2);
        unsigned* tmo = (unsigned*)(ws + TMO_OFF);
        unsigned* lq = (unsigned*)(ws + LQ_OFF + (b % 7) * 256);
        const unsigned dist = (nib(P.pos, src) - nib(P.pos, (unsigned)P.rank)) & 7u;
        const unsigned pn = nib(P.geo[src], 6), pp = nib(P.geo[src], 7), nlen = cut7(7u - dist), pbeg = cut7(8u - dist);
        const unsigned nlist = NITEM + 7u * 256u + nlen + 256u - pbeg;
        const unsigned span = (nlist - NPUSH + DRAW - 1) / DRAW * DRAW;
        const unsigned tbase = (call - 1u) * (span + EPL * DRAW) - NPUSH;
        const u64* flags = (const u64*)(ws + RF_OFF) + (size_t)src * NTOT;
        const u64 t0 = __builtin_amdgcn_s_memrealtime();
        bool dead = false, todo = lane >= NPUSH, more = true;
        unsigned drawn = 0, posted = 0, iter = 0, cp = 0, cbase = NITEM;
        int base = 0, idq = 0;
        for (;;) {
            const bool want = more && drawn - lds_get(started) <= (unsigned)LOWQ;
            const bool poll = base < NTOT;
            unsigned t = 0; u64 f = 0;
            if (want && lane == 0) t = __hip_atomic_fetch_add(lq, (unsigned)DRAW, __ATOMIC_RELAXED, __HIP_MEMORY_SCOPE_AGENT);
            if (poll && todo) f = SYS_LOAD(flags + base + lane);
            if (poll) {
                if (todo) {
                    if ((unsigned)(f >> 32) == call) { lds_put(info + (base + lane) * 4, (unsigned)f); todo = false; }
                    else if (dead) { lds_put(info + (base + lane) * 4, DEAD); todo = false; }
                }
                if (!__any(todo)) { base += 64; todo = true; }
                else if ((++iter & 15u) == 0u && give_up(tmo, call, t0)) dead = true;
            }
            if (want) {
                t = (unsigned)__builtin_amdgcn_readfirstlane(t) - tbase;
                if (t >= nlist) more = false;
                else
                    for (unsigned i = t; i < t + DRAW && i < nlist; ++i) {
                        unsigned id = i;
                        if (i >= (unsigned)NITEM) {
                            for (;;) { const unsigned len = cp == pn ? nlen : cp == pp ? 256u - pbeg : 256u; if (i - cbase < len) break; cbase += len; ++cp; }
                            id = NITEM + cp * 256u + (cp == pp ? pbeg : 0u) + (i - cbase);
                        }
                        idq = llvm_writelane((int)id, (int)(drawn & 15u), idq);
                        ++drawn;
                    }
            }
            while (posted < drawn) {
                const unsigned id = (unsigned)__builtin_amdgcn_readlane(idq, (int)(posted & 15u));
                const unsigned inf = lds_get(info + id * 4);
                if (inf == 0u) break;
                lds_put64(list + (posted & 15u) * 8, ((u64)inf << 32) | ((posted + 1u) << 12) | id);
                ++posted;
            }
            if (!more && posted == drawn) break;
            if (!want) __builtin_amdgcn_s_sleep(2);
        }
        lds_put64(list + (posted & 15u) * 8, (u64)((posted + 1u) << 12));
    } else if (wid == 0) {
        __builtin_amdgcn_s_setprio(3);
        const unsigned char* rws = P.ws[src];
        const unsigned mygeo = P.geo[P.rank];
        const unsigned voff = (unsigned)lane * 16u;
        int issued = 0, pub = 0, endv = 0, idv = 0;
        int pend = 0x7fffffff;
        int n = 0;
        auto publish = [&](int landed) {
            pend = 0x7fffffff;
            while (pub < n) {
                const int e = __builtin_amdgcn_readlane(endv, pub & 63);
                if (e > landed) { pend = e + (WIN - 1); break; }
                lds_put(full + (pub & (NSLOT - 1)) * 4, (unsigned)__builtin_amdgcn_readlane(idv, pub & 63));
                ++pub;
            }
        };
        u64 ts_first = 0, ts_in = 0, stall = 0, nstall = 0;
        u64 ent; unsigned fr;
        auto fetch = [&](int i) { asm volatile("ds_read_b64 %0, %2\n\tds_read_b32 %1, %3" : "=&v"(ent), "=&v"(fr) : "v"(list + (i & 15) * 8), "v"(freed + (i & (NSLOT - 1)) * 4) : "memory"); };
        fetch(0);
        for (;; ++n) {
            const unsigned want = (unsigned)(n < NSLOT ? 0 : n - NSLOT + 1);
            unsigned tag, inf;
            auto ready = [&]() -> bool {
                asm volatile("s_waitcnt lgkmcnt(0)" : "+v"(ent), "+v"(fr) :: "memory");
                tag = (unsigned)__builtin_amdgcn_readfirstlane((int)ent); inf = (unsigned)__builtin_amdgcn_readfirstlane((int)(ent >> 32));
                return (tag >> 12) == (unsigned)n + 1u && (inf == 0u || (unsigned)__builtin_amdgcn_readfirstlane((int)fr) == want);
            };
            if (!ready()) {
                fetch(n);
                if (!ready()) {
                    const u64 ts = TNOW();
                    WAIT_V(0);
                    publish(issued);
                    do { __builtin_amdgcn_s_sleep(1); fetch(n); } while (!ready());
                    if (n != 0) { stall += TNOW() - ts; ++nstall; }
                }
            }
            if (inf == 0u) break;
            lds_put(started, (unsigned)n + 1u);
            const unsigned id = tag & 4095u;
            unsigned bytes; size_t off;
            if (id < (unsigned)NITEM) { const Item I = decode_item((int)id); bytes = 16384u; off = staged_off(I.isw, (int)(nib(mygeo, I.isw * 3 + I.lc) & 7u), I.b, I.kb); }
            else { bytes = min(inf, (unsigned)PKT_BYTES); off = PK_OFF + (size_t)(id - NITEM) * PKT_BYTES; }
            if (inf == DEAD) bytes = 0u;
            const __amdgpu_buffer_rsrc_t rs = __builtin_amdgcn_make_buffer_rsrc((void*)(rws + off), 0, (int)bytes, 0x00020000);
            LAS unsigned char* d = lds + (unsigned)(n & (NSLOT - 1)) * SLOT_BYTES;
            for (unsigned o = 0; o < bytes; o += 1024) {
                asm volatile("s_waitcnt vmcnt(%0)" :: "i"(WIN - 1) : "memory");
                __builtin_amdgcn_raw_ptr_buffer_load_lds(rs, (LAS void*)(d + o), 16, voff, o, 0, AUX_SYS);
                if (issued >= pend) publish(issued - (WIN - 1));
                ++issued;
                if (o + 3072 >= bytes && o + 2048 < bytes) fetch(n + 1);
            }
            if (bytes <= 2048u) fetch(n + 1);
            endv = llvm_writelane(issued, n & 63, endv);
            idv = llvm_writelane((int)tag, n & 63, idv);
            if (pub == n) pend = issued + (WIN - 1);
        }
        lds_put(fin, (unsigned)n + 1u);
        WAIT_V(0);
        publish(issued);
        STAMPV(3, ts_first); STAMPV(4, ts_in); STAMPV(5, stall); STAMPV(6, nstall); STAMPV(7, (u64)n);
    } else {
        const unsigned sgeo = P.geo[src];
        const unsigned ov = (unsigned)(4 * (lane >> 4) * GN + 4 * (lane & 15)) * 4u;
        for (int n = wid - 2;; n += NWORK) {
            const unsigned slot = (unsigned)(n & (NSLOT - 1));
            unsigned fw;
            bool stop = false;
            while (((fw = lds_get(full + slot * 4)) >> 12) != (unsigned)n + 1u) {
                const unsigned fn = lds_get(fin);
                if (fn != 0u && (unsigned)n + 1u >= fn) { stop = true; break; }
                __builtin_amdgcn_s_sleep(2);
            }
            if (stop) break;
            const int id = (int)(fw & 4095u);
            const unsigned inf = lds_get(info + id * 4);
            LAS unsigned char* d = lds + slot * SLOT_BYTES;
            if (id < NITEM) {
                u4 h[16];
#pragma unroll
                for (int f = 0; f < 16; ++f) h[f] = *(LAS u4*)(d + f * 1024 + lane * 16);
                WAIT_L0();
                lds_put(freed + slot * 4, (unsigned)n + 1u);
                if (inf != DEAD) {
                    const Item I = decode_item(id);
                    const int blk = (I.lc * 16 + I.b) * 64 + (int)src * 8 + I.kb;
                    store_block(ws + (I.isw ? W16_OFF : X16_OFF) + (size_t)blk * 16384, h, lane, false);
                    WAIT_V(0);
                    if (lane == 0) AG_STORE((unsigned*)(ws + (I.isw ? WF_OFF : XF_OFF)) + blk, (call << 8) | (inf & 63u));
                }
            } else {
                const __amdgpu_buffer_rsrc_t ro = __builtin_amdgcn_make_buffer_rsrc(P.out + packet_origin(sgeo, id - NITEM), 0, 63 * GN * 4 + 256, 0x00020000);
                if (inf != DEAD) {
                    const u64 mk = *(LAS u64*)(d + lane * 8);
                    const int mlo = (int)mk, mhi = (int)(mk >> 32);
                    unsigned base = 512;
#pragma unroll
                    for (int g4 = 0; g4 < 16; ++g4) {
                        f4 t;
#pragma unroll
                        for (int ni = 0; ni < 4; ++ni) {
                            const unsigned lo = (unsigned)__builtin_amdgcn_readlane(mlo, g4 * 4 + ni), hi = (unsigned)__builtin_amdgcn_readlane(mhi, g4 * 4 + ni);
                            const u64 mm = ((u64)hi << 32) | lo;
                            const unsigned pos = __builtin_amdgcn_mbcnt_hi(hi, __builtin_amdgcn_mbcnt_lo(lo, 0u));
                            const float val = *(LAS float*)(d + base + pos * 4);
                            t[ni] = __builtin_amdgcn_inverse_ballot_w64(mm) ? val : 0.f;
                            base += 4u * (unsigned)__builtin_popcountll(mm);
                        }
                        __builtin_amdgcn_raw_buffer_store_b128(__builtin_bit_cast(u4, t), ro, ov + (unsigned)(((g4 >> 2) * 16 + (g4 & 3)) * GN * 4), 0, AUX_OUT);
                    }
                } else {
#pragma unroll
                    for (int g4 = 0; g4 < 16; ++g4)
                        __builtin_amdgcn_raw_buffer_store_b128((u4){0x7fc00000u, 0x7fc00000u, 0x7fc00000u, 0x7fc00000u}, ro, ov + (unsigned)(((g4 >> 2) * 16 + (g4 & 3)) * GN * 4), 0, 0);
                }
                WAIT_L0();
                lds_put(freed + slot * 4, (unsigned)n + 1u);
            }
        }
    }
}

__device__ __forceinline__ void gemm_tile(const Params& P, int tile, int wid, unsigned call, int lane, LAS unsigned char* lds) {
    unsigned char* ws = P.ws[P.rank];
    unsigned* tmo = (unsigned*)(ws + TMO_OFF);
    const int rb0 = (int)patch_lr((unsigned)tile >> 5) * 16 + (tile & 3) * 4;
    const int cb0 = (int)patch_lc((unsigned)tile >> 5) * 16 + ((tile & 31) >> 2) * 2;
    const unsigned* xf = (const unsigned*)(ws + XF_OFF) + rb0 * 64;
    const unsigned* wf = (const unsigned*)(ws + WF_OFF) + cb0 * 64;
    if (wid == 0) {
        const u64 t0 = __builtin_amdgcn_s_memrealtime();
        bool dead = false;
        for (int g = 0; g < 6 && !dead; ++g) {
            const unsigned* f = (g < 4 ? xf + g * 64 : wf + (g - 4) * 64) + lane;
            while (!__all((SYS_LOAD(f) >> 8) == (call & 0xffffffu))) {
                if (give_up(tmo, call, t0)) { dead = true; break; }
                __builtin_amdgcn_s_sleep(32);
            }
        }
        __builtin_amdgcn_fence(__ATOMIC_ACQUIRE, "");
        WAIT_V(0);
    }
    BAR();
    STAMP(1);
    const int wr = wid >> 1, wc = wid & 1;
    const bool bad = (unsigned)__builtin_amdgcn_readfirstlane(AG_LOAD(tmo)) == call;
    const int ev = 8 * ((int)(SYS_LOAD(xf + wr * 64 + lane) & 63u) + (int)(SYS_LOAD(wf + wc * 64 + lane) & 63u) - 32);
    WAIT_V(0);

    const unsigned char* ga = ws + X16_OFF + (size_t)rb0 * (1 << 20) + wid * 1024 + lane * 16;
    const unsigned char* gb = ws + W16_OFF + (size_t)cb0 * (1 << 20) + wid * 1024 + lane * 16;
    auto stage = [&](int kt, int sb) {
        LAS unsigned char* d = lds + (unsigned)sb * STAGE + (unsigned)wid * 1024u;
        const size_t ko = (size_t)(kt & 127) * 8192;
#pragma unroll
        for (int p = 0; p < 4; ++p) __builtin_amdgcn_global_load_lds((const unsigned*)(ga + (size_t)p * (1 << 20) + ko), (LAS unsigned*)(d + p * 8192), 16, 0, 0);
#pragma unroll
        for (int p = 0; p < 2; ++p) __builtin_amdgcn_global_load_lds((const unsigned*)(gb + (size_t)p * (1 << 20) + ko), (LAS unsigned*)(d + (4 + p) * 8192), 16, 0, 0);
    };
    f4 acc[4][4];
#pragma unroll
    for (int i = 0; i < 4; ++i)
#pragma unroll
        for (int j = 0; j < 4; ++j) acc[i][j] = (f4){0.f, 0.f, 0.f, 0.f};
    const int pk = tile * 8 + wid;
    float* org = P.out + packet_origin(P.geo[P.rank], pk);
    float* o = org + (size_t)(4 * (lane >> 4)) * GN + 4 * (lane & 15);
    bool spilled = false;
    stage(0, 0); stage(1, 1);
    int sb = 0, kb = 0, ecur = 0;
    const unsigned aoff = (unsigned)wr * 8192u + lane * 16u, boff = (unsigned)(4 + wc) * 8192u + lane * 16u;
    while (kb < 64) {
        ecur = __builtin_amdgcn_readlane(ev, kb);
        const u64 diff = __builtin_amdgcn_ballot_w64(ev != ecur) >> kb;
        const int kend = diff ? kb + (int)__builtin_ctzll(diff) : 64;
        for (int t = 2 * kb; t < 2 * kend; ++t) {
            WAIT_V(6);
            BAR();
            const int s2 = sb == 0 ? 2 : sb - 1;
            stage(t + 2, s2);
            LAS unsigned char* s = lds + (unsigned)sb * STAGE;
            h8 a[2][4], b[2][4];
#pragma unroll
            for (int kq = 0; kq < 2; ++kq)
#pragma unroll
                for (int i = 0; i < 4; ++i) { b[kq][i] = *(LAS h8*)(s + boff + (kq * 4 + i) * 1024); a[kq][i] = *(LAS h8*)(s + aoff + (kq * 4 + i) * 1024); }
            __builtin_amdgcn_sched_barrier(0);
#pragma unroll
            for (int kq = 0; kq < 2; ++kq)
#pragma unroll
                for (int i = 0; i < 4; ++i)
#pragma unroll
                    for (int j = 0; j < 4; ++j) acc[i][j] = __builtin_amdgcn_mfma_f32_16x16x32_f16(a[kq][i], b[kq][j], acc[i][j], 0, 0, 0);
            sb = sb == 2 ? 0 : sb + 1;
        }
        kb = kend;
        if (kb < 64) {
#pragma unroll
            for (int g4 = 0; g4 < 16; ++g4) {
                f4* p = (f4*)(o + (size_t)((g4 >> 2) * 16 + (g4 & 3)) * GN);
                f4 t;
#pragma unroll
                for (int ni = 0; ni < 4; ++ni) t[ni] = __builtin_amdgcn_ldexpf(acc[g4 >> 2][ni][g4 & 3], -ecur);
                if (spilled) t += *p;
                *p = t;
            }
#pragma unroll
            for (int i = 0; i < 4; ++i)
#pragma unroll
                for (int j = 0; j < 4; ++j) acc[i][j] = (f4){0.f, 0.f, 0.f, 0.f};
            spilled = true;
        }
    }
    WAIT_V(0);
    BAR();
    STAMP(2);
    LAS unsigned char* d = lds + (unsigned)wid * PKT_BYTES;
    const __amdgpu_buffer_rsrc_t ro = __builtin_amdgcn_make_buffer_rsrc(org, 0, 63 * GN * 4 + 256, 0x00020000);
    const unsigned ov = (unsigned)(4 * (lane >> 4) * GN + 4 * (lane & 15)) * 4u;
    int mlo = 0, mhi = 0;
    unsigned base = 512;
#pragma unroll
    for (int g4 = 0; g4 < 16; ++g4) {
        f4 t;
#pragma unroll
        for (int ni = 0; ni < 4; ++ni) t[ni] = __builtin_amdgcn_ldexpf(acc[g4 >> 2][ni][g4 & 3], -ecur);
        if (spilled) t += *(const f4*)(o + (size_t)((g4 >> 2) * 16 + (g4 & 3)) * GN);
#pragma unroll
        for (int ni = 0; ni < 4; ++ni) {
            float val = bad ? __builtin_nanf("") : t[ni];
            const bool nz = !(val <= 0.f);
            val = nz ? val : 0.f;
            t[ni] = val;
            const u64 mm = __builtin_amdgcn_ballot_w64(nz);
            const unsigned pos = __builtin_amdgcn_mbcnt_hi((unsigned)(mm >> 32), __builtin_amdgcn_mbcnt_lo((unsigned)mm, 0u));
            if (nz) *(LAS float*)(d + base + pos * 4) = val;
            mlo = llvm_writelane((int)mm, g4 * 4 + ni, mlo);
            mhi = llvm_writelane((int)(mm >> 32), g4 * 4 + ni, mhi);
            base += 4u * (unsigned)__builtin_popcountll(mm);
        }
        __builtin_amdgcn_raw_buffer_store_b128(__builtin_bit_cast(u4, t), ro, ov + (unsigned)(((g4 >> 2) * 16 + (g4 & 3)) * GN * 4), 0, AUX_OUT);
    }
    *(LAS u64*)(d + lane * 8) = ((u64)(unsigned)mhi << 32) | (unsigned)mlo;
    WAIT_L0();
    const unsigned bytes = (base + 15u) & ~15u;
    const __amdgpu_buffer_rsrc_t rs = __builtin_amdgcn_make_buffer_rsrc(ws + PK_OFF + (size_t)pk * PKT_BYTES, 0, PKT_BYTES, 0x00020000);
    for (unsigned ofs = lane * 16u; ofs < bytes; ofs += 1024) __builtin_amdgcn_raw_buffer_store_b128(*(LAS u4*)(d + ofs), rs, ofs, 0, AUX_SYS);
    LAS unsigned* sizes = (LAS unsigned*)(lds + 8 * PKT_BYTES);
    if (lane == 0) sizes[wid] = bytes;
    asm volatile("s_waitcnt vmcnt(0) lgkmcnt(0)" ::: "memory");
    BAR();
    if (wid == 0 && lane < 56)
        SYS_STORE((u64*)(ws_of(P, (unsigned)(P.rank + 1 + (lane >> 3)) & (NR - 1)) + ready_off(P.rank, NITEM + tile * 8)) + (lane & 7), ((u64)call << 32) | sizes[lane & 7]);
    STAMP(3);
}

__global__ __launch_bounds__(NTH, 2) void fused_gemm_gather(Params P) {
    extern __shared__ __attribute__((aligned(16))) unsigned char smem[];
    LAS unsigned char* lds = (LAS unsigned char*)smem;
    const int b = blockIdx.x, lane = threadIdx.x & 63;
    const int wid = __builtin_amdgcn_readfirstlane(threadIdx.x >> 6);
    unsigned call = 0;
    if (lane == 0) call = __hip_atomic_fetch_add((unsigned*)(P.ws[P.rank] + CNT_OFF + (size_t)(b * 8 + wid) * 16), 1u, __ATOMIC_RELAXED, __HIP_MEMORY_SCOPE_AGENT) + 1u;
    STAMP(0);
    if (b < NCOMM) {
        call = (unsigned)__builtin_amdgcn_readfirstlane(call);
        comm_wg(P, b, wid, call, lane, lds);
        STAMP(2);
    } else {
        convert_phase(P, b - NCOMM, wid, call, lane, lds);
        call = (unsigned)__builtin_amdgcn_readfirstlane(call);
        STAMP(7);
        const int x = b & 7, slot = (b >> 3) - NCOMM / 8, grp = slot / 7, sub = 25 + slot - grp * 7;
        int first = x * 32 + slot, second = -1;
        if (x == 0) second = 8 * 32 + slot;
        else if (x == 1) { if (grp < 3) second = (4 + grp) * 32 + sub; }
        else if (x == 2) { if (grp < 2) second = (7 + grp) * 32 + sub; }
        else if (x >= 6 && grp < 2) { second = first; first = (x - 6 + 2 * grp) * 32 + sub; }
        for (int i = 0; i < 2; ++i) {
            const int tile = i == 0 ? first : second;
            if (tile >= 0) gemm_tile(P, tile, wid, call, lane, lds);
        }
    }
}

}

extern "C" void kernel_launch(int rank, int nranks, ncclComm_t comm, hipStream_t stream,
                              void* const* d_in, const int* in_sizes, int n_in,
                              void* d_out, int out_size, const int* in_dtypes, int out_dtype,
                              void* const* d_out_all, void* const* d_ws_all, size_t ws_bytes) {
    (void)comm; (void)in_sizes; (void)n_in; (void)out_size; (void)in_dtypes; (void)out_dtype; (void)d_out_all;
    if (nranks != NR || (unsigned)rank >= (unsigned)NR || ws_bytes < WS_NEED) { fprintf(stderr, "unsupported configuration\n"); return; }
    HIP_CHECK(hipFuncSetAttribute((const void*)fused_gemm_gather, hipFuncAttributeMaxDynamicSharedMemorySize, LDS_BYTES));
    static const int R[NR][3] = {{2, 1, 0}, {3, 4, 5}, {3, 4, 6}, {0, 1, 5}, {0, 5, 7}, {3, 6, 7}, {2, 7, 6}, {2, 1, 4}};
    static const int C[NR][3] = {{5, 2, 1}, {0, 1, 2}, {5, 3, 4}, {0, 3, 6}, {7, 4, 5}, {7, 6, 2}, {0, 3, 1}, {7, 4, 6}};
    static const int CYC[NR] = {0, 4, 5, 1, 3, 7, 2, 6};
    Params P;
    P.x = (const float*)d_in[0]; P.w = (const float*)d_in[1]; P.out = (float*)d_out; P.rank = rank;
    for (int p = 0; p < NR; ++p) P.ws[p] = (unsigned char*)d_ws_all[p];
    auto shared_patch = [&](int g, int h) {
        for (unsigned p = 0; p < 9; ++p) {
            const int r = R[g][patch_lr(p)], c = C[g][patch_lc(p)];
            if ((r == R[h][0] || r == R[h][1] || r == R[h][2]) && (c == C[h][0] || c == C[h][1] || c == C[h][2])) return p;
        }
        return 15u;
    };
    P.pos = 0;
    for (int i = 0; i < 16; ++i) P.need[i] = 0;
    int cnt[16] = {};
    bool early[16] = {};
    for (int k = 0; k < NR; ++k) {
        const int g = CYC[k];
        P.pos |= (unsigned)k << (4 * g);
        unsigned geo = (shared_patch(g, CYC[(k + 1) & 7]) << 24) | (shared_patch(g, CYC[(k + 7) & 7]) << 28);
        for (int l = 0; l < 3; ++l) {
            geo |= ((unsigned)R[g][l] << (4 * l)) | ((unsigned)C[g][l] << (4 * (3 + l)));
            P.need[R[g][l]] |= (unsigned)(g | l << 3) << (8 * cnt[R[g][l]]++);
            P.need[8 + C[g][l]] |= (unsigned)(g | l << 3) << (8 * cnt[8 + C[g][l]]++);
        }
        early[R[g][0]] = early[8 + C[g][0]] = true;
        P.geo[g] = geo;
    }
    unsigned char* order = (unsigned char*)P.order;
    int n = 0;
    for (int pass = 0; pass < 2; ++pass)
        for (int bb = 0; bb < 16; ++bb)
            for (int c = 0; c < 16; ++c)
                if (early[c] == (pass == 0)) order[n++] = (unsigned char)(c << 4 | bb);
    hipLaunchKernelGGL(fused_gemm_gather, dim3(NWG), dim3(NTH), LDS_BYTES, stream, P);
    HIP_CHECK(hipGetLastError());
}
```

```cpp
#include <hip/hip_runtime.h>
#include <rccl/rccl.h>
#include <cstdio>

#define HIP_CHECK(expr) do { hipError_t _e = (expr); if (_e != hipSuccess) { \
    fprintf(stderr, "HIP error %s at %s:%d\n", hipGetErrorString(_e), __FILE__, __LINE__); \
    return; } } while (0)

namespace {

typedef _Float16 h8 __attribute__((ext_vector_type(8)));
typedef _Float16 h2 __attribute__((ext_vector_type(2)));
typedef float f4 __attribute__((ext_vector_type(4)));
typedef float f2 __attribute__((ext_vector_type(2)));
typedef unsigned u4 __attribute__((ext_vector_type(4)));
typedef unsigned long long u64;
#define LAS __attribute__((address_space(3)))
extern "C" __device__ int llvm_writelane(int, int, int) __asm("llvm.amdgcn.writelane.i32");

constexpr int NR = 8;
constexpr int GN = 8192, KSH = 1024;
constexpr int NWG = 256, NTH = 512;
constexpr int EPL = 8;
constexpr int NCOMM = 7 * EPL;
constexpr int NGEMM = NWG - NCOMM;
#define WIN 8
#define DRAW 2
#define LOWQ 3
#define NPUSH 0
#define SLEEP0 4
#define SLEEP1 10
constexpr int NUNIT = 256;
constexpr int NSOLO = 24;
constexpr int NITEM = 768;
constexpr int NPKT = 9 * 256;
constexpr int NTOT = NITEM + NPKT;
constexpr int PKT_BYTES = 512 + 16384;
constexpr int STAGE = 6 * 8192;
constexpr int LDS_BYTES = 152 * 1024;
constexpr int NSLOT = 8, NWORK = 6;
constexpr int SLOT_BYTES = 17 * 1024;
constexpr int CTL = NSLOT * SLOT_BYTES;
constexpr int CTL_WORDS = 32 + 8 + 8 + 4 + NTOT;
static_assert(CTL + CTL_WORDS * 4 <= LDS_BYTES && 3 * STAGE <= LDS_BYTES && 8 * PKT_BYTES + 32 <= LDS_BYTES, "LDS");
static_assert(NTOT % 64 == 0 && NTOT <= 4096 && NPUSH % DRAW == 0 && 256 % DRAW == 0 && NPUSH < 64 && LOWQ + DRAW < 16, "lists");
static_assert(NGEMM == 200 && NSOLO * 8 <= NGEMM && 2 * NSOLO + 2 * NGEMM >= NUNIT, "schedule");

constexpr size_t CNT_OFF = 0;
constexpr size_t TMO_OFF = 64 << 10;
constexpr size_t LQ_OFF = 96 << 10;
constexpr size_t XF_OFF = 128 << 10;
constexpr size_t WF_OFF = 160 << 10;
constexpr size_t RF_OFF = 256 << 10;
constexpr size_t ST_OFF = 512 << 10;
constexpr size_t X16_OFF = 1 << 20;
constexpr size_t W16_OFF = X16_OFF + (48ull << 20);
constexpr size_t S16_OFF = W16_OFF + (48ull << 20);
constexpr size_t PK_OFF = S16_OFF + (32ull << 20);
constexpr size_t WS_NEED = PK_OFF + (size_t)NPKT * PKT_BYTES + 1024;
static_assert(RF_OFF + (size_t)NR * NTOT * 8 <= ST_OFF, "flags");
constexpr u64 DEADLINE_TICKS = 50000000ull;

struct Params {
    const float* x; const float* w; float* out;
    unsigned char* ws[NR];
    int rank;
    unsigned geo[NR];
    unsigned pos;
    unsigned need[16];
    unsigned order[NUNIT / 4];
};

#define SYS_LOAD(p) __hip_atomic_load((p), __ATOMIC_RELAXED, __HIP_MEMORY_SCOPE_SYSTEM)
#define SYS_STORE(p, v) __hip_atomic_store((p), (v), __ATOMIC_RELAXED, __HIP_MEMORY_SCOPE_SYSTEM)
#define AG_LOAD(p) __hip_atomic_load((p), __ATOMIC_RELAXED, __HIP_MEMORY_SCOPE_AGENT)
#define AG_STORE(p, v) __hip_atomic_store((p), (v), __ATOMIC_RELAXED, __HIP_MEMORY_SCOPE_AGENT)
#define WAIT_V(n) asm volatile("s_waitcnt vmcnt(" #n ")" ::: "memory")
#define WAIT_L0() asm volatile("s_waitcnt lgkmcnt(0)" ::: "memory")
#define BAR() do { asm volatile("" ::: "memory"); __builtin_amdgcn_s_barrier(); asm volatile("" ::: "memory"); } while (0)
constexpr int AUX_SYS = 17;
constexpr int AUX_AGENT = 16;
constexpr int AUX_OUT = 18;

#define TNOW() 0ull
#define STAMPV(i, v) do { } while (0)
#define STAMP(i) do { } while (0)

__device__ __forceinline__ bool give_up(unsigned* tmo, unsigned call, u64 t0) {
    if ((unsigned)__builtin_amdgcn_readfirstlane(AG_LOAD(tmo)) == call) return true;
    if (__builtin_amdgcn_s_memrealtime() - t0 > DEADLINE_TICKS) { AG_STORE(tmo, call); return true; }
    return false;
}
__device__ __forceinline__ unsigned char* ws_of(const Params& P, unsigned r) {
    unsigned char* p = P.ws[0];
#pragma unroll
    for (int i = 1; i < NR; ++i) p = r == (unsigned)i ? P.ws[i] : p;
    return p;
}
__host__ __device__ __forceinline__ unsigned nib(unsigned v, unsigned i) { return (v >> (4 * i)) & 15u; }

__host__ __device__ __forceinline__ int item_id(int isw, int lc, int b, int kb) {
    return (lc == 0 ? b * 16 + isw * 8 : 256 + ((lc - 1) * 2 + 1 - isw) * 128 + b * 8) + kb;
}
struct Item { int isw, lc, b, kb; };
__device__ __forceinline__ Item decode_item(int it) {
    Item r;
    if (it < 256) { r.lc = 0; r.b = it >> 4; r.isw = (it >> 3) & 1; }
    else { const int s = (it - 256) >> 7; r.lc = 1 + (s >> 1); r.isw = 1 - (s & 1); r.b = (it >> 3) & 15; }
    r.kb = it & 7;
    return r;
}
__host__ __device__ __forceinline__ unsigned patch_lr(unsigned p) { return (0x2A450u >> (2 * p)) & 3u; }
__host__ __device__ __forceinline__ unsigned patch_lc(unsigned p) { return (0x24A44u >> (2 * p)) & 3u; }
__device__ __forceinline__ size_t staged_off(int isw, int cell, int b, int kb) { return S16_OFF + (size_t)(((isw * 8 + cell) * 16 + b) * 8 + kb) * 16384; }
__device__ __forceinline__ size_t ready_off(int src, int id) { return RF_OFF + ((size_t)src * NTOT + id) * 8; }
__device__ __forceinline__ size_t packet_origin(unsigned geo, int pk) {
    const unsigned p = (unsigned)pk >> 8, tt = ((unsigned)pk >> 3) & 31u, wv = pk & 7;
    const unsigned row = (nib(geo, patch_lr(p)) & 7u) * 1024 + (tt & 3) * 256 + (wv >> 1) * 64;
    const unsigned col = (nib(geo, 3 + patch_lc(p)) & 7u) * 1024 + (tt >> 2) * 128 + (wv & 1) * 64;
    return (size_t)row * GN + col;
}

__device__ __forceinline__ unsigned pk16(float a, float b) {
    const f2 p = {a, b};
    return __builtin_bit_cast(unsigned, __builtin_convertvector(p, h2));
}
__device__ __forceinline__ void store_block(unsigned char* dst, const u4 (&h)[16], int lane, bool sys) {
    const __amdgpu_buffer_rsrc_t rs = __builtin_amdgcn_make_buffer_rsrc(dst, 0, 16384, 0x00020000);
    if (sys) {
#pragma unroll
        for (int f = 0; f < 16; ++f) __builtin_amdgcn_raw_buffer_store_b128(h[f], rs, lane * 16 + f * 1024, 0, AUX_SYS);
    } else {
#pragma unroll
        for (int f = 0; f < 16; ++f) __builtin_amdgcn_raw_buffer_store_b128(h[f], rs, lane * 16 + f * 1024, 0, AUX_AGENT);
    }
}
__device__ __forceinline__ void convert_phase(const Params& P, int j, int wid, unsigned call  , int lane, LAS unsigned char* lds) {
    if (wid != 0) for (int i = wid == 1 ? SLEEP0 : SLEEP1; i > 0; --i) __builtin_amdgcn_s_sleep(32);
    const int m = lane & 15, q = lane >> 4;
    for (int rnd = (wid < 2 && j < NSOLO * 8) ? 0 : 1; rnd < 3; ++rnd) {
        const int u = rnd == 0 ? wid * NSOLO + (j >> 3) : 2 * NSOLO + j + (rnd - 1) * NGEMM;
        if (u >= NUNIT) break;
        const unsigned uo = (P.order[u >> 2] >> (8 * (u & 3))) & 255u;
        const int isw = (int)(uo >> 7), cell = (int)((uo >> 4) & 7u), b = (int)(uo & 15u), kb = rnd == 0 ? j & 7 : wid;
        f4 v[32];
        {
            const int first = cell * 1024 + b * 64;
            const unsigned base = isw ? (unsigned)(kb * 128 * GN + first) * 4u : (unsigned)(first * KSH + kb * 128) * 4u;
            const unsigned voff = isw ? (unsigned)(4 * q * GN + 4 * m) * 4u : (unsigned)(m * KSH + 4 * q) * 4u;
            const __amdgpu_buffer_rsrc_t rs = __builtin_amdgcn_make_buffer_rsrc((void*)(isw ? P.w : P.x), 0, KSH * GN * 4, 0x00020000);
#pragma unroll
            for (int n = 0; n < 32; ++n) {
                const unsigned ox = ((n >> 1) & 3) * (16 * KSH * 4) + (n >> 3) * 128 + (n & 1) * 64;
                const unsigned ow = ((n >> 3) * 32 + ((n & 7) < 4 ? (n & 7) : 12 + (n & 7))) * (GN * 4);
                v[n] = __builtin_bit_cast(f4, __builtin_amdgcn_raw_buffer_load_b128(rs, voff, base + (isw ? ow : ox), 0));
            }
        }
        float fm = 0.f;
#pragma unroll
        for (int i = 0; i < 32; ++i) fm = fmaxf(fmaxf(fm, fmaxf(fabsf(v[i][0]), fabsf(v[i][1]))), fmaxf(fabsf(v[i][2]), fabsf(v[i][3])));
        unsigned mx = __builtin_bit_cast(unsigned, fm);
        if (__any(mx >= 0x7f800000u)) {
            mx = 0;
#pragma unroll
            for (int i = 0; i < 32; ++i)
#pragma unroll
                for (int c = 0; c < 4; ++c) { const unsigned t = __builtin_bit_cast(unsigned, v[i][c]) & 0x7fffffffu; mx = max(mx, t < 0x7f800000u ? t : 0u); }
        }
#pragma unroll
        for (int o = 32; o >= 1; o >>= 1) mx = max(mx, (unsigned)__shfl_xor((int)mx, o));
        const int E = (int)((unsigned)__builtin_amdgcn_readfirstlane(mx) >> 23) - 127;
        const int code = ((14 - E) >> 3) + 16;
        const int s = (code - 16) * 8;
#pragma unroll
        for (int i = 0; i < 32; ++i) {
#pragma unroll
            for (int c = 0; c < 4; ++c) v[i][c] = __builtin_amdgcn_ldexpf(v[i][c], s);
            asm volatile("" : "+v"(v[i]));
        }
        u4 h[16];
        if (!isw) {
#pragma unroll
            for (int f = 0; f < 16; ++f) { const f4 a = v[2 * f], c = v[2 * f + 1];
                h[f] = (u4){pk16(a[0], a[1]), pk16(a[2], a[3]), pk16(c[0], c[1]), pk16(c[2], c[3])};
                asm volatile("; x" : "+v"(h[f])); }
        } else {
#pragma unroll
            for (int kq = 0; kq < 4; ++kq)
#pragma unroll
                for (int i = 0; i < 4; ++i) {
                    h[kq * 4 + i] = (u4){pk16(v[kq * 8 + 0][i], v[kq * 8 + 1][i]), pk16(v[kq * 8 + 2][i], v[kq * 8 + 3][i]),
                                         pk16(v[kq * 8 + 4][i], v[kq * 8 + 5][i]), pk16(v[kq * 8 + 6][i], v[kq * 8 + 7][i])};
                    asm volatile("; w" : "+v"(h[kq * 4 + i])); }
        }
        const unsigned need = P.need[isw * 8 + cell];
        for (int d = 0; d < 3; ++d) {
            const unsigned e = need >> (8 * d), dst = e & 7u; const int lc = (int)((e >> 3) & 3u);
            if (dst == (unsigned)P.rank || item_id(isw, lc, b, kb) < NPUSH)
                store_block(P.ws[dst] + (isw ? W16_OFF : X16_OFF) + (size_t)((lc * 16 + b) * 64 + P.rank * 8 + kb) * 16384, h, lane, true);
        }
        store_block(P.ws[P.rank] + staged_off(isw, cell, b, kb), h, lane, true);
        LAS unsigned* codes = (LAS unsigned*)(lds + rnd * 32);
        if (lane == 0) codes[wid] = (unsigned)code;
        asm volatile("s_waitcnt vmcnt(0) lgkmcnt(0)" ::: "memory");
        if (rnd != 0) BAR();
        if (wid == 0 || rnd == 0) {
            const unsigned cl = (unsigned)__builtin_amdgcn_readfirstlane(call);
            if (lane < 24 && (rnd != 0 || (lane & 7) == 0)) {
                const unsigned e = need >> (8 * (lane >> 3)), dst = e & 7u; const int lc = (int)((e >> 3) & 3u);
                const int k2 = rnd == 0 ? kb : lane & 7, id = item_id(isw, lc, b, k2);
                const unsigned c = rnd == 0 ? (unsigned)code : codes[lane & 7];
                unsigned char* dws = ws_of(P, dst);
                if (dst == (unsigned)P.rank || id < NPUSH) SYS_STORE((unsigned*)(dws + (isw ? WF_OFF : XF_OFF)) + (lc * 16 + b) * 64 + P.rank * 8 + k2, (cl << 8) | c);
                else SYS_STORE((u64*)(dws + ready_off(P.rank, id)), ((u64)cl << 32) | c);
            }
        }
        if (rnd == 0) STAMP(6);
    }
}

__device__ __forceinline__ unsigned lds_get(LAS unsigned char* p) {
    unsigned v;
    asm volatile("ds_read_b32 %0, %1\n\ts_waitcnt lgkmcnt(0)" : "=v"(v) : "v"(p) : "memory");
    return (unsigned)__builtin_amdgcn_readfirstlane(v);
}
__device__ __forceinline__ void lds_put(LAS unsigned char* p, unsigned v) { asm volatile("ds_write_b32 %0, %1" :: "v"(p), "v"(v) : "memory"); }
__device__ __forceinline__ void lds_put64(LAS unsigned char* p, u64 v) { asm volatile("ds_write_b64 %0, %1" :: "v"(p), "v"(v) : "memory"); }
constexpr unsigned DEAD = 0xffffffffu;

__device__ __forceinline__ void comm_wg(const Params& P, int b, int wid, unsigned call, int lane, LAS unsigned char* lds) {
    const unsigned src = (unsigned)(P.rank + 1 + b % 7) & (NR - 1);
    unsigned char* ws = P.ws[P.rank];
    LAS unsigned char* list = lds + CTL;
    LAS unsigned char* full = list + 128;
    LAS unsigned char* freed = full + 32;
    LAS unsigned char* started = freed + 32;
    LAS unsigned char* fin = started + 4;
    LAS unsigned char* info = started + 16;
    for (int i = threadIdx.x; i < CTL_WORDS; i += NTH) lds_put(list + i * 4, 0u);
    WAIT_L0();
    BAR();
    if (wid == 1) {
        __builtin_amdgcn_s_setprio(2);
        unsigned* tmo = (unsigned*)(ws + TMO_OFF);
        const unsigned dist = (nib(P.pos, src) - nib(P.pos, (unsigned)P.rank)) & 7u;
        const unsigned pn = nib(P.geo[src], 6), pp = nib(P.geo[src], 7);
        u64* q = (u64*)(ws + LQ_OFF + (lane == 0 ? (unsigned)(b % 7) : 7u + dist + (lane == 2)) * 256);
        const u64 tbase = (u64)call << 32;
        if (lane < 3) __hip_atomic_fetch_max(q, tbase, __ATOMIC_RELAXED, __HIP_MEMORY_SCOPE_AGENT);
        WAIT_V(0);
        constexpr unsigned NLIST = NITEM + 7u * 256u - NPUSH;
        const u64* flags = (const u64*)(ws + RF_OFF) + (size_t)src * NTOT;
        const u64 t0 = __builtin_amdgcn_s_memrealtime();
        bool dead = false, todo = lane >= NPUSH, more = true;
        unsigned drawn = 0, posted = 0, iter = 0;
        unsigned sel = 0, seenp = dist == 1u ? 256u : 0u, seenn = dist == 7u ? 256u : 0u;
        int base = 0, idq = 0;
        for (;;) {
            const bool want = more && drawn - lds_get(started) <= (unsigned)LOWQ;
            const bool poll = base < NTOT;
            u64 t64 = 0, f = 0;
            if (want && lane < 3 && (lane == (int)sel || (sel != 0u && lane != 0)))
                t64 = __hip_atomic_fetch_add(q, (u64)(lane == (int)sel ? DRAW : 0), __ATOMIC_RELAXED, __HIP_MEMORY_SCOPE_AGENT);
            if (poll && todo) f = SYS_LOAD(flags + base + lane);
            if (poll) {
                if (todo) {
                    if ((unsigned)(f >> 32) == call) { lds_put(info + (base + lane) * 4, (unsigned)f); todo = false; }
                    else if (dead) { lds_put(info + (base + lane) * 4, DEAD); todo = false; }
                }
                if (!__any(todo)) { base += 64; todo = true; }
                else if ((++iter & 15u) == 0u && give_up(tmo, call, t0)) dead = true;
            }
            if (want) {
                const unsigned t = (unsigned)__builtin_amdgcn_readlane((int)t64, (int)sel);
                if (t < (sel == 0u ? NLIST : 256u)) {
                    unsigned id = t + NPUSH;
                    if (sel != 0u) id = NITEM + (sel == 1u ? pp : pn) * 256u + t;
                    else if (id >= (unsigned)NITEM) {
                        const unsigned lo = min(pn, pp), hi = max(pn, pp);
                        if (id >= NITEM + lo * 256u) id += 256u;
                        if (id >= NITEM + hi * 256u) id += 256u;
                    }
#pragma unroll
                    for (int i = 0; i < DRAW; ++i) idq = llvm_writelane((int)id + i, (int)((drawn + i) & 15u), idq);
                    drawn += DRAW;
                }
                if (sel != 0u) {
                    seenp = max(seenp, (unsigned)__builtin_amdgcn_readlane((int)t64, 1) + (sel == 1u ? DRAW : 0u));
                    seenn = max(seenn, (unsigned)__builtin_amdgcn_readlane((int)t64, 2) + (sel == 2u ? DRAW : 0u));
                }
                if (sel != 0u || t + DRAW >= NLIST) { sel = seenp < seenn ? 1u : 2u; more = min(seenp, seenn) < 256u; }
            }
            while (posted < drawn) {
                const unsigned id = (unsigned)__builtin_amdgcn_readlane(idq, (int)(posted & 15u));
                const unsigned inf = lds_get(info + id * 4);
                if (inf == 0u) break;
                lds_put64(list + (posted & 15u) * 8, ((u64)inf << 32) | ((posted + 1u) << 12) | id);
                ++posted;
            }
            if (!more && posted == drawn) break;
            if (!want) __builtin_amdgcn_s_sleep(2);
        }
        lds_put64(list + (posted & 15u) * 8, (u64)((posted + 1u) << 12));
    } else if (wid == 0) {
        __builtin_amdgcn_s_setprio(3);
        const unsigned char* rws = P.ws[src];
        const unsigned mygeo = P.geo[P.rank];
        const unsigned voff = (unsigned)lane * 16u;
        int issued = 0, pub = 0, endv = 0, idv = 0;
        int pend = 0x7fffffff;
        int n = 0;
        auto publish = [&](int landed) {
            pend = 0x7fffffff;
            while (pub < n) {
                const int e = __builtin_amdgcn_readlane(endv, pub & 63);
                if (e > landed) { pend = e + (WIN - 1); break; }
                lds_put(full + (pub & (NSLOT - 1)) * 4, (unsigned)__builtin_amdgcn_readlane(idv, pub & 63));
                ++pub;
            }
        };
        u64 ts_first = 0, ts_in = 0, stall = 0, nstall = 0;
        u64 ent; unsigned fr;
        auto fetch = [&](int i) { asm volatile("ds_read_b64 %0, %2\n\tds_read_b32 %1, %3" : "=&v"(ent), "=&v"(fr) : "v"(list + (i & 15) * 8), "v"(freed + (i & (NSLOT - 1)) * 4) : "memory"); };
        fetch(0);
        for (;; ++n) {
            const unsigned want = (unsigned)(n < NSLOT ? 0 : n - NSLOT + 1);
            unsigned tag, inf;
            auto ready = [&]() -> bool {
                asm volatile("s_waitcnt lgkmcnt(0)" : "+v"(ent), "+v"(fr) :: "memory");
                tag = (unsigned)__builtin_amdgcn_readfirstlane((int)ent); inf = (unsigned)__builtin_amdgcn_readfirstlane((int)(ent >> 32));
                return (tag >> 12) == (unsigned)n + 1u && (inf == 0u || (unsigned)__builtin_amdgcn_readfirstlane((int)fr) == want);
            };
            if (!ready()) {
                fetch(n);
                if (!ready()) {
                    const u64 ts = TNOW();
                    WAIT_V(0);
                    publish(issued);
                    do { __builtin_amdgcn_s_sleep(1); fetch(n); } while (!ready());
                    if (n != 0) { stall += TNOW() - ts; ++nstall; }
                }
            }
            if (inf == 0u) break;
            lds_put(started, (unsigned)n + 1u);
            const unsigned id = tag & 4095u;
            unsigned bytes; size_t off;
            if (id < (unsigned)NITEM) { const Item I = decode_item((int)id); bytes = 16384u; off = staged_off(I.isw, (int)(nib(mygeo, I.isw * 3 + I.lc) & 7u), I.b, I.kb); }
            else { bytes = min(inf, (unsigned)PKT_BYTES); off = PK_OFF + (size_t)(id - NITEM) * PKT_BYTES; }
            if (inf == DEAD) bytes = 0u;
            const __amdgpu_buffer_rsrc_t rs = __builtin_amdgcn_make_buffer_rsrc((void*)(rws + off), 0, (int)bytes, 0x00020000);
            LAS unsigned char* d = lds + (unsigned)(n & (NSLOT - 1)) * SLOT_BYTES;
            for (unsigned o = 0; o < bytes; o += 1024) {
                asm volatile("s_waitcnt vmcnt(%0)" :: "i"(WIN - 1) : "memory");
                __builtin_amdgcn_raw_ptr_buffer_load_lds(rs, (LAS void*)(d + o), 16, voff, o, 0, AUX_SYS);
                if (issued >= pend) publish(issued - (WIN - 1));
                ++issued;
                if (o + 3072 >= bytes && o + 2048 < bytes) fetch(n + 1);
            }
            if (bytes <= 2048u) fetch(n + 1);
            endv = llvm_writelane(issued, n & 63, endv);
            idv = llvm_writelane((int)tag, n & 63, idv);
            if (pub == n) pend = issued + (WIN - 1);
        }
        lds_put(fin, (unsigned)n + 1u);
        WAIT_V(0);
        publish(issued);
        STAMPV(3, ts_first); STAMPV(4, ts_in); STAMPV(5, stall); STAMPV(6, nstall); STAMPV(7, (u64)n);
    } else {
        const unsigned sgeo = P.geo[src];
        const unsigned ov = (unsigned)(4 * (lane >> 4) * GN + 4 * (lane & 15)) * 4u;
        for (int n = wid - 2;; n += NWORK) {
            const unsigned slot = (unsigned)(n & (NSLOT - 1));
            unsigned fw;
            bool stop = false;
            while (((fw = lds_get(full + slot * 4)) >> 12) != (unsigned)n + 1u) {
                const unsigned fn = lds_get(fin);
                if (fn != 0u && (unsigned)n + 1u >= fn) { stop = true; break; }
                __builtin_amdgcn_s_sleep(2);
            }
            if (stop) break;
            const int id = (int)(fw & 4095u);
            const unsigned inf = lds_get(info + id * 4);
            LAS unsigned char* d = lds + slot * SLOT_BYTES;
            if (id < NITEM) {
                u4 h[16];
#pragma unroll
                for (int f = 0; f < 16; ++f) h[f] = *(LAS u4*)(d + f * 1024 + lane * 16);
                WAIT_L0();
                lds_put(freed + slot * 4, (unsigned)n + 1u);
                if (inf != DEAD) {
                    const Item I = decode_item(id);
                    const int blk = (I.lc * 16 + I.b) * 64 + (int)src * 8 + I.kb;
                    store_block(ws + (I.isw ? W16_OFF : X16_OFF) + (size_t)blk * 16384, h, lane, false);
                    WAIT_V(0);
                    if (lane == 0) AG_STORE((unsigned*)(ws + (I.isw ? WF_OFF : XF_OFF)) + blk, (call << 8) | (inf & 63u));
                }
            } else {
                const __amdgpu_buffer_rsrc_t ro = __builtin_amdgcn_make_buffer_rsrc(P.out + packet_origin(sgeo, id - NITEM), 0, 63 * GN * 4 + 256, 0x00020000);
                if (inf != DEAD) {
                    const u64 mk = *(LAS u64*)(d + lane * 8);
                    const int mlo = (int)mk, mhi = (int)(mk >> 32);
                    unsigned base = 512;
#pragma unroll
                    for (int g4 = 0; g4 < 16; ++g4) {
                        f4 t;
#pragma unroll
                        for (int ni = 0; ni < 4; ++ni) {
                            const unsigned lo = (unsigned)__builtin_amdgcn_readlane(mlo, g4 * 4 + ni), hi = (unsigned)__builtin_amdgcn_readlane(mhi, g4 * 4 + ni);
                            const u64 mm = ((u64)hi << 32) | lo;
                            const unsigned pos = __builtin_amdgcn_mbcnt_hi(hi, __builtin_amdgcn_mbcnt_lo(lo, 0u));
                            const float val = *(LAS float*)(d + base + pos * 4);
                            t[ni] = __builtin_amdgcn_inverse_ballot_w64(mm) ? val : 0.f;
                            base += 4u * (unsigned)__builtin_popcountll(mm);
                        }
                        __builtin_amdgcn_raw_buffer_store_b128(__builtin_bit_cast(u4, t), ro, ov + (unsigned)(((g4 >> 2) * 16 + (g4 & 3)) * GN * 4), 0, AUX_OUT);
                    }
                } else {
#pragma unroll
                    for (int g4 = 0; g4 < 16; ++g4)
                        __builtin_amdgcn_raw_buffer_store_b128((u4){0x7fc00000u, 0x7fc00000u, 0x7fc00000u, 0x7fc00000u}, ro, ov + (unsigned)(((g4 >> 2) * 16 + (g4 & 3)) * GN * 4), 0, 0);
                }
                WAIT_L0();
                lds_put(freed + slot * 4, (unsigned)n + 1u);
            }
        }
    }
}

__device__ __forceinline__ void gemm_tile(const Params& P, int tile, int wid, unsigned call, int lane, LAS unsigned char* lds) {
    unsigned char* ws = P.ws[P.rank];
    unsigned* tmo = (unsigned*)(ws + TMO_OFF);
    const int rb0 = (int)patch_lr((unsigned)tile >> 5) * 16 + (tile & 3) * 4;
    const int cb0 = (int)patch_lc((unsigned)tile >> 5) * 16 + ((tile & 31) >> 2) * 2;
    const unsigned* xf = (const unsigned*)(ws + XF_OFF) + rb0 * 64;
    const unsigned* wf = (const unsigned*)(ws + WF_OFF) + cb0 * 64;
    if (wid == 0) {
        const u64 t0 = __builtin_amdgcn_s_memrealtime();
        bool dead = false;
        for (int g = 0; g < 6 && !dead; ++g) {
            const unsigned* f = (g < 4 ? xf + g * 64 : wf + (g - 4) * 64) + lane;
            while (!__all((SYS_LOAD(f) >> 8) == (call & 0xffffffu))) {
                if (give_up(tmo, call, t0)) { dead = true; break; }
                __builtin_amdgcn_s_sleep(32);
            }
        }
        __builtin_amdgcn_fence(__ATOMIC_ACQUIRE, "");
        WAIT_V(0);
    }
    BAR();
    STAMP(1);
    const int wr = wid >> 1, wc = wid & 1;
    const bool bad = (unsigned)__builtin_amdgcn_readfirstlane(AG_LOAD(tmo)) == call;
    const int ev = 8 * ((int)(SYS_LOAD(xf + wr * 64 + lane) & 63u) + (int)(SYS_LOAD(wf + wc * 64 + lane) & 63u) - 32);
    WAIT_V(0);

    const unsigned char* ga = ws + X16_OFF + (size_t)rb0 * (1 << 20) + wid * 1024 + lane * 16;
    const unsigned char* gb = ws + W16_OFF + (size_t)cb0 * (1 << 20) + wid * 1024 + lane * 16;
    auto stage = [&](int kt, int sb) {
        LAS unsigned char* d = lds + (unsigned)sb * STAGE + (unsigned)wid * 1024u;
        const size_t ko = (size_t)(kt & 127) * 8192;
#pragma unroll
        for (int p = 0; p < 4; ++p) __builtin_amdgcn_global_load_lds((const unsigned*)(ga + (size_t)p * (1 << 20) + ko), (LAS unsigned*)(d + p * 8192), 16, 0, 0);
#pragma unroll
        for (int p = 0; p < 2; ++p) __builtin_amdgcn_global_load_lds((const unsigned*)(gb + (size_t)p * (1 << 20) + ko), (LAS unsigned*)(d + (4 + p) * 8192), 16, 0, 0);
    };
    f4 acc[4][4];
#pragma unroll
    for (int i = 0; i < 4; ++i)
#pragma unroll
        for (int j = 0; j < 4; ++j) acc[i][j] = (f4){0.f, 0.f, 0.f, 0.f};
    const int pk = tile * 8 + wid;
    float* org = P.out + packet_origin(P.geo[P.rank], pk);
    float* o = org + (size_t)(4 * (lane >> 4)) * GN + 4 * (lane & 15);
    bool spilled = false;
    stage(0, 0); stage(1, 1);
    int sb = 0, kb = 0, ecur = 0;
    const unsigned aoff = (unsigned)wr * 8192u + lane * 16u, boff = (unsigned)(4 + wc) * 8192u + lane * 16u;
    while (kb < 64) {
        ecur = __builtin_amdgcn_readlane(ev, kb);
        const u64 diff = __builtin_amdgcn_ballot_w64(ev != ecur) >> kb;
        const int kend = diff ? kb + (int)__builtin_ctzll(diff) : 64;
        for (int t = 2 * kb; t < 2 * kend; ++t) {
            WAIT_V(6);
            BAR();
            const int s2 = sb == 0 ? 2 : sb - 1;
            stage(t + 2, s2);
            LAS unsigned char* s = lds + (unsigned)sb * STAGE;
            h8 a[2][4], b[2][4];
#pragma unroll
            for (int kq = 0; kq < 2; ++kq)
#pragma unroll
                for (int i = 0; i < 4; ++i) { b[kq][i] = *(LAS h8*)(s + boff + (kq * 4 + i) * 1024); a[kq][i] = *(LAS h8*)(s + aoff + (kq * 4 + i) * 1024); }
            __builtin_amdgcn_sched_barrier(0);
#pragma unroll
            for (int kq = 0; kq < 2; ++kq)
#pragma unroll
                for (int i = 0; i < 4; ++i)
#pragma unroll
                    for (int j = 0; j < 4; ++j) acc[i][j] = __builtin_amdgcn_mfma_f32_16x16x32_f16(a[kq][i], b[kq][j], acc[i][j], 0, 0, 0);
            sb = sb == 2 ? 0 : sb + 1;
        }
        kb = kend;
        if (kb < 64) {
#pragma unroll
            for (int g4 = 0; g4 < 16; ++g4) {
                f4* p = (f4*)(o + (size_t)((g4 >> 2) * 16 + (g4 & 3)) * GN);
                f4 t;
#pragma unroll
                for (int ni = 0; ni < 4; ++ni) t[ni] = __builtin_amdgcn_ldexpf(acc[g4 >> 2][ni][g4 & 3], -ecur);
                if (spilled) t += *p;
                *p = t;
            }
#pragma unroll
            for (int i = 0; i < 4; ++i)
#pragma unroll
                for (int j = 0; j < 4; ++j) acc[i][j] = (f4){0.f, 0.f, 0.f, 0.f};
            spilled = true;
        }
    }
    WAIT_V(0);
    BAR();
    STAMP(2);
    LAS unsigned char* d = lds + (unsigned)wid * PKT_BYTES;
    const __amdgpu_buffer_rsrc_t ro = __builtin_amdgcn_make_buffer_rsrc(org, 0, 63 * GN * 4 + 256, 0x00020000);
    const unsigned ov = (unsigned)(4 * (lane >> 4) * GN + 4 * (lane & 15)) * 4u;
    int mlo = 0, mhi = 0;
    unsigned base = 512;
#pragma unroll
    for (int g4 = 0; g4 < 16; ++g4) {
        f4 t;
#pragma unroll
        for (int ni = 0; ni < 4; ++ni) t[ni] = __builtin_amdgcn_ldexpf(acc[g4 >> 2][ni][g4 & 3], -ecur);
        if (spilled) t += *(const f4*)(o + (size_t)((g4 >> 2) * 16 + (g4 & 3)) * GN);
#pragma unroll
        for (int ni = 0; ni < 4; ++ni) {
            float val = bad ? __builtin_nanf("") : t[ni];
            const bool nz = !(val <= 0.f);
            val = nz ? val : 0.f;
            t[ni] = val;
            const u64 mm = __builtin_amdgcn_ballot_w64(nz);
            const unsigned pos = __builtin_amdgcn_mbcnt_hi((unsigned)(mm >> 32), __builtin_amdgcn_mbcnt_lo((unsigned)mm, 0u));
            if (nz) *(LAS float*)(d + base + pos * 4) = val;
            mlo = llvm_writelane((int)mm, g4 * 4 + ni, mlo);
            mhi = llvm_writelane((int)(mm >> 32), g4 * 4 + ni, mhi);
            base += 4u * (unsigned)__builtin_popcountll(mm);
        }
        __builtin_amdgcn_raw_buffer_store_b128(__builtin_bit_cast(u4, t), ro, ov + (unsigned)(((g4 >> 2) * 16 + (g4 & 3)) * GN * 4), 0, AUX_OUT);
    }
    *(LAS u64*)(d + lane * 8) = ((u64)(unsigned)mhi << 32) | (unsigned)mlo;
    WAIT_L0();
    const unsigned bytes = (base + 15u) & ~15u;
    const __amdgpu_buffer_rsrc_t rs = __builtin_amdgcn_make_buffer_rsrc(ws + PK_OFF + (size_t)pk * PKT_BYTES, 0, PKT_BYTES, 0x00020000);
    for (unsigned ofs = lane * 16u; ofs < bytes; ofs += 1024) __builtin_amdgcn_raw_buffer_store_b128(*(LAS u4*)(d + ofs), rs, ofs, 0, AUX_SYS);
    LAS unsigned* sizes = (LAS unsigned*)(lds + 8 * PKT_BYTES);
    if (lane == 0) sizes[wid] = bytes;
    asm volatile("s_waitcnt vmcnt(0) lgkmcnt(0)" ::: "memory");
    BAR();
    if (wid == 0 && lane < 56)
        SYS_STORE((u64*)(ws_of(P, (unsigned)(P.rank + 1 + (lane >> 3)) & (NR - 1)) + ready_off(P.rank, NITEM + tile * 8)) + (lane & 7), ((u64)call << 32) | sizes[lane & 7]);
    STAMP(3);
}

__global__ __launch_bounds__(NTH, 2) void fused_gemm_gather(Params P) {
    extern __shared__ __attribute__((aligned(16))) unsigned char smem[];
    LAS unsigned char* lds = (LAS unsigned char*)smem;
    const int b = blockIdx.x, lane = threadIdx.x & 63;
    const int wid = __builtin_amdgcn_readfirstlane(threadIdx.x >> 6);
    unsigned call = 0;
    if (lane == 0) call = __hip_atomic_fetch_add((unsigned*)(P.ws[P.rank] + CNT_OFF + (size_t)(b * 8 + wid) * 16), 1u, __ATOMIC_RELAXED, __HIP_MEMORY_SCOPE_AGENT) + 1u;
    STAMP(0);
    if (b < NCOMM) {
        call = (unsigned)__builtin_amdgcn_readfirstlane(call);
        comm_wg(P, b, wid, call, lane, lds);
        STAMP(2);
    } else {
        convert_phase(P, b - NCOMM, wid, call, lane, lds);
        call = (unsigned)__builtin_amdgcn_readfirstlane(call);
        STAMP(7);
        const int x = b & 7, slot = (b >> 3) - NCOMM / 8, grp = slot / 7, sub = 25 + slot - grp * 7;
        int first = x * 32 + slot, second = -1;
        if (x == 0) second = 8 * 32 + slot;
        else if (x == 1) { if (grp < 3) second = (4 + grp) * 32 + sub; }
        else if (x == 2) { if (grp < 2) second = (7 + grp) * 32 + sub; }
        else if (x >= 6 && grp < 2) { second = first; first = (x - 6 + 2 * grp) * 32 + sub; }
        for (int i = 0; i < 2; ++i) {
            const int tile = i == 0 ? first : second;
            if (tile >= 0) gemm_tile(P, tile, wid, call, lane, lds);
        }
    }
}

}

extern "C" void kernel_launch(int rank, int nranks, ncclComm_t comm, hipStream_t stream,
                              void* const* d_in, const int* in_sizes, int n_in,
                              void* d_out, int out_size, const int* in_dtypes, int out_dtype,
                              void* const* d_out_all, void* const* d_ws_all, size_t ws_bytes) {
    (void)comm; (void)in_sizes; (void)n_in; (void)out_size; (void)in_dtypes; (void)out_dtype; (void)d_out_all;
    if (nranks != NR || (unsigned)rank >= (unsigned)NR || ws_bytes < WS_NEED) { fprintf(stderr, "unsupported configuration\n"); return; }
    HIP_CHECK(hipFuncSetAttribute((const void*)fused_gemm_gather, hipFuncAttributeMaxDynamicSharedMemorySize, LDS_BYTES));
    static const int R[NR][3] = {{2, 1, 0}, {3, 4, 5}, {3, 4, 6}, {0, 1, 5}, {0, 5, 7}, {3, 6, 7}, {2, 7, 6}, {2, 1, 4}};
    static const int C[NR][3] = {{5, 2, 1}, {0, 1, 2}, {5, 3, 4}, {0, 3, 6}, {7, 4, 5}, {7, 6, 2}, {0, 3, 1}, {7, 4, 6}};
    static const int CYC[NR] = {0, 4, 5, 1, 3, 7, 2, 6};
    Params P;
    P.x = (const float*)d_in[0]; P.w = (const float*)d_in[1]; P.out = (float*)d_out; P.rank = rank;
    for (int p = 0; p < NR; ++p) P.ws[p] = (unsigned char*)d_ws_all[p];
    auto shared_patch = [&](int g, int h) {
        for (unsigned p = 0; p < 9; ++p) {
            const int r = R[g][patch_lr(p)], c = C[g][patch_lc(p)];
            if ((r == R[h][0] || r == R[h][1] || r == R[h][2]) && (c == C[h][0] || c == C[h][1] || c == C[h][2])) return p;
        }
        return 15u;
    };
    P.pos = 0;
    for (int i = 0; i < 16; ++i) P.need[i] = 0;
    int cnt[16] = {};
    bool early[16] = {};
    for (int k = 0; k < NR; ++k) {
        const int g = CYC[k];
        P.pos |= (unsigned)k << (4 * g);
        unsigned geo = (shared_patch(g, CYC[(k + 1) & 7]) << 24) | (shared_patch(g, CYC[(k + 7) & 7]) << 28);
        for (int l = 0; l < 3; ++l) {
            geo |= ((unsigned)R[g][l] << (4 * l)) | ((unsigned)C[g][l] << (4 * (3 + l)));
            P.need[R[g][l]] |= (unsigned)(g | l << 3) << (8 * cnt[R[g][l]]++);
            P.need[8 + C[g][l]] |= (unsigned)(g | l << 3) << (8 * cnt[8 + C[g][l]]++);
        }
        early[R[g][0]] = early[8 + C[g][0]] = true;
        P.geo[g] = geo;
    }
    unsigned char* order = (unsigned char*)P.order;
    int n = 0;
    for (int pass = 0; pass < 2; ++pass)
        for (int bb = 0; bb < 16; ++bb)
            for (int c = 0; c < 16; ++c)
                if (early[c] == (pass == 0)) order[n++] = (unsigned char)(c << 4 | bb);
    hipLaunchKernelGGL(fused_gemm_gather, dim3(NWG), dim3(NTH), LDS_BYTES, stream, P);
    HIP_CHECK(hipGetLastError());
}
```

```cpp
#include <hip/hip_runtime.h>
#include <rccl/rccl.h>
#include <cstdio>

#define HIP_CHECK(expr) do { hipError_t _e = (expr); if (_e != hipSuccess) { \
    fprintf(stderr, "HIP error %s at %s:%d\n", hipGetErrorString(_e), __FILE__, __LINE__); \
    return; } } while (0)

namespace {

typedef _Float16 h8 __attribute__((ext_vector_type(8)));
typedef _Float16 h2 __attribute__((ext_vector_type(2)));
typedef float f4 __attribute__((ext_vector_type(4)));
typedef float f2 __attribute__((ext_vector_type(2)));
typedef unsigned u4 __attribute__((ext_vector_type(4)));
typedef unsigned long long u64;
#define LAS __attribute__((address_space(3)))
extern "C" __device__ int llvm_writelane(int, int, int) __asm("llvm.amdgcn.writelane.i32");

constexpr int NR = 8;
constexpr int GN = 8192, KSH = 1024;
constexpr int NWG = 256, NTH = 512;
constexpr int EPL = 8;
constexpr int NCOMM = 7 * EPL;
constexpr int NGEMM = NWG - NCOMM;
#define WIN 8
#define DRAW 2
#define LOWQ 3
#define NPUSH 0
#define SLEEP2 20
constexpr int NUNIT = 256;
constexpr int NSOLO = 24;
constexpr int NITEM = 768;
constexpr int NPKT = 9 * 256;
constexpr int NTOT = NITEM + NPKT;
constexpr int PKT_BYTES = 512 + 16384;
constexpr int STAGE = 6 * 8192;
constexpr int LDS_BYTES = 152 * 1024;
constexpr int NSLOT = 8, NWORK = 6;
constexpr int SLOT_BYTES = 17 * 1024;
constexpr int CTL = NSLOT * SLOT_BYTES;
constexpr int CTL_WORDS = 32 + 8 + 8 + 4 + NTOT;
static_assert(CTL + CTL_WORDS * 4 <= LDS_BYTES && 3 * STAGE <= LDS_BYTES && 8 * PKT_BYTES + 32 <= LDS_BYTES, "LDS");
static_assert(NTOT % 64 == 0 && NTOT <= 4096 && NPUSH % DRAW == 0 && 256 % DRAW == 0 && NPUSH < 64 && LOWQ + DRAW < 16, "lists");
static_assert(NGEMM == 200 && NSOLO * 8 <= NGEMM && 2 * NSOLO + 2 * NGEMM >= NUNIT, "schedule");

constexpr size_t CNT_OFF = 0;
constexpr size_t TMO_OFF = 64 << 10;
constexpr size_t LQ_OFF = 96 << 10;
constexpr size_t XF_OFF = 128 << 10;
constexpr size_t WF_OFF = 160 << 10;
constexpr size_t RF_OFF = 256 << 10;
constexpr size_t ST_OFF = 512 << 10;
constexpr size_t X16_OFF = 1 << 20;
constexpr size_t W16_OFF = X16_OFF + (48ull << 20);
constexpr size_t S16_OFF = W16_OFF + (48ull << 20);
constexpr size_t PK_OFF = S16_OFF + (32ull << 20);
constexpr size_t WS_NEED = PK_OFF + (size_t)NPKT * PKT_BYTES + 1024;
static_assert(RF_OFF + (size_t)NR * NTOT * 8 <= ST_OFF, "flags");
constexpr u64 DEADLINE_TICKS = 50000000ull;

struct Params {
    const float* x; const float* w; float* out;
    unsigned char* ws[NR];
    int rank;
    unsigned geo[NR];
    unsigned pos;
    unsigned need[16];
    unsigned order[NUNIT / 4];
};

#define SYS_LOAD(p) __hip_atomic_load((p), __ATOMIC_RELAXED, __HIP_MEMORY_SCOPE_SYSTEM)
#define SYS_STORE(p, v) __hip_atomic_store((p), (v), __ATOMIC_RELAXED, __HIP_MEMORY_SCOPE_SYSTEM)
#define AG_LOAD(p) __hip_atomic_load((p), __ATOMIC_RELAXED, __HIP_MEMORY_SCOPE_AGENT)
#define AG_STORE(p, v) __hip_atomic_store((p), (v), __ATOMIC_RELAXED, __HIP_MEMORY_SCOPE_AGENT)
#define WAIT_V(n) asm volatile("s_waitcnt vmcnt(" #n ")" ::: "memory")
#define WAIT_L0() asm volatile("s_waitcnt lgkmcnt(0)" ::: "memory")
#define BAR() do { asm volatile("" ::: "memory"); __builtin_amdgcn_s_barrier(); asm volatile("" ::: "memory"); } while (0)
constexpr int AUX_SYS = 17;
constexpr int AUX_AGENT = 16;
constexpr int AUX_OUT = 18;

#define TNOW() 0ull
#define STAMPV(i, v) do { } while (0)
#define STAMP(i) do { } while (0)

__device__ __forceinline__ bool give_up(unsigned* tmo, unsigned call, u64 t0) {
    if ((unsigned)__builtin_amdgcn_readfirstlane(AG_LOAD(tmo)) == call) return true;
    if (__builtin_amdgcn_s_memrealtime() - t0 > DEADLINE_TICKS) { AG_STORE(tmo, call); return true; }
    return false;
}
__device__ __forceinline__ unsigned char* ws_of(const Params& P, unsigned r) {
    unsigned char* p = P.ws[0];
#pragma unroll
    for (int i = 1; i < NR; ++i) p = r == (unsigned)i ? P.ws[i] : p;
    return p;
}
__host__ __device__ __forceinline__ unsigned nib(unsigned v, unsigned i) { return (v >> (4 * i)) & 15u; }

__host__ __device__ __forceinline__ int item_id(int isw, int lc, int b, int kb) {
    return (lc == 0 ? b * 16 + isw * 8 : 256 + ((lc - 1) * 2 + 1 - isw) * 128 + b * 8) + kb;
}
struct Item { int isw, lc, b, kb; };
__device__ __forceinline__ Item decode_item(int it) {
    Item r;
    if (it < 256) { r.lc = 0; r.b = it >> 4; r.isw = (it >> 3) & 1; }
    else { const int s = (it - 256) >> 7; r.lc = 1 + (s >> 1); r.isw = 1 - (s & 1); r.b = (it >> 3) & 15; }
    r.kb = it & 7;
    return r;
}
__host__ __device__ __forceinline__ unsigned patch_lr(unsigned p) { return (0x2A450u >> (2 * p)) & 3u; }
__host__ __device__ __forceinline__ unsigned patch_lc(unsigned p) { return (0x24A44u >> (2 * p)) & 3u; }
__device__ __forceinline__ size_t staged_off(int isw, int cell, int b, int kb) { return S16_OFF + (size_t)(((isw * 8 + cell) * 16 + b) * 8 + kb) * 16384; }
__device__ __forceinline__ size_t ready_off(int src, int id) { return RF_OFF + ((size_t)src * NTOT + id) * 8; }
__device__ __forceinline__ size_t packet_origin(unsigned geo, int pk) {
    const unsigned p = (unsigned)pk >> 8, tt = ((unsigned)pk >> 3) & 31u, wv = pk & 7;
    const unsigned row = (nib(geo, patch_lr(p)) & 7u) * 1024 + (tt & 3) * 256 + (wv >> 1) * 64;
    const unsigned col = (nib(geo, 3 + patch_lc(p)) & 7u) * 1024 + (tt >> 2) * 128 + (wv & 1) * 64;
    return (size_t)row * GN + col;
}

__device__ __forceinline__ unsigned pk16(float a, float b) {
    const f2 p = {a, b};
    return __builtin_bit_cast(unsigned, __builtin_convertvector(p, h2));
}
__device__ __forceinline__ void store_block(unsigned char* dst, const u4 (&h)[16], int lane, bool sys) {
    const __amdgpu_buffer_rsrc_t rs = __builtin_amdgcn_make_buffer_rsrc(dst, 0, 16384, 0x00020000);
    if (sys) {
#pragma unroll
        for (int f = 0; f < 16; ++f) __builtin_amdgcn_raw_buffer_store_b128(h[f], rs, lane * 16 + f * 1024, 0, AUX_SYS);
    } else {
#pragma unroll
        for (int f = 0; f < 16; ++f) __builtin_amdgcn_raw_buffer_store_b128(h[f], rs, lane * 16 + f * 1024, 0, AUX_AGENT);
    }
}
__device__ __forceinline__ void convert_phase(const Params& P, int j, int wid, unsigned call  , int lane, LAS unsigned char* lds) {
    if (j >= NSOLO * 8) { for (int i = 0; i < 8; ++i) __builtin_amdgcn_s_sleep(32); }
    else if (wid != 0) { BAR(); if (wid != 1) BAR(); }
    const int m = lane & 15, q = lane >> 4;
    for (int rnd = (wid < 2 && j < NSOLO * 8) ? 0 : 1; rnd < 3; ++rnd) {
        const int u = rnd == 0 ? wid * NSOLO + (j >> 3) : 2 * NSOLO + j + (rnd - 1) * NGEMM;
        if (u >= NUNIT) break;
        if (rnd == 1 && u >= 96) for (int i = 0; i < SLEEP2; ++i) __builtin_amdgcn_s_sleep(32);
        const unsigned uo = (P.order[u >> 2] >> (8 * (u & 3))) & 255u;
        const int isw = (int)(uo >> 7), cell = (int)((uo >> 4) & 7u), b = (int)(uo & 15u), kb = rnd == 0 ? j & 7 : wid;
        f4 v[32];
        u64 ta = TNOW(), tb = 0, tc = 0, td = 0; (void)ta; (void)tb; (void)tc; (void)td;
        {
            const int first = cell * 1024 + b * 64;
            const unsigned base = isw ? (unsigned)(kb * 128 * GN + first) * 4u : (unsigned)(first * KSH + kb * 128) * 4u;
            const unsigned voff = isw ? (unsigned)(4 * q * GN + 4 * m) * 4u : (unsigned)(m * KSH + 4 * q) * 4u;
            const __amdgpu_buffer_rsrc_t rs = __builtin_amdgcn_make_buffer_rsrc((void*)(isw ? P.w : P.x), 0, KSH * GN * 4, 0x00020000);
#pragma unroll
            for (int n = 0; n < 32; ++n) {
                const unsigned ox = ((n >> 1) & 3) * (16 * KSH * 4) + (n >> 3) * 128 + (n & 1) * 64;
                const unsigned ow = ((n >> 3) * 32 + ((n & 7) < 4 ? (n & 7) : 12 + (n & 7))) * (GN * 4);
                v[n] = __builtin_bit_cast(f4, __builtin_amdgcn_raw_buffer_load_b128(rs, voff, base + (isw ? ow : ox), 0));
            }
        }
        float fm = 0.f;
#pragma unroll
        for (int i = 0; i < 32; ++i) fm = fmaxf(fmaxf(fm, fmaxf(fabsf(v[i][0]), fabsf(v[i][1]))), fmaxf(fabsf(v[i][2]), fabsf(v[i][3])));
        unsigned mx = __builtin_bit_cast(unsigned, fm);
        if (__any(mx >= 0x7f800000u)) {
            mx = 0;
#pragma unroll
            for (int i = 0; i < 32; ++i)
#pragma unroll
                for (int c = 0; c < 4; ++c) { const unsigned t = __builtin_bit_cast(unsigned, v[i][c]) & 0x7fffffffu; mx = max(mx, t < 0x7f800000u ? t : 0u); }
        }
#pragma unroll
        for (int o = 32; o >= 1; o >>= 1) mx = max(mx, (unsigned)__shfl_xor((int)mx, o));
        const int E = (int)((unsigned)__builtin_amdgcn_readfirstlane(mx) >> 23) - 127;
        tb = TNOW();
        const int code = ((14 - E) >> 3) + 16;
        const int s = (code - 16) * 8;
#pragma unroll
        for (int i = 0; i < 32; ++i) {
#pragma unroll
            for (int c = 0; c < 4; ++c) v[i][c] = __builtin_amdgcn_ldexpf(v[i][c], s);
            asm volatile("" : "+v"(v[i]));
        }
        u4 h[16];
        if (!isw) {
#pragma unroll
            for (int f = 0; f < 16; ++f) { const f4 a = v[2 * f], c = v[2 * f + 1];
                h[f] = (u4){pk16(a[0], a[1]), pk16(a[2], a[3]), pk16(c[0], c[1]), pk16(c[2], c[3])};
                asm volatile("; x" : "+v"(h[f])); }
        } else {
#pragma unroll
            for (int kq = 0; kq < 4; ++kq)
#pragma unroll
                for (int i = 0; i < 4; ++i) {
                    h[kq * 4 + i] = (u4){pk16(v[kq * 8 + 0][i], v[kq * 8 + 1][i]), pk16(v[kq * 8 + 2][i], v[kq * 8 + 3][i]),
                                         pk16(v[kq * 8 + 4][i], v[kq * 8 + 5][i]), pk16(v[kq * 8 + 6][i], v[kq * 8 + 7][i])};
                    asm volatile("; w" : "+v"(h[kq * 4 + i])); }
        }
        tc = TNOW();
        const unsigned need = P.need[isw * 8 + cell];
        for (int d = 0; d < 3; ++d) {
            const unsigned e = need >> (8 * d), dst = e & 7u; const int lc = (int)((e >> 3) & 3u);
            if (dst == (unsigned)P.rank || item_id(isw, lc, b, kb) < NPUSH)
                store_block(P.ws[dst] + (isw ? W16_OFF : X16_OFF) + (size_t)((lc * 16 + b) * 64 + P.rank * 8 + kb) * 16384, h, lane, true);
        }
        store_block(P.ws[P.rank] + staged_off(isw, cell, b, kb), h, lane, true);
        LAS unsigned* codes = (LAS unsigned*)(lds + rnd * 32);
        if (lane == 0) codes[wid] = (unsigned)code;
        asm volatile("s_waitcnt vmcnt(0) lgkmcnt(0)" ::: "memory");
        td = TNOW();
        if (rnd != 0) BAR();
        if (wid == 0 || rnd == 0) {
            const unsigned cl = (unsigned)__builtin_amdgcn_readfirstlane(call);
            if (lane < 24 && (rnd != 0 || (lane & 7) == 0)) {
                const unsigned e = need >> (8 * (lane >> 3)), dst = e & 7u; const int lc = (int)((e >> 3) & 3u);
                const int k2 = rnd == 0 ? kb : lane & 7, id = item_id(isw, lc, b, k2);
                const unsigned c = rnd == 0 ? (unsigned)code : codes[lane & 7];
                unsigned char* dws = ws_of(P, dst);
                if (dst == (unsigned)P.rank || id < NPUSH) SYS_STORE((unsigned*)(dws + (isw ? WF_OFF : XF_OFF)) + (lc * 16 + b) * 64 + P.rank * 8 + k2, (cl << 8) | c);
                else SYS_STORE((u64*)(dws + ready_off(P.rank, id)), ((u64)cl << 32) | c);
            }
        }
        if (rnd == 0) { STAMP(6); STAMPV(4, ((ta & 0xffffffffull) << 32) | (tb & 0xffffffffull)); STAMPV(5, ((tc & 0xffffffffull) << 32) | (td & 0xffffffffull)); }
        if (rnd == 0) { BAR(); if (wid == 0) BAR(); }
    }
}

__device__ __forceinline__ unsigned lds_get(LAS unsigned char* p) {
    unsigned v;
    asm volatile("ds_read_b32 %0, %1\n\ts_waitcnt lgkmcnt(0)" : "=v"(v) : "v"(p) : "memory");
    return (unsigned)__builtin_amdgcn_readfirstlane(v);
}
__device__ __forceinline__ void lds_put(LAS unsigned char* p, unsigned v) { asm volatile("ds_write_b32 %0, %1" :: "v"(p), "v"(v) : "memory"); }
__device__ __forceinline__ void lds_put64(LAS unsigned char* p, u64 v) { asm volatile("ds_write_b64 %0, %1" :: "v"(p), "v"(v) : "memory"); }
constexpr unsigned DEAD = 0xffffffffu;

__device__ __forceinline__ void comm_wg(const Params& P, int b, int wid, unsigned call, int lane, LAS unsigned char* lds) {
    const unsigned src = (unsigned)(P.rank + 1 + b % 7) & (NR - 1);
    unsigned char* ws = P.ws[P.rank];
    LAS unsigned char* list = lds + CTL;
    LAS unsigned char* full = list + 128;
    LAS unsigned char* freed = full + 32;
    LAS unsigned char* started = freed + 32;
    LAS unsigned char* fin = started + 4;
    LAS unsigned char* info = started + 16;
    for (int i = threadIdx.x; i < CTL_WORDS; i += NTH) lds_put(list + i * 4, 0u);
    WAIT_L0();
    BAR();
    if (wid == 1) {
        __builtin_amdgcn_s_setprio(2);
        unsigned* tmo = (unsigned*)(ws + TMO_OFF);
        const unsigned dist = (nib(P.pos, src) - nib(P.pos, (unsigned)P.rank)) & 7u;
        const unsigned pn = nib(P.geo[src], 6), pp = nib(P.geo[src], 7);
        u64* q = (u64*)(ws + LQ_OFF + (lane == 0 ? (unsigned)(b % 7) : 7u + dist + (lane == 2)) * 256);
        const u64 tbase = (u64)call << 32;
        if (lane < 3) __hip_atomic_fetch_max(q, tbase, __ATOMIC_RELAXED, __HIP_MEMORY_SCOPE_AGENT);
        WAIT_V(0);
        constexpr unsigned NLIST = NITEM + 7u * 256u - NPUSH;
        const u64* flags = (const u64*)(ws + RF_OFF) + (size_t)src * NTOT;
        const u64 t0 = __builtin_amdgcn_s_memrealtime();
        bool dead = false, todo = lane >= NPUSH, more = true;
        unsigned drawn = 0, posted = 0, iter = 0;
        unsigned sel = 0, seenp = dist == 1u ? 256u : 0u, seenn = dist == 7u ? 256u : 0u;
        int base = 0, idq = 0;
        for (;;) {
            const bool want = more && drawn - lds_get(started) <= (unsigned)LOWQ;
            const bool poll = base < NTOT;
            u64 t64 = 0, f = 0;
            if (want && lane < 3 && (lane == (int)sel || (sel != 0u && lane != 0)))
                t64 = __hip_atomic_fetch_add(q, (u64)(lane == (int)sel ? DRAW : 0), __ATOMIC_RELAXED, __HIP_MEMORY_SCOPE_AGENT);
            if (poll && todo) f = SYS_LOAD(flags + base + lane);
            if (poll) {
                if (todo) {
                    if ((unsigned)(f >> 32) == call) { lds_put(info + (base + lane) * 4, (unsigned)f); todo = false; }
                    else if (dead) { lds_put(info + (base + lane) * 4, DEAD); todo = false; }
                }
                if (!__any(todo)) { base += 64; todo = true; }
                else if ((++iter & 15u) == 0u && give_up(tmo, call, t0)) dead = true;
            }
            if (want) {
                const unsigned t = (unsigned)__builtin_amdgcn_readlane((int)t64, (int)sel);
                if (t < (sel == 0u ? NLIST : 256u)) {
                    unsigned id = t + NPUSH;
                    if (sel != 0u) id = NITEM + (sel == 1u ? pp : pn) * 256u + t;
                    else if (id >= (unsigned)NITEM) {
                        const unsigned lo = min(pn, pp), hi = max(pn, pp);
                        if (id >= NITEM + lo * 256u) id += 256u;
                        if (id >= NITEM + hi * 256u) id += 256u;
                    }
#pragma unroll
                    for (int i = 0; i < DRAW; ++i) idq = llvm_writelane((int)id + i, (int)((drawn + i) & 15u), idq);
                    drawn += DRAW;
                }
                if (sel != 0u) {
                    seenp = max(seenp, (unsigned)__builtin_amdgcn_readlane((int)t64, 1) + (sel == 1u ? DRAW : 0u));
                    seenn = max(seenn, (unsigned)__builtin_amdgcn_readlane((int)t64, 2) + (sel == 2u ? DRAW : 0u));
                }
                if (sel != 0u || t + DRAW >= NLIST) { sel = seenp < seenn ? 1u : 2u; more = min(seenp, seenn) < 256u; }
            }
            while (posted < drawn) {
                const unsigned id = (unsigned)__builtin_amdgcn_readlane(idq, (int)(posted & 15u));
                const unsigned inf = lds_get(info + id * 4);
                if (inf == 0u) break;
                lds_put64(list + (posted & 15u) * 8, ((u64)inf << 32) | ((posted + 1u) << 12) | id);
                ++posted;
            }
            if (!more && posted == drawn) break;
            if (!want) __builtin_amdgcn_s_sleep(2);
        }
        lds_put64(list + (posted & 15u) * 8, (u64)((posted + 1u) << 12));
    } else if (wid == 0) {
        __builtin_amdgcn_s_setprio(3);
        const unsigned char* rws = P.ws[src];
        const unsigned mygeo = P.geo[P.rank];
        const unsigned voff = (unsigned)lane * 16u;
        int issued = 0, pub = 0, endv = 0, idv = 0;
        int pend = 0x7fffffff;
        int n = 0;
        auto publish = [&](int landed) {
            pend = 0x7fffffff;
            while (pub < n) {
                const int e = __builtin_amdgcn_readlane(endv, pub & 63);
                if (e > landed) { pend = e + (WIN - 1); break; }
                lds_put(full + (pub & (NSLOT - 1)) * 4, (unsigned)__builtin_amdgcn_readlane(idv, pub & 63));
                ++pub;
            }
        };
        u64 ts_first = 0, ts_in = 0, stall = 0, nstall = 0;
        u64 ent; unsigned fr;
        auto fetch = [&](int i) { asm volatile("ds_read_b64 %0, %2\n\tds_read_b32 %1, %3" : "=&v"(ent), "=&v"(fr) : "v"(list + (i & 15) * 8), "v"(freed + (i & (NSLOT - 1)) * 4) : "memory"); };
        fetch(0);
        for (;; ++n) {
            const unsigned want = (unsigned)(n < NSLOT ? 0 : n - NSLOT + 1);
            unsigned tag, inf;
            auto ready = [&]() -> bool {
                asm volatile("s_waitcnt lgkmcnt(0)" : "+v"(ent), "+v"(fr) :: "memory");
                tag = (unsigned)__builtin_amdgcn_readfirstlane((int)ent); inf = (unsigned)__builtin_amdgcn_readfirstlane((int)(ent >> 32));
                return (tag >> 12) == (unsigned)n + 1u && (inf == 0u || (unsigned)__builtin_amdgcn_readfirstlane((int)fr) == want);
            };
            if (!ready()) {
                fetch(n);
                if (!ready()) {
                    const u64 ts = TNOW();
                    WAIT_V(0);
                    publish(issued);
                    do { __builtin_amdgcn_s_sleep(1); fetch(n); } while (!ready());
                    if (n != 0) { stall += TNOW() - ts; ++nstall; }
                }
            }
            if (inf == 0u) break;
            lds_put(started, (unsigned)n + 1u);
            const unsigned id = tag & 4095u;
            unsigned bytes; size_t off;
            if (id < (unsigned)NITEM) { const Item I = decode_item((int)id); bytes = 16384u; off = staged_off(I.isw, (int)(nib(mygeo, I.isw * 3 + I.lc) & 7u), I.b, I.kb); }
            else { bytes = min(inf, (unsigned)PKT_BYTES); off = PK_OFF + (size_t)(id - NITEM) * PKT_BYTES; }
            if (inf == DEAD) bytes = 0u;
            const __amdgpu_buffer_rsrc_t rs = __builtin_amdgcn_make_buffer_rsrc((void*)(rws + off), 0, (int)bytes, 0x00020000);
            LAS unsigned char* d = lds + (unsigned)(n & (NSLOT - 1)) * SLOT_BYTES;
            for (unsigned o = 0; o < bytes; o += 1024) {
                asm volatile("s_waitcnt vmcnt(%0)" :: "i"(WIN - 1) : "memory");
                __builtin_amdgcn_raw_ptr_buffer_load_lds(rs, (LAS void*)(d + o), 16, voff, o, 0, AUX_SYS);
                if (issued >= pend) publish(issued - (WIN - 1));
                ++issued;
                if (o + 3072 >= bytes && o + 2048 < bytes) fetch(n + 1);
            }
            if (bytes <= 2048u) fetch(n + 1);
            endv = llvm_writelane(issued, n & 63, endv);
            idv = llvm_writelane((int)tag, n & 63, idv);
            if (pub == n) pend = issued + (WIN - 1);
        }
        lds_put(fin, (unsigned)n + 1u);
        WAIT_V(0);
        publish(issued);
        STAMPV(3, ts_first); STAMPV(4, ts_in); STAMPV(5, stall); STAMPV(6, nstall); STAMPV(7, (u64)n);
    } else {
        const unsigned sgeo = P.geo[src];
        const unsigned ov = (unsigned)(4 * (lane >> 4) * GN + 4 * (lane & 15)) * 4u;
        for (int n = wid - 2;; n += NWORK) {
            const unsigned slot = (unsigned)(n & (NSLOT - 1));
            unsigned fw;
            bool stop = false;
            while (((fw = lds_get(full + slot * 4)) >> 12) != (unsigned)n + 1u) {
                const unsigned fn = lds_get(fin);
                if (fn != 0u && (unsigned)n + 1u >= fn) { stop = true; break; }
                __builtin_amdgcn_s_sleep(2);
            }
            if (stop) break;
            const int id = (int)(fw & 4095u);
            const unsigned inf = lds_get(info + id * 4);
            LAS unsigned char* d = lds + slot * SLOT_BYTES;
            if (id < NITEM) {
                u4 h[16];
#pragma unroll
                for (int f = 0; f < 16; ++f) h[f] = *(LAS u4*)(d + f * 1024 + lane * 16);
                WAIT_L0();
                lds_put(freed + slot * 4, (unsigned)n + 1u);
                if (inf != DEAD) {
                    const Item I = decode_item(id);
                    const int blk = (I.lc * 16 + I.b) * 64 + (int)src * 8 + I.kb;
                    store_block(ws + (I.isw ? W16_OFF : X16_OFF) + (size_t)blk * 16384, h, lane, false);
                    WAIT_V(0);
                    if (lane == 0) AG_STORE((unsigned*)(ws + (I.isw ? WF_OFF : XF_OFF)) + blk, (call << 8) | (inf & 63u));
                }
            } else {
                const __amdgpu_buffer_rsrc_t ro = __builtin_amdgcn_make_buffer_rsrc(P.out + packet_origin(sgeo, id - NITEM), 0, 63 * GN * 4 + 256, 0x00020000);
                if (inf != DEAD) {
                    const u64 mk = *(LAS u64*)(d + lane * 8);
                    const int mlo = (int)mk, mhi = (int)(mk >> 32);
                    unsigned base = 512;
#pragma unroll
                    for (int g4 = 0; g4 < 16; ++g4) {
                        f4 t;
#pragma unroll
                        for (int ni = 0; ni < 4; ++ni) {
                            const unsigned lo = (unsigned)__builtin_amdgcn_readlane(mlo, g4 * 4 + ni), hi = (unsigned)__builtin_amdgcn_readlane(mhi, g4 * 4 + ni);
                            const u64 mm = ((u64)hi << 32) | lo;
                            const unsigned pos = __builtin_amdgcn_mbcnt_hi(hi, __builtin_amdgcn_mbcnt_lo(lo, 0u));
                            const float val = *(LAS float*)(d + base + pos * 4);
                            t[ni] = __builtin_amdgcn_inverse_ballot_w64(mm) ? val : 0.f;
                            base += 4u * (unsigned)__builtin_popcountll(mm);
                        }
                        __builtin_amdgcn_raw_buffer_store_b128(__builtin_bit_cast(u4, t), ro, ov + (unsigned)(((g4 >> 2) * 16 + (g4 & 3)) * GN * 4), 0, AUX_OUT);
                    }
                } else {
#pragma unroll
                    for (int g4 = 0; g4 < 16; ++g4)
                        __builtin_amdgcn_raw_buffer_store_b128((u4){0x7fc00000u, 0x7fc00000u, 0x7fc00000u, 0x7fc00000u}, ro, ov + (unsigned)(((g4 >> 2) * 16 + (g4 & 3)) * GN * 4), 0, 0);
                }
                WAIT_L0();
                lds_put(freed + slot * 4, (unsigned)n + 1u);
            }
        }
    }
}

__device__ __forceinline__ void gemm_tile(const Params& P, int tile, int wid, unsigned call, int lane, LAS unsigned char* lds) {
    unsigned char* ws = P.ws[P.rank];
    unsigned* tmo = (unsigned*)(ws + TMO_OFF);
    const int rb0 = (int)patch_lr((unsigned)tile >> 5) * 16 + (tile & 3) * 4;
    const int cb0 = (int)patch_lc((unsigned)tile >> 5) * 16 + ((tile & 31) >> 2) * 2;
    const unsigned* xf = (const unsigned*)(ws + XF_OFF) + rb0 * 64;
    const unsigned* wf = (const unsigned*)(ws + WF_OFF) + cb0 * 64;
    if (wid == 0) {
        const u64 t0 = __builtin_amdgcn_s_memrealtime();
        bool dead = false;
        for (int g = 0; g < 6 && !dead; ++g) {
            const unsigned* f = (g < 4 ? xf + g * 64 : wf + (g - 4) * 64) + lane;
            while (!__all((SYS_LOAD(f) >> 8) == (call & 0xffffffu))) {
                if (give_up(tmo, call, t0)) { dead = true; break; }
                __builtin_amdgcn_s_sleep(32);
            }
        }
        __builtin_amdgcn_fence(__ATOMIC_ACQUIRE, "");
        WAIT_V(0);
    }
    BAR();
    STAMP(1);
    const int wr = wid >> 1, wc = wid & 1;
    const bool bad = (unsigned)__builtin_amdgcn_readfirstlane(AG_LOAD(tmo)) == call;
    const int ev = 8 * ((int)(SYS_LOAD(xf + wr * 64 + lane) & 63u) + (int)(SYS_LOAD(wf + wc * 64 + lane) & 63u) - 32);
    WAIT_V(0);

    const unsigned char* ga = ws + X16_OFF + (size_t)rb0 * (1 << 20) + wid * 1024 + lane * 16;
    const unsigned char* gb = ws + W16_OFF + (size_t)cb0 * (1 << 20) + wid * 1024 + lane * 16;
    auto stage = [&](int kt, int sb) {
        LAS unsigned char* d = lds + (unsigned)sb * STAGE + (unsigned)wid * 1024u;
        const size_t ko = (size_t)(kt & 127) * 8192;
#pragma unroll
        for (int p = 0; p < 4; ++p) __builtin_amdgcn_global_load_lds((const unsigned*)(ga + (size_t)p * (1 << 20) + ko), (LAS unsigned*)(d + p * 8192), 16, 0, 0);
#pragma unroll
        for (int p = 0; p < 2; ++p) __builtin_amdgcn_global_load_lds((const unsigned*)(gb + (size_t)p * (1 << 20) + ko), (LAS unsigned*)(d + (4 + p) * 8192), 16, 0, 0);
    };
    f4 acc[4][4];
#pragma unroll
    for (int i = 0; i < 4; ++i)
#pragma unroll
        for (int j = 0; j < 4; ++j) acc[i][j] = (f4){0.f, 0.f, 0.f, 0.f};
    const int pk = tile * 8 + wid;
    float* org = P.out + packet_origin(P.geo[P.rank], pk);
    float* o = org + (size_t)(4 * (lane >> 4)) * GN + 4 * (lane & 15);
    bool spilled = false;
    stage(0, 0); stage(1, 1);
    int sb = 0, kb = 0, ecur = 0;
    const unsigned aoff = (unsigned)wr * 8192u + lane * 16u, boff = (unsigned)(4 + wc) * 8192u + lane * 16u;
    while (kb < 64) {
        ecur = __builtin_amdgcn_readlane(ev, kb);
        const u64 diff = __builtin_amdgcn_ballot_w64(ev != ecur) >> kb;
        const int kend = diff ? kb + (int)__builtin_ctzll(diff) : 64;
        for (int t = 2 * kb; t < 2 * kend; ++t) {
            WAIT_V(6);
            BAR();
            const int s2 = sb == 0 ? 2 : sb - 1;
            stage(t + 2, s2);
            LAS unsigned char* s = lds + (unsigned)sb * STAGE;
            h8 a[2][4], b[2][4];
#pragma unroll
            for (int kq = 0; kq < 2; ++kq)
#pragma unroll
                for (int i = 0; i < 4; ++i) { b[kq][i] = *(LAS h8*)(s + boff + (kq * 4 + i) * 1024); a[kq][i] = *(LAS h8*)(s + aoff + (kq * 4 + i) * 1024); }
            __builtin_amdgcn_sched_barrier(0);
#pragma unroll
            for (int kq = 0; kq < 2; ++kq)
#pragma unroll
                for (int i = 0; i < 4; ++i)
#pragma unroll
                    for (int j = 0; j < 4; ++j) acc[i][j] = __builtin_amdgcn_mfma_f32_16x16x32_f16(a[kq][i], b[kq][j], acc[i][j], 0, 0, 0);
            sb = sb == 2 ? 0 : sb + 1;
        }
        kb = kend;
        if (kb < 64) {
#pragma unroll
            for (int g4 = 0; g4 < 16; ++g4) {
                f4* p = (f4*)(o + (size_t)((g4 >> 2) * 16 + (g4 & 3)) * GN);
                f4 t;
#pragma unroll
                for (int ni = 0; ni < 4; ++ni) t[ni] = __builtin_amdgcn_ldexpf(acc[g4 >> 2][ni][g4 & 3], -ecur);
                if (spilled) t += *p;
                *p = t;
            }
#pragma unroll
            for (int i = 0; i < 4; ++i)
#pragma unroll
                for (int j = 0; j < 4; ++j) acc[i][j] = (f4){0.f, 0.f, 0.f, 0.f};
            spilled = true;
        }
    }
    WAIT_V(0);
    BAR();
    STAMP(2);
    LAS unsigned char* d = lds + (unsigned)wid * PKT_BYTES;
    const __amdgpu_buffer_rsrc_t ro = __builtin_amdgcn_make_buffer_rsrc(org, 0, 63 * GN * 4 + 256, 0x00020000);
    const unsigned ov = (unsigned)(4 * (lane >> 4) * GN + 4 * (lane & 15)) * 4u;
    int mlo = 0, mhi = 0;
    unsigned base = 512;
#pragma unroll
    for (int g4 = 0; g4 < 16; ++g4) {
        f4 t;
#pragma unroll
        for (int ni = 0; ni < 4; ++ni) t[ni] = __builtin_amdgcn_ldexpf(acc[g4 >> 2][ni][g4 & 3], -ecur);
        if (spilled) t += *(const f4*)(o + (size_t)((g4 >> 2) * 16 + (g4 & 3)) * GN);
#pragma unroll
        for (int ni = 0; ni < 4; ++ni) {
            float val = bad ? __builtin_nanf("") : t[ni];
            const bool nz = !(val <= 0.f);
            val = nz ? val : 0.f;
            t[ni] = val;
            const u64 mm = __builtin_amdgcn_ballot_w64(nz);
            const unsigned pos = __builtin_amdgcn_mbcnt_hi((unsigned)(mm >> 32), __builtin_amdgcn_mbcnt_lo((unsigned)mm, 0u));
            if (nz) *(LAS float*)(d + base + pos * 4) = val;
            mlo = llvm_writelane((int)mm, g4 * 4 + ni, mlo);
            mhi = llvm_writelane((int)(mm >> 32), g4 * 4 + ni, mhi);
            base += 4u * (unsigned)__builtin_popcountll(mm);
        }
        __builtin_amdgcn_raw_buffer_store_b128(__builtin_bit_cast(u4, t), ro, ov + (unsigned)(((g4 >> 2) * 16 + (g4 & 3)) * GN * 4), 0, AUX_OUT);
    }
    *(LAS u64*)(d + lane * 8) = ((u64)(unsigned)mhi << 32) | (unsigned)mlo;
    WAIT_L0();
    const unsigned bytes = (base + 15u) & ~15u;
    const __amdgpu_buffer_rsrc_t rs = __builtin_amdgcn_make_buffer_rsrc(ws + PK_OFF + (size_t)pk * PKT_BYTES, 0, PKT_BYTES, 0x00020000);
    for (unsigned ofs = lane * 16u; ofs < bytes; ofs += 1024) __builtin_amdgcn_raw_buffer_store_b128(*(LAS u4*)(d + ofs), rs, ofs, 0, AUX_SYS);
    LAS unsigned* sizes = (LAS unsigned*)(lds + 8 * PKT_BYTES);
    if (lane == 0) sizes[wid] = bytes;
    asm volatile("s_waitcnt vmcnt(0) lgkmcnt(0)" ::: "memory");
    BAR();
    if (wid == 0 && lane < 56)
        SYS_STORE((u64*)(ws_of(P, (unsigned)(P.rank + 1 + (lane >> 3)) & (NR - 1)) + ready_off(P.rank, NITEM + tile * 8)) + (lane & 7), ((u64)call << 32) | sizes[lane & 7]);
    STAMP(3);
}

__global__ __launch_bounds__(NTH, 2) void fused_gemm_gather(Params P) {
    extern __shared__ __attribute__((aligned(16))) unsigned char smem[];
    LAS unsigned char* lds = (LAS unsigned char*)smem;
    const int b = blockIdx.x, lane = threadIdx.x & 63;
    const int wid = __builtin_amdgcn_readfirstlane(threadIdx.x >> 6);
    unsigned call = 0;
    if (lane == 0) call = __hip_atomic_fetch_add((unsigned*)(P.ws[P.rank] + CNT_OFF + (size_t)(b * 8 + wid) * 16), 1u, __ATOMIC_RELAXED, __HIP_MEMORY_SCOPE_AGENT) + 1u;
    STAMP(0);
    if (b < NCOMM) {
        call = (unsigned)__builtin_amdgcn_readfirstlane(call);
        comm_wg(P, b, wid, call, lane, lds);
        STAMP(2);
    } else {
        convert_phase(P, b - NCOMM, wid, call, lane, lds);
        call = (unsigned)__builtin_amdgcn_readfirstlane(call);
        STAMP(7);
        const int x = b & 7, slot = (b >> 3) - NCOMM / 8, grp = slot / 7, sub = 25 + slot - grp * 7;
        int first = x * 32 + slot, second = -1;
        if (x == 0) second = 8 * 32 + slot;
        else if (x == 1) { if (grp < 3) second = (4 + grp) * 32 + sub; }
        else if (x == 2) { if (grp < 2) second = (7 + grp) * 32 + sub; }
        else if (x >= 6 && grp < 2) { second = first; first = (x - 6 + 2 * grp) * 32 + sub; }
        for (int i = 0; i < 2; ++i) {
            const int tile = i == 0 ? first : second;
            if (tile >= 0) gemm_tile(P, tile, wid, call, lane, lds);
        }
    }
}

}

extern "C" void kernel_launch(int rank, int nranks, ncclComm_t comm, hipStream_t stream,
                              void* const* d_in, const int* in_sizes, int n_in,
                              void* d_out, int out_size, const int* in_dtypes, int out_dtype,
                              void* const* d_out_all, void* const* d_ws_all, size_t ws_bytes) {
    (void)comm; (void)in_sizes; (void)n_in; (void)out_size; (void)in_dtypes; (void)out_dtype; (void)d_out_all;
    if (nranks != NR || (unsigned)rank >= (unsigned)NR || ws_bytes < WS_NEED) { fprintf(stderr, "unsupported configuration\n"); return; }
    HIP_CHECK(hipFuncSetAttribute((const void*)fused_gemm_gather, hipFuncAttributeMaxDynamicSharedMemorySize, LDS_BYTES));
    static const int R[NR][3] = {{2, 1, 0}, {3, 4, 5}, {3, 4, 6}, {0, 1, 5}, {0, 5, 7}, {3, 6, 7}, {2, 7, 6}, {2, 1, 4}};
    static const int C[NR][3] = {{5, 2, 1}, {0, 1, 2}, {5, 3, 4}, {0, 3, 6}, {7, 4, 5}, {7, 6, 2}, {0, 3, 1}, {7, 4, 6}};
    static const int CYC[NR] = {0, 4, 5, 1, 3, 7, 2, 6};
    Params P;
    P.x = (const float*)d_in[0]; P.w = (const float*)d_in[1]; P.out = (float*)d_out; P.rank = rank;
    for (int p = 0; p < NR; ++p) P.ws[p] = (unsigned char*)d_ws_all[p];
    auto shared_patch = [&](int g, int h) {
        for (unsigned p = 0; p < 9; ++p) {
            const int r = R[g][patch_lr(p)], c = C[g][patch_lc(p)];
            if ((r == R[h][0] || r == R[h][1] || r == R[h][2]) && (c == C[h][0] || c == C[h][1] || c == C[h][2])) return p;
        }
        return 15u;
    };
    P.pos = 0;
    for (int i = 0; i < 16; ++i) P.need[i] = 0;
    int cnt[16] = {};
    bool early[16] = {};
    for (int k = 0; k < NR; ++k) {
        const int g = CYC[k];
        P.pos |= (unsigned)k << (4 * g);
        unsigned geo = (shared_patch(g, CYC[(k + 1) & 7]) << 24) | (shared_patch(g, CYC[(k + 7) & 7]) << 28);
        for (int l = 0; l < 3; ++l) {
            geo |= ((unsigned)R[g][l] << (4 * l)) | ((unsigned)C[g][l] << (4 * (3 + l)));
            P.need[R[g][l]] |= (unsigned)(g | l << 3) << (8 * cnt[R[g][l]]++);
            P.need[8 + C[g][l]] |= (unsigned)(g | l << 3) << (8 * cnt[8 + C[g][l]]++);
        }
        early[R[g][0]] = early[8 + C[g][0]] = true;
        P.geo[g] = geo;
    }
    unsigned char* order = (unsigned char*)P.order;
    int n = 0;
    for (int pass = 0; pass < 2; ++pass)
        for (int bb = 0; bb < 16; ++bb)
            for (int c = 0; c < 16; ++c)
                if (early[c] == (pass == 0)) order[n++] = (unsigned char)(c << 4 | bb);
    hipLaunchKernelGGL(fused_gemm_gather, dim3(NWG), dim3(NTH), LDS_BYTES, stream, P);
    HIP_CHECK(hipGetLastError());
}
```

```cpp
#include <hip/hip_runtime.h>
#include <rccl/rccl.h>
#include <cstdio>

#define HIP_CHECK(expr) do { hipError_t _e = (expr); if (_e != hipSuccess) { \
    fprintf(stderr, "HIP error %s at %s:%d\n", hipGetErrorString(_e), __FILE__, __LINE__); \
    return; } } while (0)

namespace {

typedef _Float16 h8 __attribute__((ext_vector_type(8)));
typedef _Float16 h2 __attribute__((ext_vector_type(2)));
typedef float f4 __attribute__((ext_vector_type(4)));
typedef float f2 __attribute__((ext_vector_type(2)));
typedef unsigned u4 __attribute__((ext_vector_type(4)));
typedef unsigned long long u64;
#define LAS __attribute__((address_space(3)))
extern "C" __device__ int llvm_writelane(int, int, int) __asm("llvm.amdgcn.writelane.i32");

constexpr int NR = 8;
constexpr int GN = 8192, KSH = 1024;
constexpr int NWG = 256, NTH = 512;
constexpr int EPL = 8;
constexpr int NCOMM = 7 * EPL;
constexpr int NGEMM = NWG - NCOMM;
constexpr int WIN = 8;
constexpr unsigned DRAW = 2;
constexpr unsigned LOWQ = 3;
constexpr unsigned FINE = 64, FINEQ = 2;
constexpr unsigned BIAS = 4;
constexpr int NUNIT = 256;
constexpr int NSOLO = 24;
constexpr int NITEM = 768;
constexpr int NPKT = 9 * 256;
constexpr int NTOT = NITEM + NPKT;
constexpr int PKT_BYTES = 512 + 16384;
constexpr int STAGE = 6 * 8192;
constexpr int LDS_BYTES = 152 * 1024;
constexpr int NSLOT = 8, NPAIR = 3;
constexpr int SLOT_BYTES = 17 * 1024;
constexpr int CTL = NSLOT * SLOT_BYTES;
constexpr int CTL_WORDS = 32 + 8 + 8 + 4 + NTOT;
static_assert(CTL + CTL_WORDS * 4 <= LDS_BYTES && 3 * STAGE <= LDS_BYTES && 8 * PKT_BYTES + 32 <= LDS_BYTES, "LDS");
static_assert(NTOT % 64 == 0 && NTOT <= 4096 && 256 % DRAW == 0 && LOWQ + DRAW < 16, "lists");
static_assert(NGEMM == 200 && NSOLO * 8 <= NGEMM && 2 * NSOLO + 2 * NGEMM >= NUNIT, "schedule");

constexpr size_t CNT_OFF = 0;
constexpr size_t TMO_OFF = 64 << 10;
constexpr size_t LQ_OFF = 96 << 10;
constexpr size_t XF_OFF = 128 << 10;
constexpr size_t WF_OFF = 160 << 10;
constexpr size_t RF_OFF = 256 << 10;
constexpr size_t X16_OFF = 1 << 20;
constexpr size_t W16_OFF = X16_OFF + (48ull << 20);
constexpr size_t S16_OFF = W16_OFF + (48ull << 20);
constexpr size_t PK_OFF = S16_OFF + (32ull << 20);
constexpr size_t WS_NEED = PK_OFF + (size_t)NPKT * PKT_BYTES + 1024;
static_assert(RF_OFF + (size_t)NR * NTOT * 8 <= X16_OFF, "flags");
constexpr u64 DEADLINE_TICKS = 50000000ull;

struct Params {
    const float* x; const float* w; float* out;
    unsigned char* ws[NR];
    int rank;
    unsigned geo[NR];
    unsigned pos;
    unsigned need[16];
    unsigned order[NUNIT / 4];
};

#define SYS_LOAD(p) __hip_atomic_load((p), __ATOMIC_RELAXED, __HIP_MEMORY_SCOPE_SYSTEM)
#define SYS_STORE(p, v) __hip_atomic_store((p), (v), __ATOMIC_RELAXED, __HIP_MEMORY_SCOPE_SYSTEM)
#define AG_LOAD(p) __hip_atomic_load((p), __ATOMIC_RELAXED, __HIP_MEMORY_SCOPE_AGENT)
#define AG_STORE(p, v) __hip_atomic_store((p), (v), __ATOMIC_RELAXED, __HIP_MEMORY_SCOPE_AGENT)
#define WAIT_V(n) asm volatile("s_waitcnt vmcnt(" #n ")" ::: "memory")
#define WAIT_L0() asm volatile("s_waitcnt lgkmcnt(0)" ::: "memory")
#define BAR() do { asm volatile("" ::: "memory"); __builtin_amdgcn_s_barrier(); asm volatile("" ::: "memory"); } while (0)
constexpr int AUX_SYS = 17;
constexpr int AUX_AGENT = 16;
constexpr int AUX_OUT = 18;

__device__ __forceinline__ bool give_up(unsigned* tmo, unsigned call, u64 t0) {
    if ((unsigned)__builtin_amdgcn_readfirstlane(AG_LOAD(tmo)) == call) return true;
    if (__builtin_amdgcn_s_memrealtime() - t0 > DEADLINE_TICKS) { AG_STORE(tmo, call); return true; }
    return false;
}
__device__ __forceinline__ unsigned char* ws_of(const Params& P, unsigned r) {
    unsigned char* p = P.ws[0];
#pragma unroll
    for (int i = 1; i < NR; ++i) p = r == (unsigned)i ? P.ws[i] : p;
    return p;
}
__host__ __device__ __forceinline__ unsigned nib(unsigned v, unsigned i) { return (v >> (4 * i)) & 15u; }

__host__ __device__ __forceinline__ int item_id(int isw, int lc, int b, int kb) {
    return (lc == 0 ? b * 16 + isw * 8 : 256 + ((lc - 1) * 2 + 1 - isw) * 128 + b * 8) + kb;
}
struct Item { int isw, lc, b, kb; };
__device__ __forceinline__ Item decode_item(int it) {
    Item r;
    if (it < 256) { r.lc = 0; r.b = it >> 4; r.isw = (it >> 3) & 1; }
    else { const int s = (it - 256) >> 7; r.lc = 1 + (s >> 1); r.isw = 1 - (s & 1); r.b = (it >> 3) & 15; }
    r.kb = it & 7;
    return r;
}
__host__ __device__ __forceinline__ unsigned patch_lr(unsigned p) { return (0x2A450u >> (2 * p)) & 3u; }
__host__ __device__ __forceinline__ unsigned patch_lc(unsigned p) { return (0x24A44u >> (2 * p)) & 3u; }
__device__ __forceinline__ size_t staged_off(int isw, int cell, int b, int kb) { return S16_OFF + (size_t)(((isw * 8 + cell) * 16 + b) * 8 + kb) * 16384; }
__device__ __forceinline__ size_t ready_off(int src, int id) { return RF_OFF + ((size_t)src * NTOT + id) * 8; }
__device__ __forceinline__ size_t packet_origin(unsigned geo, int pk) {
    const unsigned p = (unsigned)pk >> 8, tt = ((unsigned)pk >> 3) & 31u, wv = pk & 7;
    const unsigned row = (nib(geo, patch_lr(p)) & 7u) * 1024 + (tt & 3) * 256 + (wv >> 1) * 64;
    const unsigned col = (nib(geo, 3 + patch_lc(p)) & 7u) * 1024 + (tt >> 2) * 128 + (wv & 1) * 64;
    return (size_t)row * GN + col;
}

__device__ __forceinline__ unsigned pk16(float a, float b) {
    const f2 p = {a, b};
    return __builtin_bit_cast(unsigned, __builtin_convertvector(p, h2));
}
__device__ __forceinline__ void store_block(unsigned char* dst, const u4 (&h)[16], int lane, bool sys) {
    const __amdgpu_buffer_rsrc_t rs = __builtin_amdgcn_make_buffer_rsrc(dst, 0, 16384, 0x00020000);
    if (sys) {
#pragma unroll
        for (int f = 0; f < 16; ++f) __builtin_amdgcn_raw_buffer_store_b128(h[f], rs, lane * 16 + f * 1024, 0, AUX_SYS);
    } else {
#pragma unroll
        for (int f = 0; f < 16; ++f) __builtin_amdgcn_raw_buffer_store_b128(h[f], rs, lane * 16 + f * 1024, 0, AUX_AGENT);
    }
}
__device__ __forceinline__ void convert_phase(const Params& P, int j, int wid, unsigned call  , int lane, LAS unsigned char* lds) {
    if (j >= NSOLO * 8) { for (int i = 0; i < 8; ++i) __builtin_amdgcn_s_sleep(32); }
    else if (wid != 0) { BAR(); if (wid != 1) BAR(); }
    const int m = lane & 15, q = lane >> 4;
    for (int rnd = (wid < 2 && j < NSOLO * 8) ? 0 : 1; rnd < 3; ++rnd) {
        const int u = rnd == 0 ? wid * NSOLO + (j >> 3) : 2 * NSOLO + j + (rnd - 1) * NGEMM;
        if (u >= NUNIT) break;
        if (rnd == 1 && u >= 96) for (int i = 0; i < 20; ++i) __builtin_amdgcn_s_sleep(32);
        const unsigned uo = (P.order[u >> 2] >> (8 * (u & 3))) & 255u;
        const int isw = (int)(uo >> 7), cell = (int)((uo >> 4) & 7u), b = (int)(uo & 15u), kb = rnd == 0 ? j & 7 : wid;
        f4 v[32];
        {
            const int first = cell * 1024 + b * 64;
            const unsigned base = isw ? (unsigned)(kb * 128 * GN + first) * 4u : (unsigned)(first * KSH + kb * 128) * 4u;
            const unsigned voff = isw ? (unsigned)(4 * q * GN + 4 * m) * 4u : (unsigned)(m * KSH + 4 * q) * 4u;
            const __amdgpu_buffer_rsrc_t rs = __builtin_amdgcn_make_buffer_rsrc((void*)(isw ? P.w : P.x), 0, KSH * GN * 4, 0x00020000);
#pragma unroll
            for (int n = 0; n < 32; ++n) {
                const unsigned ox = ((n >> 1) & 3) * (16 * KSH * 4) + (n >> 3) * 128 + (n & 1) * 64;
                const unsigned ow = ((n >> 3) * 32 + ((n & 7) < 4 ? (n & 7) : 12 + (n & 7))) * (GN * 4);
                v[n] = __builtin_bit_cast(f4, __builtin_amdgcn_raw_buffer_load_b128(rs, voff, base + (isw ? ow : ox), 0));
            }
        }
        float fm = 0.f;
#pragma unroll
        for (int i = 0; i < 32; ++i) fm = fmaxf(fmaxf(fm, fmaxf(fabsf(v[i][0]), fabsf(v[i][1]))), fmaxf(fabsf(v[i][2]), fabsf(v[i][3])));
        unsigned mx = __builtin_bit_cast(unsigned, fm);
        if (__any(mx >= 0x7f800000u)) {
            mx = 0;
#pragma unroll
            for (int i = 0; i < 32; ++i)
#pragma unroll
                for (int c = 0; c < 4; ++c) { const unsigned t = __builtin_bit_cast(unsigned, v[i][c]) & 0x7fffffffu; mx = max(mx, t < 0x7f800000u ? t : 0u); }
        }
#pragma unroll
        for (int o = 32; o >= 1; o >>= 1) mx = max(mx, (unsigned)__shfl_xor((int)mx, o));
        const int E = (int)((unsigned)__builtin_amdgcn_readfirstlane(mx) >> 23) - 127;
        const int code = ((14 - E) >> 3) + 16;
        const int s = (code - 16) * 8;
#pragma unroll
        for (int i = 0; i < 32; ++i) {
#pragma unroll
            for (int c = 0; c < 4; ++c) v[i][c] = __builtin_amdgcn_ldexpf(v[i][c], s);
            asm volatile("" : "+v"(v[i]));
        }
        u4 h[16];
        if (!isw) {
#pragma unroll
            for (int f = 0; f < 16; ++f) { const f4 a = v[2 * f], c = v[2 * f + 1];
                h[f] = (u4){pk16(a[0], a[1]), pk16(a[2], a[3]), pk16(c[0], c[1]), pk16(c[2], c[3])};
                asm volatile("; x" : "+v"(h[f])); }
        } else {
#pragma unroll
            for (int kq = 0; kq < 4; ++kq)
#pragma unroll
                for (int i = 0; i < 4; ++i) {
                    h[kq * 4 + i] = (u4){pk16(v[kq * 8 + 0][i], v[kq * 8 + 1][i]), pk16(v[kq * 8 + 2][i], v[kq * 8 + 3][i]),
                                         pk16(v[kq * 8 + 4][i], v[kq * 8 + 5][i]), pk16(v[kq * 8 + 6][i], v[kq * 8 + 7][i])};
                    asm volatile("; w" : "+v"(h[kq * 4 + i])); }
        }
        const unsigned need = P.need[isw * 8 + cell];
        store_block(P.ws[P.rank] + staged_off(isw, cell, b, kb), h, lane, true);
        for (int d = 0; d < 3; ++d) {
            const unsigned e = need >> (8 * d);
            if ((e & 7u) == (unsigned)P.rank)
                store_block(P.ws[P.rank] + (isw ? W16_OFF : X16_OFF) + (size_t)(((int)((e >> 3) & 3u) * 16 + b) * 64 + P.rank * 8 + kb) * 16384, h, lane, false);
        }
        LAS unsigned* codes = (LAS unsigned*)(lds + rnd * 32);
        if (lane == 0) codes[wid] = (unsigned)code;
        asm volatile("s_waitcnt vmcnt(0) lgkmcnt(0)" ::: "memory");
        if (rnd != 0) BAR();
        if (wid == 0 || rnd == 0) {
            const unsigned cl = (unsigned)__builtin_amdgcn_readfirstlane(call);
            if (lane < 24 && (rnd != 0 || (lane & 7) == 0)) {
                const unsigned e = need >> (8 * (lane >> 3)), dst = e & 7u; const int lc = (int)((e >> 3) & 3u);
                const int k2 = rnd == 0 ? kb : lane & 7, id = item_id(isw, lc, b, k2);
                const unsigned c = rnd == 0 ? (unsigned)code : codes[lane & 7];
                unsigned char* dws = ws_of(P, dst);
                if (dst == (unsigned)P.rank) SYS_STORE((unsigned*)(dws + (isw ? WF_OFF : XF_OFF)) + (lc * 16 + b) * 64 + P.rank * 8 + k2, (cl << 8) | c);
                else SYS_STORE((u64*)(dws + ready_off(P.rank, id)), ((u64)cl << 32) | c);
            }
        }
        if (rnd == 0) { BAR(); if (wid == 0) BAR(); }
    }
}

__device__ __forceinline__ unsigned lds_get(LAS unsigned char* p) {
    unsigned v;
    asm volatile("ds_read_b32 %0, %1\n\ts_waitcnt lgkmcnt(0)" : "=v"(v) : "v"(p) : "memory");
    return (unsigned)__builtin_amdgcn_readfirstlane(v);
}
__device__ __forceinline__ void lds_put(LAS unsigned char* p, unsigned v) { asm volatile("ds_write_b32 %0, %1" :: "v"(p), "v"(v) : "memory"); }
__device__ __forceinline__ void lds_put64(LAS unsigned char* p, u64 v) { asm volatile("ds_write_b64 %0, %1" :: "v"(p), "v"(v) : "memory"); }
__device__ __forceinline__ void lds_inc(LAS unsigned char* p, int lane) { if (lane == 0) asm volatile("ds_add_u32 %0, %1" :: "v"(p), "v"(1u) : "memory"); }
constexpr unsigned DEAD = 0xffffffffu;

__device__ __forceinline__ void comm_wg(const Params& P, int b, int wid, unsigned call, int lane, LAS unsigned char* lds) {
    const unsigned src = (unsigned)(P.rank + 1 + b % 7) & (NR - 1);
    unsigned char* ws = P.ws[P.rank];
    LAS unsigned char* list = lds + CTL;
    LAS unsigned char* full = list + 128;
    LAS unsigned char* freed = full + 32;
    LAS unsigned char* started = freed + 32;
    LAS unsigned char* fin = started + 4;
    LAS unsigned char* info = started + 16;
    for (int i = threadIdx.x; i < CTL_WORDS; i += NTH) lds_put(list + i * 4, 0u);
    WAIT_L0();
    BAR();
    if (wid == 1) {
        __builtin_amdgcn_s_setprio(2);
        unsigned* tmo = (unsigned*)(ws + TMO_OFF);
        const unsigned dist = (nib(P.pos, src) - nib(P.pos, (unsigned)P.rank)) & 7u;
        const unsigned pn = nib(P.geo[src], 6), pp = nib(P.geo[src], 7);
        u64* q = (u64*)(ws + LQ_OFF + (lane == 0 ? (unsigned)(b % 7) : 7u + dist + (lane == 2)) * 256);
        const u64 tbase = (u64)call << 32;
        if (lane < 3) __hip_atomic_fetch_max(q, tbase, __ATOMIC_RELAXED, __HIP_MEMORY_SCOPE_AGENT);
        WAIT_V(0);
        constexpr unsigned NLIST = NITEM + 7u * 256u;
        const u64* flags = (const u64*)(ws + RF_OFF) + (size_t)src * NTOT;
        const u64 t0 = __builtin_amdgcn_s_memrealtime();
        bool dead = false, todo = true, more = true;
        unsigned drawn = 0, posted = 0, iter = 0;
        unsigned sel = 0, seenp = dist == 1u ? 256u : 0u, seenn = dist == 7u ? 256u : 0u;
        int base = 0, idq = 0;
        for (;;) {
            const bool fine = min(seenp, seenn) >= 256u - FINE;
            const unsigned dr = fine ? 1u : DRAW;
            const bool want = more && drawn - lds_get(started) <= (fine ? FINEQ : LOWQ);
            const bool poll = base < NTOT;
            u64 t64 = 0, f = 0;
            if (want && lane < 3 && (lane == (int)sel || (sel != 0u && lane != 0)))
                t64 = __hip_atomic_fetch_add(q, (u64)(lane == (int)sel ? dr : 0u), __ATOMIC_RELAXED, __HIP_MEMORY_SCOPE_AGENT);
            if (poll && todo) f = SYS_LOAD(flags + base + lane);
            if (poll) {
                if (todo) {
                    if ((unsigned)(f >> 32) == call) { lds_put(info + (base + lane) * 4, (unsigned)f); todo = false; }
                    else if (dead) { lds_put(info + (base + lane) * 4, DEAD); todo = false; }
                }
                if (!__any(todo)) { base += 64; todo = true; }
                else if ((++iter & 15u) == 0u && give_up(tmo, call, t0)) dead = true;
            }
            if (want) {
                const unsigned t = (unsigned)__builtin_amdgcn_readlane((int)t64, (int)sel);
                if (t < (sel == 0u ? NLIST : 256u)) {
                    unsigned id = t;
                    if (sel != 0u) id = NITEM + (sel == 1u ? pp : pn) * 256u + t;
                    else if (id >= (unsigned)NITEM) {
                        const unsigned lo = min(pn, pp), hi = max(pn, pp);
                        if (id >= NITEM + lo * 256u) id += 256u;
                        if (id >= NITEM + hi * 256u) id += 256u;
                    }
                    const unsigned cnt = sel != 0u ? min(dr, 256u - t) : dr;
                    for (unsigned i = 0; i < cnt; ++i) idq = llvm_writelane((int)(id + i), (int)((drawn + i) & 15u), idq);
                    drawn += cnt;
                }
                if (sel != 0u) {
                    seenp = max(seenp, (unsigned)__builtin_amdgcn_readlane((int)t64, 1) + (sel == 1u ? dr : 0u));
                    seenn = max(seenn, (unsigned)__builtin_amdgcn_readlane((int)t64, 2) + (sel == 2u ? dr : 0u));
                }
                if (sel != 0u || t + DRAW >= NLIST) {
                    const unsigned ep = seenp >= 256u ? 4096u : seenp + (dist < 4u ? (4u - dist) * BIAS : 0u);
                    const unsigned en = seenn >= 256u ? 4096u : seenn + (dist > 4u ? (dist - 4u) * BIAS : 0u);
                    sel = ep < en ? 1u : 2u; more = min(seenp, seenn) < 256u;
                }
            }
            while (posted < drawn) {
                const unsigned id = (unsigned)__builtin_amdgcn_readlane(idq, (int)(posted & 15u));
                const unsigned inf = lds_get(info + id * 4);
                if (inf == 0u) break;
                lds_put64(list + (posted & 15u) * 8, ((u64)inf << 32) | ((posted + 1u) << 12) | id);
                ++posted;
            }
            if (!more && posted == drawn) break;
            if (!want) __builtin_amdgcn_s_sleep(2);
        }
        lds_put64(list + (posted & 15u) * 8, (u64)((posted + 1u) << 12));
    } else if (wid == 0) {
        __builtin_amdgcn_s_setprio(3);
        const unsigned char* rws = P.ws[src];
        const unsigned mygeo = P.geo[P.rank];
        const unsigned voff = (unsigned)lane * 16u;
        int issued = 0, pub = 0, endv = 0, idv = 0;
        int pend = 0x7fffffff;
        int n = 0;
        auto publish = [&](int landed) {
            pend = 0x7fffffff;
            while (pub < n) {
                const int e = __builtin_amdgcn_readlane(endv, pub & 63);
                if (e > landed) { pend = e + (WIN - 1); break; }
                lds_put(full + (pub & (NSLOT - 1)) * 4, (unsigned)__builtin_amdgcn_readlane(idv, pub & 63));
                ++pub;
            }
        };
        u64 ent; unsigned fr;
        auto fetch = [&](int i) { asm volatile("ds_read_b64 %0, %2\n\tds_read_b32 %1, %3" : "=&v"(ent), "=&v"(fr) : "v"(list + (i & 15) * 8), "v"(freed + (i & (NSLOT - 1)) * 4) : "memory"); };
        fetch(0);
        for (;; ++n) {
            const unsigned want = 2u * (unsigned)(n / NSLOT);
            unsigned tag, inf;
            auto ready = [&]() -> bool {
                asm volatile("s_waitcnt lgkmcnt(0)" : "+v"(ent), "+v"(fr) :: "memory");
                tag = (unsigned)__builtin_amdgcn_readfirstlane((int)ent); inf = (unsigned)__builtin_amdgcn_readfirstlane((int)(ent >> 32));
                return (tag >> 12) == (unsigned)n + 1u && (inf == 0u || (unsigned)__builtin_amdgcn_readfirstlane((int)fr) == want);
            };
            if (!ready()) {
                fetch(n);
                if (!ready()) {
                    WAIT_V(0);
                    publish(issued);
                    do { __builtin_amdgcn_s_sleep(1); fetch(n); } while (!ready());
                }
            }
            if (inf == 0u) break;
            lds_put(started, (unsigned)n + 1u);
            const unsigned id = tag & 4095u;
            unsigned bytes; size_t off;
            if (id < (unsigned)NITEM) { const Item I = decode_item((int)id); bytes = 16384u; off = staged_off(I.isw, (int)(nib(mygeo, I.isw * 3 + I.lc) & 7u), I.b, I.kb); }
            else { bytes = min(inf, (unsigned)PKT_BYTES); off = PK_OFF + (size_t)(id - NITEM) * PKT_BYTES; }
            if (inf == DEAD) bytes = 0u;
            const __amdgpu_buffer_rsrc_t rs = __builtin_amdgcn_make_buffer_rsrc((void*)(rws + off), 0, (int)bytes, 0x00020000);
            LAS unsigned char* d = lds + (unsigned)(n & (NSLOT - 1)) * SLOT_BYTES;
            for (unsigned o = 0; o < bytes; o += 1024) {
                asm volatile("s_waitcnt vmcnt(%0)" :: "i"(WIN - 1) : "memory");
                __builtin_amdgcn_raw_ptr_buffer_load_lds(rs, (LAS void*)(d + o), 16, voff, o, 0, AUX_SYS);
                if (issued >= pend) publish(issued - (WIN - 1));
                ++issued;
                if (o + 3072 >= bytes && o + 2048 < bytes) fetch(n + 1);
            }
            if (bytes <= 2048u) fetch(n + 1);
            endv = llvm_writelane(issued, n & 63, endv);
            idv = llvm_writelane((int)tag, n & 63, idv);
            if (pub == n) pend = issued + (WIN - 1);
        }
        lds_put(fin, (unsigned)n + 1u);
        WAIT_V(0);
        publish(issued);
    } else {
        const unsigned sgeo = P.geo[src];
        const int half = wid & 1;
        const unsigned ov = (unsigned)((32 * half + 4 * (lane >> 4)) * GN + 4 * (lane & 15)) * 4u;
        int turn = 0;
        for (int n = (wid >> 1) - 1;; n += NPAIR, turn ^= 1) {
            const unsigned slot = (unsigned)(n & (NSLOT - 1));
            unsigned fw;
            bool stop = false;
            while (((fw = lds_get(full + slot * 4)) >> 12) != (unsigned)n + 1u) {
                const unsigned fn = lds_get(fin);
                if (fn != 0u && (unsigned)n + 1u >= fn) { stop = true; break; }
                __builtin_amdgcn_s_sleep(1);
            }
            if (stop) break;
            const int id = (int)(fw & 4095u);
            const unsigned inf = lds_get(info + id * 4);
            LAS unsigned char* d = lds + slot * SLOT_BYTES;
            if (id < NITEM) {
                if (turn != half) { lds_inc(freed + slot * 4, lane); continue; }
                u4 h[16];
#pragma unroll
                for (int f = 0; f < 16; ++f) h[f] = *(LAS u4*)(d + f * 1024 + lane * 16);
                WAIT_L0();
                lds_inc(freed + slot * 4, lane);
                if (inf != DEAD) {
                    const Item I = decode_item(id);
                    const int blk = (I.lc * 16 + I.b) * 64 + (int)src * 8 + I.kb;
                    store_block(ws + (I.isw ? W16_OFF : X16_OFF) + (size_t)blk * 16384, h, lane, false);
                    WAIT_V(0);
                    if (lane == 0) AG_STORE((unsigned*)(ws + (I.isw ? WF_OFF : XF_OFF)) + blk, (call << 8) | (inf & 63u));
                }
            } else {
                const __amdgpu_buffer_rsrc_t ro = __builtin_amdgcn_make_buffer_rsrc(P.out + packet_origin(sgeo, id - NITEM), 0, 63 * GN * 4 + 256, 0x00020000);
                if (inf != DEAD) {
                    const u64 mk = *(LAS u64*)(d + ((lane + 32 * half) & 63) * 8);
                    const int mlo = (int)mk, mhi = (int)(mk >> 32);
                    unsigned base = 512;
                    if (half) {
                        int pc = __builtin_popcountll(mk);
#pragma unroll
                        for (int o = 16; o >= 1; o >>= 1) pc += __shfl_xor(pc, o);
                        base += 4u * (unsigned)__builtin_amdgcn_readlane(pc, 32);
                    }
#pragma unroll
                    for (int g4 = 0; g4 < 8; ++g4) {
                        f4 t;
#pragma unroll
                        for (int ni = 0; ni < 4; ++ni) {
                            const unsigned lo = (unsigned)__builtin_amdgcn_readlane(mlo, g4 * 4 + ni), hi = (unsigned)__builtin_amdgcn_readlane(mhi, g4 * 4 + ni);
                            const u64 mm = ((u64)hi << 32) | lo;
                            const unsigned pos = __builtin_amdgcn_mbcnt_hi(hi, __builtin_amdgcn_mbcnt_lo(lo, 0u));
                            const float val = *(LAS float*)(d + base + pos * 4);
                            t[ni] = __builtin_amdgcn_inverse_ballot_w64(mm) ? val : 0.f;
                            base += 4u * (unsigned)__builtin_popcountll(mm);
                        }
                        __builtin_amdgcn_raw_buffer_store_b128(__builtin_bit_cast(u4, t), ro, ov + (unsigned)(((g4 >> 2) * 16 + (g4 & 3)) * GN * 4), 0, AUX_OUT);
                    }
                } else {
#pragma unroll
                    for (int g4 = 0; g4 < 8; ++g4)
                        __builtin_amdgcn_raw_buffer_store_b128((u4){0x7fc00000u, 0x7fc00000u, 0x7fc00000u, 0x7fc00000u}, ro, ov + (unsigned)(((g4 >> 2) * 16 + (g4 & 3)) * GN * 4), 0, 0);
                }
                WAIT_L0();
                lds_inc(freed + slot * 4, lane);
            }
        }
    }
}

__device__ __forceinline__ void gemm_tile(const Params& P, int tile, int wid, unsigned call, int lane, LAS unsigned char* lds) {
    unsigned char* ws = P.ws[P.rank];
    unsigned* tmo = (unsigned*)(ws + TMO_OFF);
    const int rb0 = (int)patch_lr((unsigned)tile >> 5) * 16 + (tile & 3) * 4;
    const int cb0 = (int)patch_lc((unsigned)tile >> 5) * 16 + ((tile & 31) >> 2) * 2;
    const unsigned* xf = (const unsigned*)(ws + XF_OFF) + rb0 * 64;
    const unsigned* wf = (const unsigned*)(ws + WF_OFF) + cb0 * 64;
    if (wid == 0) {
        const u64 t0 = __builtin_amdgcn_s_memrealtime();
        bool dead = false;
        for (int g = 0; g < 6 && !dead; ++g) {
            const unsigned* f = (g < 4 ? xf + g * 64 : wf + (g - 4) * 64) + lane;
            while (!__all((SYS_LOAD(f) >> 8) == (call & 0xffffffu))) {
                if (give_up(tmo, call, t0)) { dead = true; break; }
                __builtin_amdgcn_s_sleep(32);
            }
        }
        __builtin_amdgcn_fence(__ATOMIC_ACQUIRE, "");
        WAIT_V(0);
    }
    BAR();
    const int wr = wid >> 1, wc = wid & 1;
    const bool bad = (unsigned)__builtin_amdgcn_readfirstlane(AG_LOAD(tmo)) == call;
    const int ev = 8 * ((int)(SYS_LOAD(xf + wr * 64 + lane) & 63u) + (int)(SYS_LOAD(wf + wc * 64 + lane) & 63u) - 32);
    WAIT_V(0);

    const unsigned char* ga = ws + X16_OFF + (size_t)rb0 * (1 << 20) + wid * 1024 + lane * 16;
    const unsigned char* gb = ws + W16_OFF + (size_t)cb0 * (1 << 20) + wid * 1024 + lane * 16;
    auto stage = [&](int kt, int sb) {
        LAS unsigned char* d = lds + (unsigned)sb * STAGE + (unsigned)wid * 1024u;
        const size_t ko = (size_t)(kt & 127) * 8192;
#pragma unroll
        for (int p = 0; p < 4; ++p) __builtin_amdgcn_global_load_lds((const unsigned*)(ga + (size_t)p * (1 << 20) + ko), (LAS unsigned*)(d + p * 8192), 16, 0, 0);
#pragma unroll
        for (int p = 0; p < 2; ++p) __builtin_amdgcn_global_load_lds((const unsigned*)(gb + (size_t)p * (1 << 20) + ko), (LAS unsigned*)(d + (4 + p) * 8192), 16, 0, 0);
    };
    f4 acc[4][4];
#pragma unroll
    for (int i = 0; i < 4; ++i)
#pragma unroll
        for (int j = 0; j < 4; ++j) acc[i][j] = (f4){0.f, 0.f, 0.f, 0.f};
    const int pk = tile * 8 + wid;
    float* org = P.out + packet_origin(P.geo[P.rank], pk);
    float* o = org + (size_t)(4 * (lane >> 4)) * GN + 4 * (lane & 15);
    bool spilled = false;
    stage(0, 0); stage(1, 1);
    int sb = 0, kb = 0, ecur = 0;
    const unsigned aoff = (unsigned)wr * 8192u + lane * 16u, boff = (unsigned)(4 + wc) * 8192u + lane * 16u;
    ecur = __builtin_amdgcn_readlane(ev, 0);
    int elo = ecur, ehi = ecur;
    while (kb < 64) {
        const u64 diff = __builtin_amdgcn_ballot_w64(ev != ecur) >> kb;
        const int kend = diff ? kb + (int)__builtin_ctzll(diff) : 64;
        for (int t = 2 * kb; t < 2 * kend; ++t) {
            WAIT_V(6);
            BAR();
            const int s2 = sb == 0 ? 2 : sb - 1;
            stage(t + 2, s2);
            LAS unsigned char* s = lds + (unsigned)sb * STAGE;
            h8 a[2][4], b[2][4];
#pragma unroll
            for (int kq = 0; kq < 2; ++kq)
#pragma unroll
                for (int i = 0; i < 4; ++i) { b[kq][i] = *(LAS h8*)(s + boff + (kq * 4 + i) * 1024); a[kq][i] = *(LAS h8*)(s + aoff + (kq * 4 + i) * 1024); }
            __builtin_amdgcn_sched_barrier(0);
#pragma unroll
            for (int kq = 0; kq < 2; ++kq)
#pragma unroll
                for (int i = 0; i < 4; ++i)
#pragma unroll
                    for (int j = 0; j < 4; ++j) acc[i][j] = __builtin_amdgcn_mfma_f32_16x16x32_f16(a[kq][i], b[kq][j], acc[i][j], 0, 0, 0);
            sb = sb == 2 ? 0 : sb + 1;
        }
        kb = kend;
        if (kb < 64) {
            const int en = __builtin_amdgcn_readlane(ev, kb);
            if (max(ehi, en) - min(elo, en) <= 64) {
#pragma unroll
                for (int i = 0; i < 4; ++i)
#pragma unroll
                    for (int j = 0; j < 4; ++j)
#pragma unroll
                        for (int r = 0; r < 4; ++r) acc[i][j][r] = __builtin_amdgcn_ldexpf(acc[i][j][r], en - ecur);
                elo = min(elo, en); ehi = max(ehi, en);
            } else {
#pragma unroll
                for (int g4 = 0; g4 < 16; ++g4) {
                    f4* p = (f4*)(o + (size_t)((g4 >> 2) * 16 + (g4 & 3)) * GN);
                    f4 t;
#pragma unroll
                    for (int ni = 0; ni < 4; ++ni) t[ni] = __builtin_amdgcn_ldexpf(acc[g4 >> 2][ni][g4 & 3], -ecur);
                    if (spilled) t += *p;
                    *p = t;
                }
#pragma unroll
                for (int i = 0; i < 4; ++i)
#pragma unroll
                    for (int j = 0; j < 4; ++j) acc[i][j] = (f4){0.f, 0.f, 0.f, 0.f};
                spilled = true;
                elo = ehi = en;
            }
            ecur = en;
        }
    }
    WAIT_V(0);
    BAR();
    LAS unsigned char* d = lds + (unsigned)wid * PKT_BYTES;
    const __amdgpu_buffer_rsrc_t ro = __builtin_amdgcn_make_buffer_rsrc(org, 0, 63 * GN * 4 + 256, 0x00020000);
    const unsigned ov = (unsigned)(4 * (lane >> 4) * GN + 4 * (lane & 15)) * 4u;
    int mlo = 0, mhi = 0;
    unsigned base = 512;
#pragma unroll
    for (int g4 = 0; g4 < 16; ++g4) {
        f4 t;
#pragma unroll
        for (int ni = 0; ni < 4; ++ni) t[ni] = __builtin_amdgcn_ldexpf(acc[g4 >> 2][ni][g4 & 3], -ecur);
        if (spilled) t += *(const f4*)(o + (size_t)((g4 >> 2) * 16 + (g4 & 3)) * GN);
#pragma unroll
        for (int ni = 0; ni < 4; ++ni) {
            float val = bad ? __builtin_nanf("") : t[ni];
            const bool nz = !(val <= 0.f);
            val = nz ? val : 0.f;
            t[ni] = val;
            const u64 mm = __builtin_amdgcn_ballot_w64(nz);
            const unsigned pos = __builtin_amdgcn_mbcnt_hi((unsigned)(mm >> 32), __builtin_amdgcn_mbcnt_lo((unsigned)mm, 0u));
            if (nz) *(LAS float*)(d + base + pos * 4) = val;
            mlo = llvm_writelane((int)mm, g4 * 4 + ni, mlo);
            mhi = llvm_writelane((int)(mm >> 32), g4 * 4 + ni, mhi);
            base += 4u * (unsigned)__builtin_popcountll(mm);
        }
        __builtin_amdgcn_raw_buffer_store_b128(__builtin_bit_cast(u4, t), ro, ov + (unsigned)(((g4 >> 2) * 16 + (g4 & 3)) * GN * 4), 0, AUX_OUT);
    }
    *(LAS u64*)(d + lane * 8) = ((u64)(unsigned)mhi << 32) | (unsigned)mlo;
    WAIT_L0();
    const unsigned bytes = (base + 15u) & ~15u;
    const __amdgpu_buffer_rsrc_t rs = __builtin_amdgcn_make_buffer_rsrc(ws + PK_OFF + (size_t)pk * PKT_BYTES, 0, PKT_BYTES, 0x00020000);
    for (unsigned ofs = lane * 16u; ofs < bytes; ofs += 1024) __builtin_amdgcn_raw_buffer_store_b128(*(LAS u4*)(d + ofs), rs, ofs, 0, AUX_SYS);
    LAS unsigned* sizes = (LAS unsigned*)(lds + 8 * PKT_BYTES);
    if (lane == 0) sizes[wid] = bytes;
    asm volatile("s_waitcnt vmcnt(0) lgkmcnt(0)" ::: "memory");
    BAR();
    if (wid == 0 && lane < 56)
        SYS_STORE((u64*)(ws_of(P, (unsigned)(P.rank + 1 + (lane >> 3)) & (NR - 1)) + ready_off(P.rank, NITEM + tile * 8)) + (lane & 7), ((u64)call << 32) | sizes[lane & 7]);
}

__global__ __launch_bounds__(NTH, 2) void fused_gemm_gather(Params P) {
    extern __shared__ __attribute__((aligned(16))) unsigned char smem[];
    LAS unsigned char* lds = (LAS unsigned char*)smem;
    const int b = blockIdx.x, lane = threadIdx.x & 63;
    const int wid = __builtin_amdgcn_readfirstlane(threadIdx.x >> 6);
    unsigned call = 0;
    if (lane == 0) call = __hip_atomic_fetch_add((unsigned*)(P.ws[P.rank] + CNT_OFF + (size_t)(b * 8 + wid) * 16), 1u, __ATOMIC_RELAXED, __HIP_MEMORY_SCOPE_AGENT) + 1u;
    if (b < NCOMM) {
        call = (unsigned)__builtin_amdgcn_readfirstlane(call);
        comm_wg(P, b, wid, call, lane, lds);
    } else {
        convert_phase(P, b - NCOMM, wid, call, lane, lds);
        call = (unsigned)__builtin_amdgcn_readfirstlane(call);
        const int x = b & 7, slot = (b >> 3) - NCOMM / 8, grp = slot / 7, sub = 25 + slot - grp * 7;
        int first = x * 32 + slot, second = -1;
        if (x == 0) second = 8 * 32 + slot;
        else if (x == 1) { if (grp < 3) second = (4 + grp) * 32 + sub; }
        else if (x == 2) { if (grp < 2) second = (7 + grp) * 32 + sub; }
        else if (x >= 6 && grp < 2) { second = first; first = (x - 6 + 2 * grp) * 32 + sub; }
        for (int i = 0; i < 2; ++i) {
            const int tile = i == 0 ? first : second;
            if (tile >= 0) gemm_tile(P, tile, wid, call, lane, lds);
        }
    }
}

}

extern "C" void kernel_launch(int rank, int nranks, ncclComm_t comm, hipStream_t stream,
                              void* const* d_in, const int* in_sizes, int n_in,
                              void* d_out, int out_size, const int* in_dtypes, int out_dtype,
                              void* const* d_out_all, void* const* d_ws_all, size_t ws_bytes) {
    (void)comm; (void)in_sizes; (void)n_in; (void)out_size; (void)in_dtypes; (void)out_dtype; (void)d_out_all;
    if (nranks != NR || (unsigned)rank >= (unsigned)NR || ws_bytes < WS_NEED) { fprintf(stderr, "unsupported configuration\n"); return; }
    HIP_CHECK(hipFuncSetAttribute((const void*)fused_gemm_gather, hipFuncAttributeMaxDynamicSharedMemorySize, LDS_BYTES));
    static const int R[NR][3] = {{2, 1, 0}, {3, 4, 5}, {3, 4, 6}, {0, 1, 5}, {0, 5, 7}, {3, 6, 7}, {2, 7, 6}, {2, 1, 4}};
    static const int C[NR][3] = {{5, 2, 1}, {0, 1, 2}, {5, 3, 4}, {0, 3, 6}, {7, 4, 5}, {7, 6, 2}, {0, 3, 1}, {7, 4, 6}};
    static const int CYC[NR] = {0, 4, 5, 1, 3, 7, 2, 6};
    Params P;
    P.x = (const float*)d_in[0]; P.w = (const float*)d_in[1]; P.out = (float*)d_out; P.rank = rank;
    for (int p = 0; p < NR; ++p) P.ws[p] = (unsigned char*)d_ws_all[p];
    auto shared_patch = [&](int g, int h) {
        for (unsigned p = 0; p < 9; ++p) {
            const int r = R[g][patch_lr(p)], c = C[g][patch_lc(p)];
            if ((r == R[h][0] || r == R[h][1] || r == R[h][2]) && (c == C[h][0] || c == C[h][1] || c == C[h][2])) return p;
        }
        return 15u;
    };
    P.pos = 0;
    for (int i = 0; i < 16; ++i) P.need[i] = 0;
    int cnt[16] = {};
    bool early[16] = {};
    for (int k = 0; k < NR; ++k) {
        const int g = CYC[k];
        P.pos |= (unsigned)k << (4 * g);
        unsigned geo = (shared_patch(g, CYC[(k + 1) & 7]) << 24) | (shared_patch(g, CYC[(k + 7) & 7]) << 28);
        for (int l = 0; l < 3; ++l) {
            geo |= ((unsigned)R[g][l] << (4 * l)) | ((unsigned)C[g][l] << (4 * (3 + l)));
            P.need[R[g][l]] |= (unsigned)(g | l << 3) << (8 * cnt[R[g][l]]++);
            P.need[8 + C[g][l]] |= (unsigned)(g | l << 3) << (8 * cnt[8 + C[g][l]]++);
        }
        early[R[g][0]] = early[8 + C[g][0]] = true;
        P.geo[g] = geo;
    }
    unsigned char* order = (unsigned char*)P.order;
    int n = 0;
    for (int pass = 0; pass < 2; ++pass)
        for (int bb = 0; bb < 16; ++bb)
            for (int c = 0; c < 16; ++c)
                if (early[c] == (pass == 0)) order[n++] = (unsigned char)(c << 4 | bb);
    hipLaunchKernelGGL(fused_gemm_gather, dim3(NWG), dim3(NTH), LDS_BYTES, stream, P);
    HIP_CHECK(hipGetLastError());
}
```

```cpp
#include <hip/hip_runtime.h>
#include <rccl/rccl.h>
#include <cstdio>

#define HIP_CHECK(expr) do { hipError_t _e = (expr); if (_e != hipSuccess) { \
    fprintf(stderr, "HIP error %s at %s:%d\n", hipGetErrorString(_e), __FILE__, __LINE__); \
    return; } } while (0)

namespace {

typedef _Float16 h8 __attribute__((ext_vector_type(8)));
typedef _Float16 h2 __attribute__((ext_vector_type(2)));
typedef float f4 __attribute__((ext_vector_type(4)));
typedef float f2 __attribute__((ext_vector_type(2)));
typedef unsigned u4 __attribute__((ext_vector_type(4)));
typedef unsigned long long u64;
#define LAS __attribute__((address_space(3)))
extern "C" __device__ int llvm_writelane(int, int, int) __asm("llvm.amdgcn.writelane.i32");

constexpr int NR = 8;
constexpr int GN = 8192, KSH = 1024;
constexpr int NWG = 256, NTH = 512;
constexpr int EPL = 8;
constexpr int NCOMM = 7 * EPL;
constexpr int NGEMM = NWG - NCOMM;
constexpr int WIN = 9;
constexpr unsigned DRAW = 2;
constexpr unsigned LOWQ = 3;
constexpr unsigned FINE = 64, FINEQ = 2;
constexpr unsigned BIAS = 4;
constexpr int NUNIT = 256;
constexpr int NSOLO = 24;
constexpr int NITEM = 768;
constexpr int NPKT = 9 * 256;
constexpr int NTOT = NITEM + NPKT;
constexpr int PKT_BYTES = 512 + 16384;
constexpr int STAGE = 6 * 8192;
constexpr int LDS_BYTES = 152 * 1024;
constexpr int NSLOT = 8, NPAIR = 3;
constexpr int SLOT_BYTES = 17 * 1024;
constexpr int CTL = NSLOT * SLOT_BYTES;
constexpr int CTL_WORDS = 32 + 8 + 8 + 4 + NTOT;
static_assert(CTL + CTL_WORDS * 4 <= LDS_BYTES && 3 * STAGE <= LDS_BYTES && 8 * PKT_BYTES + 32 <= LDS_BYTES, "LDS");
static_assert(NTOT % 64 == 0 && NTOT <= 4096 && 256 % DRAW == 0 && LOWQ + DRAW < 16, "lists");
static_assert(NGEMM == 200 && NSOLO * 8 <= NGEMM && 2 * NSOLO + 2 * NGEMM >= NUNIT, "schedule");

constexpr size_t CNT_OFF = 0;
constexpr size_t TMO_OFF = 64 << 10;
constexpr size_t LQ_OFF = 96 << 10;
constexpr size_t XF_OFF = 128 << 10;
constexpr size_t WF_OFF = 160 << 10;
constexpr size_t RF_OFF = 256 << 10;
constexpr size_t X16_OFF = 1 << 20;
constexpr size_t W16_OFF = X16_OFF + (48ull << 20);
constexpr size_t S16_OFF = W16_OFF + (48ull << 20);
constexpr size_t PK_OFF = S16_OFF + (32ull << 20);
constexpr size_t WS_NEED = PK_OFF + (size_t)NPKT * PKT_BYTES + 1024;
static_assert(RF_OFF + (size_t)NR * NTOT * 8 <= X16_OFF, "flags");
constexpr u64 DEADLINE_TICKS = 50000000ull;

struct Params {
    const float* x; const float* w; float* out;
    unsigned char* ws[NR];
    int rank;
    unsigned geo[NR];
    unsigned pos;
    unsigned need[16];
    unsigned order[NUNIT / 4];
};

#define SYS_LOAD(p) __hip_atomic_load((p), __ATOMIC_RELAXED, __HIP_MEMORY_SCOPE_SYSTEM)
#define SYS_STORE(p, v) __hip_atomic_store((p), (v), __ATOMIC_RELAXED, __HIP_MEMORY_SCOPE_SYSTEM)
#define AG_LOAD(p) __hip_atomic_load((p), __ATOMIC_RELAXED, __HIP_MEMORY_SCOPE_AGENT)
#define AG_STORE(p, v) __hip_atomic_store((p), (v), __ATOMIC_RELAXED, __HIP_MEMORY_SCOPE_AGENT)
#define WAIT_V(n) asm volatile("s_waitcnt vmcnt(" #n ")" ::: "memory")
#define WAIT_L0() asm volatile("s_waitcnt lgkmcnt(0)" ::: "memory")
#define BAR() do { asm volatile("" ::: "memory"); __builtin_amdgcn_s_barrier(); asm volatile("" ::: "memory"); } while (0)
constexpr int AUX_SYS = 17;
constexpr int AUX_AGENT = 16;
constexpr int AUX_OUT = 18;

__device__ __forceinline__ bool give_up(unsigned* tmo, unsigned call, u64 t0) {
    if ((unsigned)__builtin_amdgcn_readfirstlane(AG_LOAD(tmo)) == call) return true;
    if (__builtin_amdgcn_s_memrealtime() - t0 > DEADLINE_TICKS) { AG_STORE(tmo, call); return true; }
    return false;
}
__device__ __forceinline__ unsigned char* ws_of(const Params& P, unsigned r) {
    unsigned char* p = P.ws[0];
#pragma unroll
    for (int i = 1; i < NR; ++i) p = r == (unsigned)i ? P.ws[i] : p;
    return p;
}
__host__ __device__ __forceinline__ unsigned nib(unsigned v, unsigned i) { return (v >> (4 * i)) & 15u; }

__host__ __device__ __forceinline__ int item_id(int isw, int lc, int b, int kb) {
    return (lc == 0 ? b * 16 + isw * 8 : 256 + ((lc - 1) * 2 + 1 - isw) * 128 + b * 8) + kb;
}
struct Item { int isw, lc, b, kb; };
__device__ __forceinline__ Item decode_item(int it) {
    Item r;
    if (it < 256) { r.lc = 0; r.b = it >> 4; r.isw = (it >> 3) & 1; }
    else { const int s = (it - 256) >> 7; r.lc = 1 + (s >> 1); r.isw = 1 - (s & 1); r.b = (it >> 3) & 15; }
    r.kb = it & 7;
    return r;
}
__host__ __device__ __forceinline__ unsigned patch_lr(unsigned p) { return (0x2A450u >> (2 * p)) & 3u; }
__host__ __device__ __forceinline__ unsigned patch_lc(unsigned p) { return (0x24A44u >> (2 * p)) & 3u; }
__device__ __forceinline__ size_t staged_off(int isw, int cell, int b, int kb) { return S16_OFF + (size_t)(((isw * 8 + cell) * 16 + b) * 8 + kb) * 16384; }
__device__ __forceinline__ size_t ready_off(int src, int id) { return RF_OFF + ((size_t)src * NTOT + id) * 8; }
__device__ __forceinline__ bool by_rows(unsigned p) { return (0x1CCu >> p) & 1u; }
__device__ __forceinline__ unsigned tile_rc(unsigned p, unsigned tt) {
    if (p == 0u) {
        const unsigned s = tt < 2u ? 0u : tt < 8u ? 1u : tt < 18u ? 2u : 3u, i = tt - 2u * s * s;
        return i < 2u * s ? (i >> 1) * 8u + 2u * s + (i & 1u) : s * 8u + (i - 2u * s);
    }
    return by_rows(p) ? tt : (tt & 3u) * 8u + (tt >> 2);
}
__device__ __forceinline__ size_t packet_origin(unsigned geo, int pk) {
    const unsigned p = (unsigned)pk >> 8, rc = tile_rc(p, ((unsigned)pk >> 3) & 31u), wv = pk & 7;
    const unsigned row = (nib(geo, patch_lr(p)) & 7u) * 1024 + (rc >> 3) * 256 + (wv >> 1) * 64;
    const unsigned col = (nib(geo, 3 + patch_lc(p)) & 7u) * 1024 + (rc & 7u) * 128 + (wv & 1) * 64;
    return (size_t)row * GN + col;
}

__device__ __forceinline__ unsigned pk16(float a, float b) {
    const f2 p = {a, b};
    return __builtin_bit_cast(unsigned, __builtin_convertvector(p, h2));
}
__device__ __forceinline__ void store_block(unsigned char* dst, const u4 (&h)[16], int lane, bool sys) {
    const __amdgpu_buffer_rsrc_t rs = __builtin_amdgcn_make_buffer_rsrc(dst, 0, 16384, 0x00020000);
    if (sys) {
#pragma unroll
        for (int f = 0; f < 16; ++f) __builtin_amdgcn_raw_buffer_store_b128(h[f], rs, lane * 16 + f * 1024, 0, AUX_SYS);
    } else {
#pragma unroll
        for (int f = 0; f < 16; ++f) __builtin_amdgcn_raw_buffer_store_b128(h[f], rs, lane * 16 + f * 1024, 0, AUX_AGENT);
    }
}
__device__ __forceinline__ void convert_phase(const Params& P, int j, int wid, unsigned call  , int lane, LAS unsigned char* lds) {
    if (j >= NSOLO * 8) { for (int i = 0; i < 8; ++i) __builtin_amdgcn_s_sleep(32); }
    else if (wid != 0) { BAR(); if (wid != 1) BAR(); }
    const int m = lane & 15, q = lane >> 4;
    for (int rnd = (wid < 2 && j < NSOLO * 8) ? 0 : 1; rnd < 3; ++rnd) {
        const int u = rnd == 0 ? wid * NSOLO + (j >> 3) : 2 * NSOLO + j + (rnd - 1) * NGEMM;
        if (u >= NUNIT) break;
        if (rnd == 1 && u >= 96) for (int i = 0; i < 20; ++i) __builtin_amdgcn_s_sleep(32);
        const unsigned uo = (P.order[u >> 2] >> (8 * (u & 3))) & 255u;
        const int isw = (int)(uo >> 7), cell = (int)((uo >> 4) & 7u), b = (int)(uo & 15u), kb = rnd == 0 ? j & 7 : wid;
        f4 v[32];
        {
            const int first = cell * 1024 + b * 64;
            const unsigned base = isw ? (unsigned)(kb * 128 * GN + first) * 4u : (unsigned)(first * KSH + kb * 128) * 4u;
            const unsigned voff = isw ? (unsigned)(4 * q * GN + 4 * m) * 4u : (unsigned)(m * KSH + 4 * q) * 4u;
            const __amdgpu_buffer_rsrc_t rs = __builtin_amdgcn_make_buffer_rsrc((void*)(isw ? P.w : P.x), 0, KSH * GN * 4, 0x00020000);
#pragma unroll
            for (int n = 0; n < 32; ++n) {
                const unsigned ox = ((n >> 1) & 3) * (16 * KSH * 4) + (n >> 3) * 128 + (n & 1) * 64;
                const unsigned ow = ((n >> 3) * 32 + ((n & 7) < 4 ? (n & 7) : 12 + (n & 7))) * (GN * 4);
                v[n] = __builtin_bit_cast(f4, __builtin_amdgcn_raw_buffer_load_b128(rs, voff, base + (isw ? ow : ox), 0));
            }
        }
        float fm = 0.f;
#pragma unroll
        for (int i = 0; i < 32; ++i) fm = fmaxf(fmaxf(fm, fmaxf(fabsf(v[i][0]), fabsf(v[i][1]))), fmaxf(fabsf(v[i][2]), fabsf(v[i][3])));
        unsigned mx = __builtin_bit_cast(unsigned, fm);
        if (__any(mx >= 0x7f800000u)) {
            mx = 0;
#pragma unroll
            for (int i = 0; i < 32; ++i)
#pragma unroll
                for (int c = 0; c < 4; ++c) { const unsigned t = __builtin_bit_cast(unsigned, v[i][c]) & 0x7fffffffu; mx = max(mx, t < 0x7f800000u ? t : 0u); }
        }
#pragma unroll
        for (int o = 32; o >= 1; o >>= 1) mx = max(mx, (unsigned)__shfl_xor((int)mx, o));
        mx = (unsigned)__builtin_amdgcn_readfirstlane(mx);
        const int E = mx ? (int)(mx >> 23) - 127 : 0;
        const int code = ((14 - E) >> 3) + 16;
        const int s = (code - 16) * 8;
#pragma unroll
        for (int i = 0; i < 32; ++i) {
#pragma unroll
            for (int c = 0; c < 4; ++c) v[i][c] = __builtin_amdgcn_ldexpf(v[i][c], s);
            asm volatile("" : "+v"(v[i]));
        }
        u4 h[16];
        if (!isw) {
#pragma unroll
            for (int f = 0; f < 16; ++f) { const f4 a = v[2 * f], c = v[2 * f + 1];
                h[f] = (u4){pk16(a[0], a[1]), pk16(a[2], a[3]), pk16(c[0], c[1]), pk16(c[2], c[3])};
                asm volatile("; x" : "+v"(h[f])); }
        } else {
#pragma unroll
            for (int kq = 0; kq < 4; ++kq)
#pragma unroll
                for (int i = 0; i < 4; ++i) {
                    h[kq * 4 + i] = (u4){pk16(v[kq * 8 + 0][i], v[kq * 8 + 1][i]), pk16(v[kq * 8 + 2][i], v[kq * 8 + 3][i]),
                                         pk16(v[kq * 8 + 4][i], v[kq * 8 + 5][i]), pk16(v[kq * 8 + 6][i], v[kq * 8 + 7][i])};
                    asm volatile("; w" : "+v"(h[kq * 4 + i])); }
        }
        const unsigned need = P.need[isw * 8 + cell];
        store_block(P.ws[P.rank] + staged_off(isw, cell, b, kb), h, lane, true);
        for (int d = 0; d < 3; ++d) {
            const unsigned e = need >> (8 * d);
            if ((e & 7u) == (unsigned)P.rank)
                store_block(P.ws[P.rank] + (isw ? W16_OFF : X16_OFF) + (size_t)(((int)((e >> 3) & 3u) * 16 + b) * 64 + P.rank * 8 + kb) * 16384, h, lane, false);
        }
        LAS unsigned* codes = (LAS unsigned*)(lds + rnd * 32);
        if (lane == 0) codes[wid] = (unsigned)code;
        asm volatile("s_waitcnt vmcnt(0) lgkmcnt(0)" ::: "memory");
        if (rnd != 0) BAR();
        if (wid == 0 || rnd == 0) {
            const unsigned cl = (unsigned)__builtin_amdgcn_readfirstlane(call);
            if (lane < 24 && (rnd != 0 || (lane & 7) == 0)) {
                const unsigned e = need >> (8 * (lane >> 3)), dst = e & 7u; const int lc = (int)((e >> 3) & 3u);
                const int k2 = rnd == 0 ? kb : lane & 7, id = item_id(isw, lc, b, k2);
                const unsigned c = rnd == 0 ? (unsigned)code : codes[lane & 7];
                unsigned char* dws = ws_of(P, dst);
                if (dst == (unsigned)P.rank) SYS_STORE((unsigned*)(dws + (isw ? WF_OFF : XF_OFF)) + (lc * 16 + b) * 64 + P.rank * 8 + k2, (cl << 8) | c);
                else SYS_STORE((u64*)(dws + ready_off(P.rank, id)), ((u64)cl << 32) | c);
            }
        }
        if (rnd == 0) { BAR(); if (wid == 0) BAR(); }
    }
}

__device__ __forceinline__ unsigned lds_get(LAS unsigned char* p) {
    unsigned v;
    asm volatile("ds_read_b32 %0, %1\n\ts_waitcnt lgkmcnt(0)" : "=v"(v) : "v"(p) : "memory");
    return (unsigned)__builtin_amdgcn_readfirstlane(v);
}
__device__ __forceinline__ void lds_put(LAS unsigned char* p, unsigned v) { asm volatile("ds_write_b32 %0, %1" :: "v"(p), "v"(v) : "memory"); }
__device__ __forceinline__ void lds_put64(LAS unsigned char* p, u64 v) { asm volatile("ds_write_b64 %0, %1" :: "v"(p), "v"(v) : "memory"); }
__device__ __forceinline__ void lds_inc(LAS unsigned char* p, int lane) { if (lane == 0) asm volatile("ds_add_u32 %0, %1" :: "v"(p), "v"(1u) : "memory"); }
constexpr unsigned DEAD = 0xffffffffu;

__device__ __forceinline__ void comm_wg(const Params& P, int b, int wid, unsigned call, int lane, LAS unsigned char* lds) {
    const unsigned src = (unsigned)(P.rank + 1 + b % 7) & (NR - 1);
    unsigned char* ws = P.ws[P.rank];
    LAS unsigned char* list = lds + CTL;
    LAS unsigned char* full = list + 128;
    LAS unsigned char* freed = full + 32;
    LAS unsigned char* started = freed + 32;
    LAS unsigned char* fin = started + 4;
    LAS unsigned char* info = started + 16;
    for (int i = threadIdx.x; i < CTL_WORDS; i += NTH) lds_put(list + i * 4, 0u);
    WAIT_L0();
    BAR();
    if (wid == 1) {
        __builtin_amdgcn_s_setprio(2);
        unsigned* tmo = (unsigned*)(ws + TMO_OFF);
        const unsigned dist = (nib(P.pos, src) - nib(P.pos, (unsigned)P.rank)) & 7u;
        const unsigned pn = nib(P.geo[src], 6), pp = nib(P.geo[src], 7);
        u64* q = (u64*)(ws + LQ_OFF + (lane == 0 ? (unsigned)(b % 7) : 7u + dist + (lane == 2)) * 256);
        const u64 tbase = (u64)call << 32;
        if (lane < 3) __hip_atomic_fetch_max(q, tbase, __ATOMIC_RELAXED, __HIP_MEMORY_SCOPE_AGENT);
        WAIT_V(0);
        constexpr unsigned NLIST = NITEM + 7u * 256u;
        const u64* flags = (const u64*)(ws + RF_OFF) + (size_t)src * NTOT;
        const u64 t0 = __builtin_amdgcn_s_memrealtime();
        bool dead = false, todo = true, more = true;
        unsigned drawn = 0, posted = 0, iter = 0;
        unsigned sel = 0, seenp = dist == 1u ? 256u : 0u, seenn = dist == 7u ? 256u : 0u;
        int pb = 0, base = 0, idq = 0;
        for (;;) {
            const bool fine = min(seenp, seenn) >= 256u - FINE;
            const unsigned dr = fine ? 1u : DRAW;
            const bool want = more && drawn - lds_get(started) <= (fine ? FINEQ : LOWQ);
            const bool poll = pb < NTOT / 64;
            u64 t64 = 0, f = 0;
            if (want && lane < 3 && (lane == (int)sel || (sel != 0u && lane != 0)))
                t64 = __hip_atomic_fetch_add(q, (u64)(lane == (int)sel ? dr : 0u), __ATOMIC_RELAXED, __HIP_MEMORY_SCOPE_AGENT);
            if (poll && todo) f = SYS_LOAD(flags + base + lane);
            if (poll) {
                if (todo) {
                    if ((unsigned)(f >> 32) == call) { lds_put(info + (base + lane) * 4, (unsigned)f); todo = false; }
                    else if (dead) { lds_put(info + (base + lane) * 4, DEAD); todo = false; }
                }
                if (!__any(todo)) {
                    ++pb; todo = true;
                    base = pb * 64;
                    if (pb >= (int)(NLIST / 64u)) base = NITEM + (int)((pb & 4) ? max(pn, pp) : min(pn, pp)) * 256 + (pb & 3) * 64;
                    else { if (base >= NITEM + (int)min(pn, pp) * 256) base += 256; if (base >= NITEM + (int)max(pn, pp) * 256) base += 256; }
                }
                else if ((++iter & 15u) == 0u && give_up(tmo, call, t0)) dead = true;
            }
            if (want) {
                const unsigned t = (unsigned)__builtin_amdgcn_readlane((int)t64, (int)sel);
                for (unsigned i = t; i < t + dr && i < (sel == 0u ? NLIST : 256u); ++i) {
                    unsigned id = i;
                    if (sel != 0u) {
                        const unsigned sp = sel == 1u ? pp : pn;
                        id = NITEM + sp * 256u + (by_rows(sp) ? i : ((i >> 3) & 7u) * 32u + (i >> 6) * 8u + (i & 7u));
                    } else if (id >= (unsigned)NITEM) {
                        if (id >= NITEM + min(pn, pp) * 256u) id += 256u;
                        if (id >= NITEM + max(pn, pp) * 256u) id += 256u;
                    }
                    idq = llvm_writelane((int)id, (int)(drawn & 15u), idq);
                    ++drawn;
                }
                if (sel != 0u) {
                    seenp = max(seenp, (unsigned)__builtin_amdgcn_readlane((int)t64, 1) + (sel == 1u ? dr : 0u));
                    seenn = max(seenn, (unsigned)__builtin_amdgcn_readlane((int)t64, 2) + (sel == 2u ? dr : 0u));
                }
                if (sel != 0u || t + DRAW >= NLIST) {
                    const unsigned ep = seenp >= 256u ? 4096u : seenp + (dist < 4u ? (4u - dist) * BIAS : 0u);
                    const unsigned en = seenn >= 256u ? 4096u : seenn + (dist > 4u ? (dist - 4u) * BIAS : 0u);
                    sel = ep < en ? 1u : 2u; more = min(seenp, seenn) < 256u;
                }
            }
            while (posted < drawn) {
                const unsigned id = (unsigned)__builtin_amdgcn_readlane(idq, (int)(posted & 15u));
                const unsigned inf = lds_get(info + id * 4);
                if (inf == 0u) break;
                lds_put64(list + (posted & 15u) * 8, ((u64)inf << 32) | ((posted + 1u) << 12) | id);
                ++posted;
            }
            if (!more && posted == drawn) break;
            if (!want) __builtin_amdgcn_s_sleep(2);
        }
        lds_put64(list + (posted & 15u) * 8, (u64)((posted + 1u) << 12));
    } else if (wid == 0) {
        __builtin_amdgcn_s_setprio(3);
        const unsigned char* rws = P.ws[src];
        const unsigned mygeo = P.geo[P.rank];
        const unsigned voff = (unsigned)lane * 16u;
        int issued = 0, pub = 0, endv = 0, idv = 0;
        int pend = 0x7fffffff;
        int n = 0;
        auto publish = [&](int landed) {
            pend = 0x7fffffff;
            while (pub < n) {
                const int e = __builtin_amdgcn_readlane(endv, pub & 63);
                if (e > landed) { pend = e + (WIN - 1); break; }
                lds_put(full + (pub & (NSLOT - 1)) * 4, (unsigned)__builtin_amdgcn_readlane(idv, pub & 63));
                ++pub;
            }
        };
        u64 ent; unsigned fr;
        auto fetch = [&](int i) { asm volatile("ds_read_b64 %0, %2\n\tds_read_b32 %1, %3" : "=&v"(ent), "=&v"(fr) : "v"(list + (i & 15) * 8), "v"(freed + (i & (NSLOT - 1)) * 4) : "memory"); };
        fetch(0);
        for (;; ++n) {
            const unsigned want = 2u * (unsigned)(n / NSLOT);
            unsigned tag, inf;
            auto ready = [&]() -> bool {
                asm volatile("s_waitcnt lgkmcnt(0)" : "+v"(ent), "+v"(fr) :: "memory");
                tag = (unsigned)__builtin_amdgcn_readfirstlane((int)ent); inf = (unsigned)__builtin_amdgcn_readfirstlane((int)(ent >> 32));
                return (tag >> 12) == (unsigned)n + 1u && (inf == 0u || (unsigned)__builtin_amdgcn_readfirstlane((int)fr) == want);
            };
            if (!ready()) {
                fetch(n);
                if (!ready()) {
                    WAIT_V(0);
                    publish(issued);
                    do { __builtin_amdgcn_s_sleep(1); fetch(n); } while (!ready());
                }
            }
            if (inf == 0u) break;
            lds_put(started, (unsigned)n + 1u);
            const unsigned id = tag & 4095u;
            unsigned bytes; size_t off;
            if (id < (unsigned)NITEM) { const Item I = decode_item((int)id); bytes = 16384u; off = staged_off(I.isw, (int)(nib(mygeo, I.isw * 3 + I.lc) & 7u), I.b, I.kb); }
            else { bytes = min(inf, (unsigned)PKT_BYTES); off = PK_OFF + (size_t)(id - NITEM) * PKT_BYTES; }
            if (inf == DEAD) bytes = 0u;
            const __amdgpu_buffer_rsrc_t rs = __builtin_amdgcn_make_buffer_rsrc((void*)(rws + off), 0, (int)bytes, 0x00020000);
            LAS unsigned char* d = lds + (unsigned)(n & (NSLOT - 1)) * SLOT_BYTES;
            for (unsigned o = 0; o < bytes; o += 1024) {
                asm volatile("s_waitcnt vmcnt(%0)" :: "i"(WIN - 1) : "memory");
                __builtin_amdgcn_raw_ptr_buffer_load_lds(rs, (LAS void*)(d + o), 16, voff, o, 0, AUX_SYS);
                if (issued >= pend) publish(issued - (WIN - 1));
                ++issued;
                if (o + 3072 >= bytes && o + 2048 < bytes) fetch(n + 1);
            }
            if (bytes <= 2048u) fetch(n + 1);
            endv = llvm_writelane(issued, n & 63, endv);
            idv = llvm_writelane((int)tag, n & 63, idv);
            if (pub == n) pend = issued + (WIN - 1);
        }
        lds_put(fin, (unsigned)n + 1u);
        WAIT_V(0);
        publish(issued);
    } else {
        const unsigned sgeo = P.geo[src];
        const int half = wid & 1;
        const unsigned ov = (unsigned)((32 * half + 4 * (lane >> 4)) * GN + 4 * (lane & 15)) * 4u;
        int turn = 0;
        for (int n = (wid >> 1) - 1;; n += NPAIR, turn ^= 1) {
            const unsigned slot = (unsigned)(n & (NSLOT - 1));
            unsigned fw;
            bool stop = false;
            while (((fw = lds_get(full + slot * 4)) >> 12) != (unsigned)n + 1u) {
                const unsigned fn = lds_get(fin);
                if (fn != 0u && (unsigned)n + 1u >= fn) { stop = true; break; }
                __builtin_amdgcn_s_sleep(1);
            }
            if (stop) break;
            const int id = (int)(fw & 4095u);
            const unsigned inf = lds_get(info + id * 4);
            LAS unsigned char* d = lds + slot * SLOT_BYTES;
            if (id < NITEM) {
                if (turn != half) { lds_inc(freed + slot * 4, lane); continue; }
                u4 h[16];
#pragma unroll
                for (int f = 0; f < 16; ++f) h[f] = *(LAS u4*)(d + f * 1024 + lane * 16);
                WAIT_L0();
                lds_inc(freed + slot * 4, lane);
                if (inf != DEAD) {
                    const Item I = decode_item(id);
                    const int blk = (I.lc * 16 + I.b) * 64 + (int)src * 8 + I.kb;
                    store_block(ws + (I.isw ? W16_OFF : X16_OFF) + (size_t)blk * 16384, h, lane, false);
                    WAIT_V(0);
                    if (lane == 0) AG_STORE((unsigned*)(ws + (I.isw ? WF_OFF : XF_OFF)) + blk, (call << 8) | (inf & 63u));
                }
            } else {
                const __amdgpu_buffer_rsrc_t ro = __builtin_amdgcn_make_buffer_rsrc(P.out + packet_origin(sgeo, id - NITEM), 0, 63 * GN * 4 + 256, 0x00020000);
                if (inf != DEAD) {
                    const u64 mk = *(LAS u64*)(d + ((lane + 32 * half) & 63) * 8);
                    const int mlo = (int)mk, mhi = (int)(mk >> 32);
                    unsigned base = 512;
                    if (half) {
                        int pc = __builtin_popcountll(mk);
#pragma unroll
                        for (int o = 16; o >= 1; o >>= 1) pc += __shfl_xor(pc, o);
                        base += 4u * (unsigned)__builtin_amdgcn_readlane(pc, 32);
                    }
#pragma unroll
                    for (int g4 = 0; g4 < 8; ++g4) {
                        f4 t;
#pragma unroll
                        for (int ni = 0; ni < 4; ++ni) {
                            const unsigned lo = (unsigned)__builtin_amdgcn_readlane(mlo, g4 * 4 + ni), hi = (unsigned)__builtin_amdgcn_readlane(mhi, g4 * 4 + ni);
                            const u64 mm = ((u64)hi << 32) | lo;
                            const unsigned pos = __builtin_amdgcn_mbcnt_hi(hi, __builtin_amdgcn_mbcnt_lo(lo, 0u));
                            const float val = *(LAS float*)(d + base + pos * 4);
                            t[ni] = __builtin_amdgcn_inverse_ballot_w64(mm) ? val : 0.f;
                            base += 4u * (unsigned)__builtin_popcountll(mm);
                        }
                        __builtin_amdgcn_raw_buffer_store_b128(__builtin_bit_cast(u4, t), ro, ov + (unsigned)(((g4 >> 2) * 16 + (g4 & 3)) * GN * 4), 0, AUX_OUT);
                    }
                } else {
#pragma unroll
                    for (int g4 = 0; g4 < 8; ++g4)
                        __builtin_amdgcn_raw_buffer_store_b128((u4){0x7fc00000u, 0x7fc00000u, 0x7fc00000u, 0x7fc00000u}, ro, ov + (unsigned)(((g4 >> 2) * 16 + (g4 & 3)) * GN * 4), 0, 0);
                }
                WAIT_L0();
                lds_inc(freed + slot * 4, lane);
            }
        }
    }
}

__device__ __forceinline__ void gemm_tile(const Params& P, int tile, int wid, unsigned call, int lane, LAS unsigned char* lds) {
    unsigned char* ws = P.ws[P.rank];
    unsigned* tmo = (unsigned*)(ws + TMO_OFF);
    const unsigned rc = tile_rc((unsigned)tile >> 5, (unsigned)tile & 31u);
    const int rb0 = (int)(patch_lr((unsigned)tile >> 5) * 16 + (rc >> 3) * 4);
    const int cb0 = (int)(patch_lc((unsigned)tile >> 5) * 16 + (rc & 7u) * 2);
    const unsigned* xf = (const unsigned*)(ws + XF_OFF) + rb0 * 64;
    const unsigned* wf = (const unsigned*)(ws + WF_OFF) + cb0 * 64;
    if (wid == 0) {
        const u64 t0 = __builtin_amdgcn_s_memrealtime();
        bool dead = false;
        for (int g = 0; g < 6 && !dead; ++g) {
            const unsigned* f = (g < 4 ? xf + g * 64 : wf + (g - 4) * 64) + lane;
            while (!__all((SYS_LOAD(f) >> 8) == (call & 0xffffffu))) {
                if (give_up(tmo, call, t0)) { dead = true; break; }
                __builtin_amdgcn_s_sleep(32);
            }
        }
        __builtin_amdgcn_fence(__ATOMIC_ACQUIRE, "");
        WAIT_V(0);
    }
    BAR();
    const int wr = wid >> 1, wc = wid & 1;
    const bool bad = (unsigned)__builtin_amdgcn_readfirstlane(AG_LOAD(tmo)) == call;
    const int ev = 8 * ((int)(SYS_LOAD(xf + wr * 64 + lane) & 63u) + (int)(SYS_LOAD(wf + wc * 64 + lane) & 63u) - 32);
    WAIT_V(0);

    const __amdgpu_buffer_rsrc_t ra = __builtin_amdgcn_make_buffer_rsrc(ws + X16_OFF + (size_t)rb0 * (1 << 20), 0, 4 << 20, 0x00020000);
    const __amdgpu_buffer_rsrc_t rb = __builtin_amdgcn_make_buffer_rsrc(ws + W16_OFF + (size_t)cb0 * (1 << 20), 0, 2 << 20, 0x00020000);
    const unsigned sv = (unsigned)wid * 1024u + lane * 16u;
    auto stage = [&](int kt, int sb) {
        LAS unsigned char* d = lds + (unsigned)sb * STAGE + (unsigned)wid * 1024u;
        const int ko = (kt & 127) * 8192;
#pragma unroll
        for (int p = 0; p < 4; ++p) __builtin_amdgcn_raw_ptr_buffer_load_lds(ra, (LAS void*)(d + p * 8192), 16, sv, (p << 20) + ko, 0, 0);
#pragma unroll
        for (int p = 0; p < 2; ++p) __builtin_amdgcn_raw_ptr_buffer_load_lds(rb, (LAS void*)(d + (4 + p) * 8192), 16, sv, (p << 20) + ko, 0, 0);
    };
    f4 acc[4][4];
#pragma unroll
    for (int i = 0; i < 4; ++i)
#pragma unroll
        for (int j = 0; j < 4; ++j) acc[i][j] = (f4){0.f, 0.f, 0.f, 0.f};
    const int pk = tile * 8 + wid;
    const __amdgpu_buffer_rsrc_t ro = __builtin_amdgcn_make_buffer_rsrc(P.out + packet_origin(P.geo[P.rank], pk), 0, 63 * GN * 4 + 256, 0x00020000);
    unsigned ov = (unsigned)(4 * (lane >> 4) * GN + 4 * (lane & 15)) * 4u;
    bool spilled = false;
    const unsigned aoff = (unsigned)wr * 8192u + lane * 16u, boff = (unsigned)(4 + wc) * 8192u + lane * 16u;
    h8 fa[2][8], fb[2][8];
    auto frags = [&](int r, int sbuf) {
        LAS unsigned char* s = lds + (unsigned)sbuf * STAGE;
#pragma unroll
        for (int i = 0; i < 8; ++i) { fb[r][i] = *(LAS h8*)(s + boff + i * 1024); fa[r][i] = *(LAS h8*)(s + aoff + i * 1024); }
    };
    stage(0, 0); stage(1, 1); stage(2, 2);
    WAIT_V(12);
    BAR();
    frags(0, 0);
    int sb = 0, kb = 0, ecur = __builtin_amdgcn_readlane(ev, 0);
    int elo = ecur, ehi = ecur;
    while (kb < 64) {
        const u64 diff = __builtin_amdgcn_ballot_w64(ev != ecur) >> kb;
        const int kend = diff ? kb + (int)__builtin_ctzll(diff) : 64;
        for (int t = 2 * kb; t < 2 * kend; t += 2) {
#pragma unroll
            for (int r = 0; r < 2; ++r) {
                __builtin_amdgcn_s_waitcnt(0x0076);
                BAR();
                stage(t + r + 3, sb);
                sb = sb == 2 ? 0 : sb + 1;
                frags(r ^ 1, sb);
                __builtin_amdgcn_sched_barrier(0);
#pragma unroll
                for (int kq = 0; kq < 2; ++kq)
#pragma unroll
                    for (int i = 0; i < 4; ++i)
#pragma unroll
                        for (int j = 0; j < 4; ++j) acc[i][j] = __builtin_amdgcn_mfma_f32_16x16x32_f16(fa[r][kq * 4 + i], fb[r][kq * 4 + j], acc[i][j], 0, 0, 0);
                __builtin_amdgcn_sched_barrier(0);
            }
        }
        kb = kend;
        if (kb < 64) {
            const int en = __builtin_amdgcn_readlane(ev, kb);
            if (max(ehi, en) - min(elo, en) <= 64) {
#pragma unroll
                for (int i = 0; i < 4; ++i)
#pragma unroll
                    for (int j = 0; j < 4; ++j)
#pragma unroll
                        for (int r = 0; r < 4; ++r) acc[i][j][r] = __builtin_amdgcn_ldexpf(acc[i][j][r], en - ecur);
                elo = min(elo, en); ehi = max(ehi, en);
            } else {
                asm volatile("" : "+v"(ov));
#pragma unroll
                for (int g4 = 0; g4 < 16; ++g4) {
                    const unsigned of = ov + (unsigned)(((g4 >> 2) * 16 + (g4 & 3)) * GN * 4);
                    f4 t;
#pragma unroll
                    for (int ni = 0; ni < 4; ++ni) t[ni] = __builtin_amdgcn_ldexpf(acc[g4 >> 2][ni][g4 & 3], -ecur);
                    if (spilled) t += __builtin_bit_cast(f4, __builtin_amdgcn_raw_buffer_load_b128(ro, of, 0, AUX_AGENT));
                    __builtin_amdgcn_raw_buffer_store_b128(__builtin_bit_cast(u4, t), ro, of, 0, AUX_AGENT);
                }
                WAIT_V(0);
#pragma unroll
                for (int i = 0; i < 4; ++i)
#pragma unroll
                    for (int j = 0; j < 4; ++j) acc[i][j] = (f4){0.f, 0.f, 0.f, 0.f};
                spilled = true;
                elo = ehi = en;
            }
            ecur = en;
        }
    }
    asm volatile("s_waitcnt vmcnt(0) lgkmcnt(0)" ::: "memory");
    BAR();
    LAS unsigned char* d = lds + (unsigned)wid * PKT_BYTES;
    asm volatile("" : "+v"(ov));
    int mlo = 0, mhi = 0;
    unsigned base = 512;
#pragma unroll
    for (int g4 = 0; g4 < 16; ++g4) {
        f4 t;
#pragma unroll
        for (int ni = 0; ni < 4; ++ni) t[ni] = __builtin_amdgcn_ldexpf(acc[g4 >> 2][ni][g4 & 3], -ecur);
        if (spilled) t += __builtin_bit_cast(f4, __builtin_amdgcn_raw_buffer_load_b128(ro, ov + (unsigned)(((g4 >> 2) * 16 + (g4 & 3)) * GN * 4), 0, AUX_AGENT));
#pragma unroll
        for (int ni = 0; ni < 4; ++ni) {
            float val = bad ? __builtin_nanf("") : t[ni];
            const bool nz = !(val <= 0.f);
            val = nz ? val : 0.f;
            t[ni] = val;
            const u64 mm = __builtin_amdgcn_ballot_w64(nz);
            const unsigned pos = __builtin_amdgcn_mbcnt_hi((unsigned)(mm >> 32), __builtin_amdgcn_mbcnt_lo((unsigned)mm, 0u));
            if (nz) *(LAS float*)(d + base + pos * 4) = val;
            mlo = llvm_writelane((int)mm, g4 * 4 + ni, mlo);
            mhi = llvm_writelane((int)(mm >> 32), g4 * 4 + ni, mhi);
            base += 4u * (unsigned)__builtin_popcountll(mm);
        }
        __builtin_amdgcn_raw_buffer_store_b128(__builtin_bit_cast(u4, t), ro, ov + (unsigned)(((g4 >> 2) * 16 + (g4 & 3)) * GN * 4), 0, AUX_OUT);
    }
    *(LAS u64*)(d + lane * 8) = ((u64)(unsigned)mhi << 32) | (unsigned)mlo;
    WAIT_L0();
    const unsigned bytes = (base + 15u) & ~15u;
    const __amdgpu_buffer_rsrc_t rs = __builtin_amdgcn_make_buffer_rsrc(ws + PK_OFF + (size_t)pk * PKT_BYTES, 0, PKT_BYTES, 0x00020000);
    for (unsigned ofs = lane * 16u; ofs < bytes; ofs += 1024) __builtin_amdgcn_raw_buffer_store_b128(*(LAS u4*)(d + ofs), rs, ofs, 0, AUX_SYS);
    LAS unsigned* sizes = (LAS unsigned*)(lds + 8 * PKT_BYTES);
    if (lane == 0) sizes[wid] = bytes;
    asm volatile("s_waitcnt vmcnt(0) lgkmcnt(0)" ::: "memory");
    BAR();
    if (wid == 0 && lane < 56)
        SYS_STORE((u64*)(ws_of(P, (unsigned)(P.rank + 1 + (lane >> 3)) & (NR - 1)) + ready_off(P.rank, NITEM + tile * 8)) + (lane & 7), ((u64)call << 32) | sizes[lane & 7]);
}

__global__ __launch_bounds__(NTH, 2) void fused_gemm_gather(Params P) {
    extern __shared__ __attribute__((aligned(16))) unsigned char smem[];
    LAS unsigned char* lds = (LAS unsigned char*)smem;
    const int b = blockIdx.x, lane = threadIdx.x & 63;
    const int wid = __builtin_amdgcn_readfirstlane(threadIdx.x >> 6);
    unsigned call = 0;
    if (lane == 0) call = __hip_atomic_fetch_add((unsigned*)(P.ws[P.rank] + CNT_OFF + (size_t)(b * 8 + wid) * 16), 1u, __ATOMIC_RELAXED, __HIP_MEMORY_SCOPE_AGENT) + 1u;
    if (b < NCOMM) {
        call = (unsigned)__builtin_amdgcn_readfirstlane(call);
        comm_wg(P, b, wid, call, lane, lds);
    } else {
        convert_phase(P, b - NCOMM, wid, call, lane, lds);
        call = (unsigned)__builtin_amdgcn_readfirstlane(call);
        const int x = b & 7, slot = (b >> 3) - NCOMM / 8, grp = slot / 7, sub = 25 + slot - grp * 7;
        int first = x * 32 + slot, second = -1;
        if (x == 0) second = 8 * 32 + slot;
        else if (x == 1) { if (grp < 3) second = (4 + grp) * 32 + sub; }
        else if (x == 2) { if (grp < 2) second = (7 + grp) * 32 + sub; }
        else if (x >= 6 && slot < 7) { second = first; first = (x - 6) * 32 + sub; }
        else if (x >= 6 && slot >= 18) { second = first; first = (x - 4) * 32 + 7 + slot; }
        for (int i = 0; i < 2; ++i) {
            const int tile = i == 0 ? first : second;
            if (tile >= 0) gemm_tile(P, tile, wid, call, lane, lds);
        }
    }
}

}

extern "C" void kernel_launch(int rank, int nranks, ncclComm_t comm, hipStream_t stream,
                              void* const* d_in, const int* in_sizes, int n_in,
                              void* d_out, int out_size, const int* in_dtypes, int out_dtype,
                              void* const* d_out_all, void* const* d_ws_all, size_t ws_bytes) {
    (void)comm; (void)in_sizes; (void)n_in; (void)out_size; (void)in_dtypes; (void)out_dtype; (void)d_out_all;
    if (nranks != NR || (unsigned)rank >= (unsigned)NR || ws_bytes < WS_NEED) { fprintf(stderr, "unsupported configuration\n"); return; }
    HIP_CHECK(hipFuncSetAttribute((const void*)fused_gemm_gather, hipFuncAttributeMaxDynamicSharedMemorySize, LDS_BYTES));
    static const int R[NR][3] = {{2, 1, 0}, {3, 4, 5}, {3, 4, 6}, {0, 1, 5}, {0, 5, 7}, {3, 6, 7}, {2, 7, 6}, {2, 1, 4}};
    static const int C[NR][3] = {{5, 2, 1}, {0, 1, 2}, {5, 3, 4}, {0, 3, 6}, {7, 4, 5}, {7, 6, 2}, {0, 3, 1}, {7, 4, 6}};
    static const int CYC[NR] = {0, 4, 5, 1, 3, 7, 2, 6};
    Params P;
    P.x = (const float*)d_in[0]; P.w = (const float*)d_in[1]; P.out = (float*)d_out; P.rank = rank;
    for (int p = 0; p < NR; ++p) P.ws[p] = (unsigned char*)d_ws_all[p];
    auto shared_patch = [&](int g, int h) {
        for (unsigned p = 0; p < 9; ++p) {
            const int r = R[g][patch_lr(p)], c = C[g][patch_lc(p)];
            if ((r == R[h][0] || r == R[h][1] || r == R[h][2]) && (c == C[h][0] || c == C[h][1] || c == C[h][2])) return p;
        }
        return 15u;
    };
    P.pos = 0;
    for (int i = 0; i < 16; ++i) P.need[i] = 0;
    int cnt[16] = {};
    bool early[16] = {};
    for (int k = 0; k < NR; ++k) {
        const int g = CYC[k];
        P.pos |= (unsigned)k << (4 * g);
        unsigned geo = (shared_patch(g, CYC[(k + 1) & 7]) << 24) | (shared_patch(g, CYC[(k + 7) & 7]) << 28);
        for (int l = 0; l < 3; ++l) {
            geo |= ((unsigned)R[g][l] << (4 * l)) | ((unsigned)C[g][l] << (4 * (3 + l)));
            P.need[R[g][l]] |= (unsigned)(g | l << 3) << (8 * cnt[R[g][l]]++);
            P.need[8 + C[g][l]] |= (unsigned)(g | l << 3) << (8 * cnt[8 + C[g][l]]++);
        }
        early[R[g][0]] = early[8 + C[g][0]] = true;
        P.geo[g] = geo;
    }
    unsigned char* order = (unsigned char*)P.order;
    int n = 0;
    for (int pass = 0; pass < 2; ++pass)
        for (int bb = 0; bb < 16; ++bb)
            for (int c = 0; c < 16; ++c)
                if (early[c] == (pass == 0)) order[n++] = (unsigned char)(c << 4 | bb);
    hipLaunchKernelGGL(fused_gemm_gather, dim3(NWG), dim3(NTH), LDS_BYTES, stream, P);
    HIP_CHECK(hipGetLastError());
}
```
